```python
import numpy as np

kernel_name = "gpu_gemm_ar_m4096_k4096_n16384_e4m3e5m2_dqsilu"


ABSMAX_RTOL = 0.05

COMPUTE_DTYPE = "float32"
EPILOGUE = "dequant_silu"
IN_QUANT = None
OUT_QUANT = None
IN_FP8 = "e4m3e5m2"
IN_DTYPES = {"x": "e4m3", "w_mat": "e5m2",
             "scale_x": "float32", "scale_w": "float32"}
WORLD_SIZE = 8

def _snap_fp8(x, mant_bits, max_abs, min_normal):
    a = np.abs(x.astype(np.float64))
    _m, _e = np.frexp(a)
    q = np.where(a >= min_normal, np.ldexp(1.0, _e - 1 - mant_bits),
                 min_normal * 2.0 ** -mant_bits)
    snapped = np.minimum(np.round(a / q) * q, max_abs)
    return (np.sign(x) * snapped).astype(np.float32)

def setup_inputs():

    x = _snap_fp8(np.random.randn(4096, 4096).astype(np.float32), 3, 448.0, 2.0 ** -6)
    w_mat = _snap_fp8(np.random.randn(4096, 16384).astype(np.float32), 2, 57344.0, 2.0 ** -14)
    scale_x = (0.02 * (1.0 + np.random.rand(1))).astype(np.float32)
    scale_w = (0.02 * (1.0 + np.random.rand(1))).astype(np.float32)
    return {"x": x, "w_mat": w_mat, "scale_x": scale_x, "scale_w": scale_w}

def reference(x, w_mat, scale_x, scale_w):
    acc = (x @ w_mat).astype(np.float32)
    y = (acc * (scale_x * scale_w)).astype(np.float32)
    return (y / (1.0 + np.exp(-np.clip(y, -60.0, 60.0)))).astype(np.float32)
```

```cpp
#include <hip/hip_runtime.h>
#include <rccl/rccl.h>
#include <cstdio>

#define HIP_CHECK(expr) do { hipError_t _e = (expr); if (_e != hipSuccess) { \
    fprintf(stderr, "HIP error %s at %s:%d\n", hipGetErrorString(_e), __FILE__, __LINE__); \
    return; } } while (0)

namespace {
typedef unsigned u32x4 __attribute__((ext_vector_type(4)));
typedef unsigned u32x2 __attribute__((ext_vector_type(2)));
#define LAS __attribute__((address_space(3)))
#define DI __device__ __forceinline__

constexpr int M = 4096, KS = 512, N = 16384, NR = 8;
constexpr int PIECE = 8192;
constexpr int NPC = 320;
constexpr int SLAB = NPC * PIECE;
constexpr unsigned CNT_OFF = 0, TMO_OFF = 32768, TKT_OFF = 33024, RDY_OFF = 34816, SIG_OFF = 40960, DONE_OFF = 45056, FLAG_OFF = 65536, DATA_OFF = 1u << 20;
constexpr unsigned FWD_OFF = DATA_OFF + 64u * SLAB, WS_USED = FWD_OFF + (2u << 20);
constexpr unsigned OOB = 0x80000000u;
#define RBASE 2
#define RSTEP 1
#define RBITS "sc1"
#define CEVEN 2
constexpr int NCOMM = 16, NCS = 15, GRID = 256;
constexpr int TAB_OFF = 131072, LDS_BYTES = TAB_OFF + 2048;
constexpr unsigned DEADLINE = 30000000u;

#define IOV 7
constexpr int NFIX = 6, IO = IOV;
constexpr __host__ __device__ int ngroups(int cs) { return cs < 2 ? 17 : 16; }
constexpr __host__ __device__ int group(int cs, int i, int rank) { return cs >= 2 || i < IO ? i * NCS + cs : i == IO ? 240 + 2 * rank + cs : (i - 1) * NCS + cs; }
constexpr unsigned NSEND = 256 + 4 * 240;
struct alignas(8) SendTable { unsigned e[2 * (NSEND + 256)]; int n, start[20]; };
constexpr SendTable make_table() {
  SendTable t{}; int x = 0;
  auto put = [&](int c, int pc) { t.e[2 * t.n] = (unsigned)c * 8 * SLAB + pc * PIECE; t.e[2 * t.n + 1] = (c * NPC + pc) * 32; ++t.n; };
  for (; x < NFIX * NCOMM; ++x) put(x >> 6, x & 63);
  for (int p = 0; p < 20; ++p) {
    t.start[p] = t.n;
    for (; x < 256 && x < 64 * (p + 1); ++x) put(x >> 6, x & 63);
    for (int k = 0; k < NCS; ++k) {
      const int cs = k < 2 ? k : NCS + 1 - k;
      for (int i = p - 3; i <= p; ++i) if (i >= 0 && i < ngroups(cs) && group(cs, i, 0) < 240) put(p & 3, 64 + group(cs, i, 0));
    }
  }
  for (unsigned i = 0; i < 256; ++i) { t.e[2 * (NSEND + i)] = (i >> 4) * 131072 + (i & 15) * 4096; t.e[2 * (NSEND + i) + 1] = (i * 16 * N + 240 * 64) * 4; }
  return t;
}
static_assert(make_table().n == NSEND, "every piece is sent once");
__constant__ const SendTable d_send = make_table();

struct Params { const unsigned char* x; const unsigned char* w; const float* sx; const float* sw; float* out; char* me; char* dst[13]; char* odst[13]; long rank; };

DI unsigned rfl(unsigned v) { return __builtin_amdgcn_readfirstlane(v); }
DI unsigned now() { return (unsigned)__builtin_amdgcn_s_memrealtime(); }

DI void tr4(unsigned a, unsigned b, unsigned c, unsigned d, unsigned& o0, unsigned& o1, unsigned& o2, unsigned& o3) {
  const unsigned t0 = __builtin_amdgcn_perm(b, a, 0x05010400u), t1 = __builtin_amdgcn_perm(b, a, 0x07030602u);
  const unsigned t2 = __builtin_amdgcn_perm(d, c, 0x05010400u), t3 = __builtin_amdgcn_perm(d, c, 0x07030602u);
  o0 = __builtin_amdgcn_perm(t2, t0, 0x05040100u); o1 = __builtin_amdgcn_perm(t2, t0, 0x07060302u);
  o2 = __builtin_amdgcn_perm(t3, t1, 0x05040100u); o3 = __builtin_amdgcn_perm(t3, t1, 0x07060302u);
}

DI void retile(const Params& p, char* ws, LAS char* lds, int c, int pc, int lane, unsigned cv) {
  u32x4 v[16];
  if (pc < 64) {
    const unsigned char* src = p.x + (size_t)(pc * 64 + (lane & 15)) * KS + c * 128 + (lane >> 4) * 16;
#pragma unroll
    for (int j = 0; j < 16; ++j) v[j] = *(const u32x4*)(src + (j >> 1) * 16 * KS + (j & 1) * 64);
  } else {
    const int kq = lane >> 3, cq = lane & 7;
    const unsigned char* src = p.w + (size_t)(c * 128 + kq * 16) * N + (pc - 64) * 64 + cq * 16;
#pragma unroll
    for (int i = 0; i < 16; ++i) v[i] = *(const u32x4*)(src + (size_t)i * N);
    LAS char* dst = lds + (cq >> 2) * 8192 + (cq & 3) * 64 + kq * 256;
#pragma unroll
    for (int w = 0; w < 4; ++w) {
      unsigned t[4][4];
#pragma unroll
      for (int q = 0; q < 4; ++q) tr4(v[4 * q][w], v[4 * q + 1][w], v[4 * q + 2][w], v[4 * q + 3][w], t[q][0], t[q][1], t[q][2], t[q][3]);
#pragma unroll
      for (int j = 0; j < 4; ++j) *(LAS u32x4*)(dst + j * 2048 + w * 16) = (u32x4){t[0][j], t[1][j], t[2][j], t[3][j]};
    }
    asm volatile("s_waitcnt lgkmcnt(0)" ::: "memory");
#pragma unroll
    for (int j = 0; j < 16; ++j) v[j] = *(const LAS u32x4*)(lds + j * 1024 + lane * 16);
  }
  const unsigned call = rfl(cv) + 1u, dbase = DATA_OFF + (call & 1u) * 32u * SLAB;
  char* dst = ws + dbase + (size_t)(8 * c) * SLAB + pc * PIECE + lane * 16;
#pragma unroll
  for (int j = 0; j < 16; ++j) asm volatile("global_store_dwordx4 %0, %1, off " RBITS "\n\ts_nop 1" :: "v"(dst + j * 1024), "v"(v[j]) : "memory");
  asm volatile("s_waitcnt vmcnt(0)" ::: "memory");
  if (lane < 2) __hip_atomic_store((unsigned*)(ws + FLAG_OFF) + (c * NPC + pc + lane) * 8, call, __ATOMIC_RELAXED, __HIP_MEMORY_SCOPE_SYSTEM);
  if (lane == 0) __hip_atomic_fetch_add((unsigned*)(ws + RDY_OFF) + ((pc >> 1) & 15) * 32, 1u, __ATOMIC_RELAXED, __HIP_MEMORY_SCOPE_AGENT);
}

constexpr unsigned NALL = NSEND + 256, DRAWS = NALL - NFIX * NCOMM + 3 * NCOMM;
#define ST1(d, p, o) "global_store_dwordx4 %[o], %[" d "], %[" p "] " o " sc0 sc1\n\t"
#define ST7(d, o) ST1(d, "p0", o) ST1(d, "p1", o) ST1(d, "p2", o) ST1(d, "p3", o) ST1(d, "p4", o) ST1(d, "p5", o) ST1(d, "p6", o)
#define FBITS "sc0 sc1"
#define ST1O(o, d, p) "global_store_dwordx4 %[" o "], %[" d "], %[" p "] " FBITS "\n\t"
#define ST7O(o, d) ST1O(o, d, "p0") ST1O(o, d, "p1") ST1O(o, d, "p2") ST1O(o, d, "p3") ST1O(o, d, "p4") ST1O(o, d, "p5") ST1O(o, d, "p6")
#define PEERS [p0] "s"(dst[0]), [p1] "s"(dst[1]), [p2] "s"(dst[2]), [p3] "s"(dst[3]), [p4] "s"(dst[4]), [p5] "s"(dst[5]), [p6] "s"(dst[6])
DI unsigned push_role(const Params& p, LAS char* lds, int cb, int wid, int lane, unsigned cnt) {
  char* ws = p.me; const unsigned char* px = p.x;
  LAS unsigned* box = (LAS unsigned*)(lds + TAB_OFF);
  auto fixed = [&](int s) { const unsigned n = cb + NCOMM * s; return (u32x2){(n >> 6) * 8 * SLAB + (n & 63u) * PIECE, ((n >> 6) * NPC + (n & 63u)) * 32}; };
  char* dst[7];
#pragma unroll
  for (int i = 0; i < 7; ++i) dst[i] = p.dst[(cb * 4 + wid * 2) % 7 + i];
  static_assert(NFIX == 6 && NCOMM == 16, "the fixed pieces: cb, cb + 16, cb + 32, cb + 48 of layer 0 and cb, cb + 16 of layer 1");
  u32x4 fx[NFIX][2], cur, cur2, nxt, nxt2; unsigned cv, lo;
  { const unsigned a0 = (unsigned)((cb * 64 + wid * 16 + (lane & 15)) * KS + (lane >> 4) * 16);
#define LD2(i, a, o0, o1) "global_load_dwordx4 %[f" i "a], %[" a "], %[x] offset:" o0 "\n\tglobal_load_dwordx4 %[f" i "b], %[" a "], %[x] offset:" o1 "\n\t"
    asm volatile("global_load_dword %[cv], %[cnt], %[ws] sc1\n\t" LD2("0", "a0", "0", "64") LD2("1", "a1", "0", "64") LD2("2", "a2", "0", "64") LD2("3", "a3", "0", "64")
                 LD2("4", "a0", "128", "192") LD2("5", "a1", "128", "192") "s_waitcnt vmcnt(10)\n\t"
                 "v_and_b32 %[o], 1, %[cv]\n\tv_xor_b32 %[o], 1, %[o]\n\tv_mul_u32_u24 %[o], 0x500000, %[o]\n\tv_lshl_add_u32 %[o], %[o], 4, %[b]\n\t"
                 ST7("f0a", "") ST7("f0b", "offset:1024") "s_nop 1\n\ts_waitcnt vmcnt(14)"
                 : [cv] "=&v"(cv), [o] "=&v"(lo), [f0a] "=&v"(fx[0][0]), [f0b] "=&v"(fx[0][1]), [f1a] "=&v"(fx[1][0]), [f1b] "=&v"(fx[1][1]), [f2a] "=&v"(fx[2][0]), [f2b] "=&v"(fx[2][1]),
                   [f3a] "=&v"(fx[3][0]), [f3b] "=&v"(fx[3][1]), [f4a] "=&v"(fx[4][0]), [f4b] "=&v"(fx[4][1]), [f5a] "=&v"(fx[5][0]), [f5b] "=&v"(fx[5][1])
                 : [cnt] "v"(cnt), [ws] "s"(ws), [x] "s"(px), [a0] "v"(a0), [a1] "v"(a0 + 16 * 64 * KS), [a2] "v"(a0 + 32 * 64 * KS), [a3] "v"(a0 + 48 * 64 * KS),
                   [b] "v"(DATA_OFF + cb * PIECE + wid * 2048 + lane * 16), PEERS : "memory");
    static_assert(32u * SLAB == 0x500000u << 4, "size of a staging buffer");
    lo -= cb * PIECE; }
  const unsigned call = rfl(cv) + 1u;
  char* fl = dst[0];
#pragma unroll
  for (int i = 1; i < 7; ++i) fl = lane == i ? dst[i] : fl;
  const unsigned myj = ((cb * 4 + wid * 2) % 7 + lane) % 7u, rank = p.rank;
  fl += FLAG_OFF + (7 - myj) * 4 - (long)(7 - myj) * SLAB;
  const unsigned tbase = (call - 1u) * DRAWS - NFIX * NCOMM;
  const unsigned ro = RDY_OFF + (lane & 15) * 128, rneed = call * (NPC / 8);
#define DRAW "s_cmp_lg_u32 %[w], 0\n\ts_cbranch_scc1 1f\n\ts_mov_b64 exec, 1\n\tglobal_atomic_add %[tk], %[ta], %[one], %[ws] sc0\n\ts_mov_b64 exec, -1\n1:\n\t"
#define PEEK "global_load_dword %[rd], %[ro], %[ws] sc1\n\t"
#define LOOKUP "ds_read_b32 %[tm], %[bx]\n\ts_waitcnt lgkmcnt(0)\n\tv_readfirstlane_b32 %[t], %[tm]\n\ts_min_u32 %[ts], %[t], %[last]\n\ts_lshl_b32 %[ts], %[ts], 3\n\ts_load_dwordx2 %[e], %[tab], %[ts]\n\t"
#define NEXTO(o) "global_load_dwordx4 %[n], %[so], %[ws] sc0 sc1\n\tglobal_load_dwordx4 %[n2], %[so], %[ws] offset:" o " sc0 sc1\n\t"
#define NEXT NEXTO("1024")
#define OUTS [tk] "=&v"(tk), [t] "=&s"(t), [tm] "=&v"(tm), [ts] "=&s"(ts), [e] "=&s"(e2)
#define INS [d] "v"(cur), [d2] "v"(cur2), [ta] "v"(TKT_OFF), [one] "v"(1u), [ws] "s"(ws), [bx] "v"(TAB_OFF + 32 + ((s + 1) & 1) * 4), [tab] "s"(d_send.e), [last] "i"(NALL - 1)
  u32x2 e0 = fixed(1), e1 = fixed(2), e2; unsigned fprev = fixed(0).y, tk, t, tm, ts, rd; int s = 1;
  cur = fx[1][0]; cur2 = fx[1][1];
  auto publish = [&](unsigned f) { if (lane < 7) asm volatile("global_store_dword %0, %1, off sc0 sc1" :: "v"(fl + f), "v"(call) : "memory"); };
  auto rest = [&]() {
    t = rfl(t); e2.x = rfl(e2.x); e2.y = rfl(e2.y);
    if (wid == 0 && lane == 0) box[8 + (s & 1)] = tk - tbase;
    asm volatile("s_waitcnt lgkmcnt(0)\n\ts_barrier" ::: "memory");
    if (wid == (s & 3)) publish(fprev);
    fprev = e0.y;
  };
#pragma unroll
  for (; s < NFIX - 1; ++s) {
    asm volatile(DRAW PEEK ST7("d", "") LOOKUP ST7("d2", "offset:1024") "s_nop 1\n\ts_waitcnt vmcnt(14) lgkmcnt(0)"
                 : OUTS, [rd] "=&v"(rd) : [o] "v"(lo + e0.x), INS, PEERS, [w] "s"(s >= NFIX - 3 ? wid : 1), [ro] "v"(s == NFIX - 2 ? ro : TMO_OFF) : "memory", "scc");
    const unsigned mine = __all(rd >= rneed) ? 0u : 1u;
    if (lane == 0) box[wid] = mine;
    rest();
    cur = fx[s + 1][0]; cur2 = fx[s + 1][1]; e0 = e1; e1 = s < NFIX - 2 ? fixed(s + 2) : e2;
  }
  auto expired = [&](unsigned t0) { return rfl(__hip_atomic_load((unsigned*)(ws + TMO_OFF), __ATOMIC_RELAXED, __HIP_MEMORY_SCOPE_AGENT)) == call || now() - t0 > DEADLINE; };
  auto giveup = [&]() { __hip_atomic_store((unsigned*)(ws + TMO_OFF), call, __ATOMIC_RELAXED, __HIP_MEMORY_SCOPE_AGENT); p.out[(240 + 2 * rank) * 64] = __builtin_nanf(""); };
  auto await = [&](unsigned off, unsigned need) -> bool {
    const unsigned t0 = now();
    for (;;) {
      __syncthreads();
      const bool ok = __hip_atomic_load((const unsigned*)(ws + off), __ATOMIC_RELAXED, __HIP_MEMORY_SCOPE_SYSTEM) >= need;
      const unsigned mine = (__all(ok) ? 0u : 1u) | (expired(t0) ? 2u : 0u);
      if (lane == 0) box[wid] = mine;
      __syncthreads();
      const unsigned v = box[lane & 3];
      if (!__any(v & 1u)) return true;
      if (__any(v & 2u)) { giveup(); return false; }
      __builtin_amdgcn_s_sleep(2);
    }
  };
  if (__any(box[lane & 3]) && !await(ro, rneed)) return cv;
  const unsigned xl = (IO + (cb >> 1)) & 3, xp = 64 + 240 + (cb & 1), rot = (cb * 4 + wid * 2) % 7;
  auto owner = [&](unsigned i) { return (rank + 1 + (rot + i) % 7u) & 7u; };
  constexpr unsigned X0 = make_table().start[IO], X1 = make_table().start[IO + 1], X2 = make_table().start[IO + 2], X3 = make_table().start[IO + 3];
  unsigned xt = cb < 2 ? X0 - 24 : cb < 4 ? X1 - 24 : cb < 6 ? X2 - 24 : cb < 8 ? X3 - 24 : ~0u, stage = 0, t1 = t;
  auto extra = [&]() {
    if (stage == 0) {
      const __amdgpu_buffer_rsrc_t rws = __builtin_amdgcn_make_buffer_rsrc(ws, 0, WS_USED, 0x00020000);
#pragma unroll 1
      for (unsigned i = 0; i < 7; ++i)
        asm volatile("s_mov_b32 m0, %2\n\ts_nop 0\n\tbuffer_load_dwordx4 %0, %1, %3 offen sc0 sc1 lds\n\tbuffer_load_dwordx4 %0, %1, %3 offen offset:1024 sc0 sc1 lds"
                     :: "v"(lo), "s"(rws), "s"(wid * 32768 + i * 2048), "s"(xl * 8 * SLAB + (xp + 2 * owner(i)) * PIECE) : "memory");
    } else if (stage == 1) {
#pragma unroll
      for (int i = 0; i < 7; ++i) {
        const u32x4 a = *(const LAS u32x4*)(lds + wid * 32768 + i * 2048 + lane * 16), b = *(const LAS u32x4*)(lds + wid * 32768 + i * 2048 + 1024 + lane * 16);
        asm volatile(ST1("d", "p0", "") ST1("d2", "p0", "offset:1024") "s_nop 1" :: [o] "v"(lo + xl * 8 * SLAB + (xp + 2 * owner(i)) * PIECE), [d] "v"(a), [d2] "v"(b), [p0] "s"(dst[i]) : "memory");
      }
    } else { if (wid == 0) publish((xl * NPC + xp + 2 * owner(lane)) * 32); xt = ~0u; }
    ++stage;
  };
  auto shift = [&]() { cur = nxt; cur2 = nxt2; e0 = e1; e1 = e2; ++s; };
#define STEP(from, second) do { asm volatile(DRAW NEXTO(second) ST7("d", "") LOOKUP ST7("d2", "offset:1024") "s_nop 1\n\ts_waitcnt vmcnt(14) lgkmcnt(0)" \
                 : [n] "=&v"(nxt), [n2] "=&v"(nxt2), OUTS : [so] "v"(from + e1.x), [o] "v"(lo + e0.x), INS, PEERS, [w] "s"(wid) : "memory", "scc"); rest(); } while (0)
  for (; t1 < NSEND - 64; t1 = t) {
    STEP(lo, "1024");
    if (t >= xt) extra();
    shift();
  }
  while (xt != ~0u) { asm volatile("s_waitcnt vmcnt(0)\n\ts_barrier" ::: "memory"); extra(); }
  const unsigned so = SIG_OFF + (wid & 1) * 256 + lane * 4, fo = FWD_OFF + wid * 1024 + ((lane >> 4) & 1) * 65536 + (lane >> 5) * 256 + (lane & 15) * 16;
  bool late = true;
  if (t1 < NSEND) {
    asm volatile(DRAW PEEK NEXT ST7("d", "") LOOKUP ST7("d2", "offset:1024") "s_nop 1\n\ts_waitcnt vmcnt(14) lgkmcnt(0)"
                 : [n] "=&v"(nxt), [n2] "=&v"(nxt2), OUTS, [rd] "=&v"(rd) : [so] "v"(lo + e1.x), [o] "v"(lo + e0.x), INS, PEERS, [w] "s"(wid), [ro] "v"(so) : "memory", "scc");
    { const unsigned mine = __all(rd >= call) ? 0u : 1u; if (lane == 0) box[wid] = mine; }
    rest(); shift(); t1 = t;
    late = rfl(__any(box[lane & 3])) != 0u;
  }
  for (; t1 < NSEND; t1 = t) { STEP(lo, "1024"); shift(); }
  STEP(fo, "512");
  if (late) {
    asm volatile("s_waitcnt vmcnt(0)\n\ts_barrier" ::: "memory");
    if (wid == 0) publish(fprev);
    if (!await(so, call)) return cv;
    asm volatile(NEXTO("512") "s_waitcnt vmcnt(0)" : [n] "=&v"(nxt), [n2] "=&v"(nxt2) : [so] "v"(fo + e1.x), [ws] "s"(ws) : "memory");
  }
  shift();
#pragma unroll
  for (int i = 0; i < 7; ++i) dst[i] = p.odst[rot + i];
  const unsigned oo = (unsigned)((wid + (lane >> 5) * 4) * N + rank * 128 + (lane & 31) * 4) * 4u;
  unsigned t0 = t1; t1 = t; bool owed = true;
  for (; t0 < NALL; t0 = t1, t1 = t) {
    asm volatile(DRAW NEXTO("512") ST7O("o", "d") LOOKUP "v_add_u32 %[tm], 0x80000, %[o]\n\t" ST7O("tm", "d2") "s_nop 1\n\ts_waitcnt vmcnt(14) lgkmcnt(0)"
                 : [n] "=&v"(nxt), [n2] "=&v"(nxt2), OUTS : [so] "v"(fo + e1.x), [o] "v"(oo + e0.y), INS, PEERS, [w] "s"(wid) : "memory", "scc");
    t = rfl(t); e2.x = rfl(e2.x); e2.y = rfl(e2.y);
    if (wid == 0 && lane == 0) box[8 + (s & 1)] = tk - tbase;
    asm volatile("s_waitcnt lgkmcnt(0)\n\ts_barrier" ::: "memory");
    if (owed && wid == 1) publish(fprev);
    owed = false;
    shift();
  }
  asm volatile("s_waitcnt vmcnt(0)" ::: "memory");
  if (lane < 7) asm volatile("global_store_dword %0, %1, off sc0 sc1" :: "v"(fl + (DONE_OFF + cb * 128 + wid * 32) - FLAG_OFF), "v"(call) : "memory");
  if (owed) { __builtin_amdgcn_s_barrier(); if (wid == 1) publish(fprev); }
  { const unsigned w0 = now(); const unsigned* f = (const unsigned*)(ws + DONE_OFF + cb * 128 + wid * 32) + 1 + lane % 7;
    while (!__all(__hip_atomic_load(f, __ATOMIC_RELAXED, __HIP_MEMORY_SCOPE_SYSTEM) >= call)) {
      if (expired(w0)) { giveup(); break; }
      __builtin_amdgcn_s_sleep(1);
    } }
  return cv;
}

#define STBITS "nt sc1"
#define PSLEEP 8
#define XSTR2(x) #x
#define XSTR(x) XSTR2(x)
#define GEMM_CLOBBERS \
  "v0","v1","v2","v3","v4","v5","v6","v7","v8","v9","v10","v11","v12","v13","v14","v15","v16","v17","v18","v19","v20","v21","v22","v23","v24","v25","v26","v27","v28","v29","v30","v31", \
  "v32","v33","v34","v35","v36","v37","v38","v39","v40","v41","v42","v43","v44","v45","v46","v47","v48","v49","v50","v51","v52","v53","v54","v55","v56","v57","v58","v59","v60","v61","v62","v63", \
  "v64","v65","v66","v67","v68","v69","v70","v71","v72","v73","v74","v75","v76","v77","v78","v79","v80","v81","v82","v83","v84","v85","v86","v87","v88","v89","v90","v91","v92","v93","v94","v95", \
  "v96","v97","v98","v99","v100","v101","v102","v103","v104","v105","v106","v107","v108","v109","v110","v111","v112","v113","v114","v115","v116","v117","v118","v119","v120","v121","v122","v123","v124","v125","v126","v127", \
  "v128","v129","v130","v131","v132","v133","v134","v135","v136","v137","v138","v139","v140","v141","v142","v143","v144","v145","v146","v147","v148","v149","v150","v151","v152","v153","v154","v155","v156","v157","v158","v159", \
  "v160","v161","v162","v163","v164","v165","v166","v167","v168","v169","v170","v171","v172","v173","v174","v175","v176","v177","v178","v179","v180","v181","v182","v183","v184","v185","v186","v187","v188","v189","v190","v191", \
  "v192","v193","v194","v195","v196","v197","v198","v199","v200","v201","v202","v203","v204","v205","v206","v207","v208","v209","v210","v211","v212","v213","v214", \
  "a0","a1","a2","a3","a4","a5","a6","a7","a8","a9","a10","a11","a12","a13","a14","a15","a16","a17","a18","a19","a20","a21","a22","a23","a24","a25","a26","a27","a28","a29","a30","a31", \
  "a32","a33","a34","a35","a36","a37","a38","a39","a40","a41","a42","a43","a44","a45","a46","a47","a48","a49","a50","a51","a52","a53","a54","a55","a56","a57","a58","a59","a60","a61","a62","a63", \
  "a64","a65","a66","a67","a68","a69","a70","a71","a72","a73","a74","a75","a76","a77","a78","a79","a80","a81","a82","a83","a84","a85","a86","a87","a88","a89","a90","a91","a92","a93","a94","a95", \
  "a96","a97","a98","a99","a100","a101","a102","a103","a104","a105","a106","a107","a108","a109","a110","a111","a112","a113","a114","a115","a116","a117","a118","a119","a120","a121","a122","a123","a124","a125","a126","a127", \
  "a128","a129","a130","a131","a132","a133","a134","a135","a136","a137","a138","a139","a140","a141","a142","a143","a144","a145","a146","a147","a148","a149","a150","a151","a152","a153","a154","a155","a156","a157","a158","a159", \
  "a160","a161","a162","a163","a164","a165","a166","a167","a168","a169","a170","a171","a172","a173","a174","a175","a176","a177","a178","a179","a180","a181","a182","a183","a184","a185","a186","a187","a188","a189","a190","a191", \
  "a192","a193","a194","a195","a196","a197","a198","a199","a200","a201","a202","a203","a204","a205","a206","a207","a208","a209","a210","a211","a212","a213","a214","a215","a216","a217","a218","a219","a220","a221","a222","a223", \
  "a224","a225","a226","a227","a228","a229","a230","a231","a232","a233","a234","a235","a236","a237","a238","a239","a240","a241","a242","a243","a244","a245","a246","a247","a248","a249","a250","a251","a252","a253","a254","a255", \
  "s40","s41","s42","s43","s44","s45","s46","s47","s48","s49","s50","s51","s52","s53","s54","s55","s56","s57","s58","s59","s60","s61","s62","s63","s64","s65","s66","s67","s68","s69","s70","s71","s72","vcc","scc","memory"

DI void gemm_loop(int voffx, int voffw, int boff, int ooff, int tab, int fbase, __amdgpu_buffer_rsrc_t rw, __amdgpu_buffer_rsrc_t ro,
                  unsigned mw, float c2, float is, unsigned call, unsigned np, unsigned dbase, unsigned fwd, unsigned long long pc0) {
  asm volatile(R"(
.macro SIL u, st, g
  .set RA, (\u>>2)*16+(\g&3)*4+(\u&3)
  .set RT, 192+(\g&3)
  .if (\st == 0) && (\g < 4)
  v_mul_f32 v[RT], s61, v[RA]
  .elseif \st == 0
  v_med3_f32 v[RT], v[RT], s55, -s55
  .elseif (\st == 1) && (\g < 4)
  v_exp_f32 v[RT], v[RT]
  .elseif \st == 1
  v_fma_f32 v[RT], v[RT], s52, s52
  .elseif \g < 4
  v_rcp_f32 v[RT], v[RT]
  .else
  v_mul_f32 v[RT+4+(\u&1)*4], v[RA], v[RT]
  .endif
.endm
.macro MM d, c, f, a, b
  .if \c
  v_mfma_f32_16x16x128_f8f6f4 a[\d:\d+3], \f[\a:\a+7], v[\b:\b+7], a[\d:\d+3] blgp:1
  .else
  v_mfma_f32_16x16x128_f8f6f4 a[\d:\d+3], \f[\a:\a+7], v[\b:\b+7], 0 blgp:1
  .endif
.endm
.macro CPY d, a
  v_accvgpr_read_b32 v[\d], a[\a]
  v_accvgpr_read_b32 v[\d+1], a[\a+1]
  v_accvgpr_read_b32 v[\d+2], a[\a+2]
  v_accvgpr_read_b32 v[\d+3], a[\a+3]
.endm
.macro STADR u
  s_add_u32 s[70+(\u&1)], s47, (((\u>>2)*16+(\u&3))*16384*4)
.endm
.macro STORE u
  buffer_store_dwordx4 v[196+(\u&1)*4:199+(\u&1)*4], %3, %7, s[70+(\u&1)] offen )" STBITS R"(
.endm
.macro DUP u, k
  s_bitcmp1_b32 s46, 16
  s_cbranch_scc1 .Ldup%=_\k
.Ldupb%=_\k:
.endm
.macro DUPO u, k
.Ldup%=_\k:
  s_add_u32 s69, %14, (\u*1024)
  buffer_store_dwordx4 v[196+(\u&1)*4:199+(\u&1)*4], %2, %6, s69 offen sc1
  s_nop 1
  s_branch .Ldupb%=_\k
.endm
.macro CHKF i
  .if \i < 4
  s_bfe_u32 s69, vcc_lo, (0x80000+\i*8)
  .else
  s_bfe_u32 s69, vcc_hi, (0x80000+(\i-4)*8)
  .endif
  s_cmp_eq_u32 s69, 0xff
.endm
.macro SETW o, w
  s_add_u32 m0, %8, \o
  s_add_u32 s51, s40, s[\w]
.endm
.macro LDW o
  buffer_load_dwordx4 %1, %6, s51 offen offset:\o sc0 sc1 lds
.endm
.macro LDX f, r, o
  buffer_load_dwordx4 \f[\r:\r+3], %0, %6, s50 offen offset:\o sc0 sc1
.endm
.macro RD d, a, o
  ds_read_b128 v[\d:\d+3], \a offset:\o
.endm
.macro SLOW k, i
.Lslow%=_\k:
  s_memrealtime s[56:57]
.Lpoll%=_\k:
  buffer_load_dword v208, v206, %6, s49 offen sc0 sc1
  s_mov_b32 s69, 32768
  buffer_load_dword v209, off, %6, s69 sc1
  s_memrealtime s[58:59]
  s_waitcnt vmcnt(0) lgkmcnt(0)
  v_cmp_le_u32 vcc, s53, v208
  CHKF \i
  s_cbranch_scc1 .Lback%=_\k
  v_readfirstlane_b32 s69, v209
  s_sub_u32 s58, s58, s56
  s_cmp_eq_u32 s69, %11
  s_cbranch_scc1 .Lfail%=_\k
  s_cmp_gt_u32 s58, 30000000
  s_cbranch_scc1 .Lfail%=_\k
  s_sleep )" XSTR(PSLEEP) R"(
  s_branch .Lpoll%=_\k
.Lfail%=_\k:
  v_mov_b32 v209, %11
  s_mov_b32 s69, 32768
  buffer_store_dword v209, off, %6, s69 sc1
  s_mov_b32 s52, 0x7fc00000
  s_mov_b32 s53, 0
  s_mov_b64 vcc, -1
  s_branch .Lback%=_\k
.endm
.macro SIL8 u, st
  SIL \u, \st, 0
  SIL \u, \st, 1
  SIL \u, \st, 2
  SIL \u, \st, 3
  SIL \u, \st, 4
  SIL \u, \st, 5
  SIL \u, \st, 6
  SIL \u, \st, 7
.endm
.macro LAST u
  SIL8 \u, 0
  SIL8 \u, 1
  SIL8 \u, 2
  STADR \u
  STORE \u
.endm
.macro LAST4 m
  CPY (\m*16), (\m*16)
  CPY (\m*16+4), (\m*16+4)
  CPY (\m*16+8), (\m*16+8)
  CPY (\m*16+12), (\m*16+12)
  LAST (\m*4)
  LAST (\m*4+1)
  LAST (\m*4+2)
  LAST (\m*4+3)
.endm
  ; read the kernel's code once through the data path: later first executions then fetch from the Infinity Cache, not from HBM
  s_getpc_b64 s[56:57]
.Lpc%=:
  s_add_u32 s58, s56, .Lend%=-.Lpc%=
  s_mov_b64 s[56:57], %15
  s_sub_u32 s58, s58, s56
  s_and_b32 s57, s57, 0xffff
  s_mov_b32 s59, 0x20000
  s_lshr_b32 s68, %8, 1
  .set RA, 0
  .rept 18
  buffer_load_dwordx4 v[RA:RA+3], %2, s[56:59], s68 offen
  s_add_u32 s68, s68, 4096
  .set RA, RA+4
  .endr
  v_mov_b32 v210, %4
  ds_read_b32 v204, v210
  ds_read_b32 v205, v210 offset:32
  s_add_u32 s40, %13, (31*2621440)
  s_add_u32 s60, %13, (32*2621440)
  s_mov_b32 s42, 0x80000000
  s_mov_b32 s43, 0x80000000
  s_mov_b32 s44, 0x80000000
  s_mov_b32 s45, 0x80000000
  s_mov_b32 s46, 0xff
  s_mov_b32 s47, 0x80000000
  s_mov_b32 s48, 0x80000000
  s_mov_b32 s49, 30720
  v_readfirstlane_b32 s52, %10
  v_readfirstlane_b32 s61, %9
  s_mov_b32 s53, %11
  s_mov_b32 s54, %12
  s_mov_b32 s55, 0x42ad1f97
  s_waitcnt vmcnt(0) lgkmcnt(0)
  v_add_u32 v207, v204, %5
  v_add_u32 v211, 65536, %2
  v_mov_b32 v214, %11
  s_lshr_b32 s66, %8, 9
  s_branch .Lentry%=
SLOW G_0_2n, 2
SLOW G_0_2f, 2
SLOW G_0_2s, 2
DUPO 0, 0
SLOW G_1_2n, 3
SLOW G_1_2s, 3
DUPO 1, 1
DUPO 2, 2
DUPO 3, 3
SLOW G_2_2n, 4
SLOW G_2_2s, 4
DUPO 4, 4
DUPO 5, 5
SLOW G_3_2n, 5
SLOW G_3_2s, 5
DUPO 6, 6
DUPO 7, 7
DUPO 8, 8
SLOW G_4_2n, 6
SLOW G_4_2s, 6
DUPO 9, 9
DUPO 10, 10
SLOW G_5_2n, 7
SLOW G_5_2s, 7
DUPO 11, 11
DUPO 12, 12
DUPO 13, 13
SLOW G_6_2n, 0
SLOW G_6_2s, 0
DUPO 14, 14
DUPO 15, 15
SLOW G_7_2n, 1
SLOW G_7_2s, 1
SLOW T3_0_2n, 2
SLOW T3_1_2n, 3
SLOW T3_2_2n, 4
SLOW T3_3_2n, 5
SLOW T3_4_2n, 6
SLOW T3_5_2n, 7
SLOW T3_6_2n, 0
SLOW T3_7_2n, 1
SLOW T2_0_2n, 2
SLOW T2_1_2n, 3
SLOW T2_2_2n, 4
SLOW T2_3_2n, 5
SLOW T2_4_2n, 6
SLOW T2_5_2n, 7
SLOW T2_6_2n, 0
SLOW T2_7_2n, 1
SLOW T1_0_2n, 2
SLOW T1_0_3n, 3
SLOW T1_1_3n, 4
SLOW T1_2_3n, 5
SLOW T1_3_3n, 6
SLOW T1_4_3n, 7
.Lspec%=_0_0:
s_bitcmp1_b32 s46, 12
s_cbranch_scc0 .Lnotf%=_0_0
CPY 0, 0
s_waitcnt vmcnt(17) lgkmcnt(0)
MM 0, 0, v, 64, 128
RD 160, %2, 8192
s_add_u32 s50, s40, s48
CPY 4, 4
MM 4, 0, v, 64, 136
LDX v, 96, 0
CPY 8, 8
MM 8, 0, v, 64, 144
RD 164, %2, 9216
CPY 12, 12
MM 12, 0, v, 64, 152
RD 168, %2, 10240
CPY 28, 28
s_waitcnt vmcnt(15)
MM 28, 0, v, 72, 152
RD 172, %2, 11264
CPY 24, 24
MM 24, 0, v, 72, 144
LDX v, 100, 1024
CPY 20, 20
MM 20, 0, v, 72, 136
RD 176, %2, 12288
CPY 16, 16
MM 16, 0, v, 72, 128
RD 180, %2, 13312
CPY 32, 32
s_waitcnt vmcnt(14)
MM 32, 0, v, 80, 128
RD 184, %2, 14336
CPY 36, 36
MM 36, 0, v, 80, 136
LDX v, 104, 2048
CPY 40, 40
MM 40, 0, v, 80, 144
RD 188, %2, 15360
CPY 44, 44
MM 44, 0, v, 80, 152
CPY 60, 60
s_waitcnt vmcnt(13)
MM 60, 0, v, 88, 152
CPY 56, 56
MM 56, 0, v, 88, 144
LDX v, 108, 3072
CPY 52, 52
MM 52, 0, v, 88, 136
CPY 48, 48
MM 48, 0, v, 88, 128
s_branch .Ljoin%=_0_0
.Lnotf%=_0_0:
s_waitcnt vmcnt(17) lgkmcnt(0)
RD 160, %2, 8192
s_add_u32 s50, s40, s48
LDX v, 96, 0
RD 164, %2, 9216
RD 168, %2, 10240
s_waitcnt vmcnt(15)
RD 172, %2, 11264
LDX v, 100, 1024
RD 176, %2, 12288
RD 180, %2, 13312
s_waitcnt vmcnt(14)
RD 184, %2, 14336
LDX v, 104, 2048
RD 188, %2, 15360
s_waitcnt vmcnt(13)
LDX v, 108, 3072
s_branch .Ljoin%=_0_0
.Lspec%=_0_1:
s_bitcmp1_b32 s46, 13
s_cbranch_scc0 .Lnotf%=_0_1
CPY 48, 112
s_waitcnt lgkmcnt(0)
MM 112, 0, v, 88, 160
RD 128, %2, 16384
s_add_u32 s50, s50, 4096
CPY 52, 116
MM 116, 0, v, 88, 168
LDX v, 112, 0
CPY 56, 120
MM 120, 0, v, 88, 176
RD 132, %2, 17408
CPY 60, 124
MM 124, 0, v, 88, 184
RD 136, %2, 18432
CPY 44, 108
MM 108, 0, v, 80, 184
RD 140, %2, 19456
CPY 40, 104
MM 104, 0, v, 80, 176
LDX v, 116, 1024
CPY 36, 100
MM 100, 0, v, 80, 168
RD 144, %2, 20480
CPY 32, 96
MM 96, 0, v, 80, 160
RD 148, %2, 21504
CPY 16, 80
MM 80, 0, v, 72, 160
RD 152, %2, 22528
CPY 20, 84
MM 84, 0, v, 72, 168
LDX v, 120, 2048
CPY 24, 88
MM 88, 0, v, 72, 176
RD 156, %2, 23552
CPY 28, 92
MM 92, 0, v, 72, 184
s_add_u32 s72, s72, s66
CPY 12, 76
MM 76, 0, v, 64, 184
CPY 8, 72
MM 72, 0, v, 64, 176
LDX v, 124, 3072
CPY 4, 68
MM 68, 0, v, 64, 168
CPY 0, 64
MM 64, 0, v, 64, 160
s_branch .Ljoin%=_0_1
.Lnotf%=_0_1:
s_waitcnt lgkmcnt(0)
RD 128, %2, 16384
s_add_u32 s50, s50, 4096
LDX v, 112, 0
RD 132, %2, 17408
RD 136, %2, 18432
RD 140, %2, 19456
LDX v, 116, 1024
RD 144, %2, 20480
RD 148, %2, 21504
RD 152, %2, 22528
LDX v, 120, 2048
RD 156, %2, 23552
s_add_u32 s72, s72, s66
LDX v, 124, 3072
s_branch .Ljoin%=_0_1
.Lspec%=_0_2:
s_bitcmp1_b32 s46, 14
s_cbranch_scc0 .Lnotf%=_0_2
CPY 0, 128
s_waitcnt lgkmcnt(0)
MM 128, 0, v, 64, 128
RD 160, %2, 24576
CPY 4, 132
MM 132, 0, v, 64, 136
CPY 8, 136
MM 136, 0, v, 64, 144
RD 164, %2, 25600
CPY 12, 140
MM 140, 0, v, 64, 152
CPY 28, 156
MM 156, 0, v, 72, 152
RD 168, %2, 26624
CPY 24, 152
MM 152, 0, v, 72, 144
RD 172, %2, 27648
CPY 20, 148
MM 148, 0, v, 72, 136
RD 176, %2, 28672
CPY 16, 144
MM 144, 0, v, 72, 128
RD 180, %2, 29696
CPY 32, 160
MM 160, 0, v, 80, 128
RD 184, %2, 30720
CPY 36, 164
MM 164, 0, v, 80, 136
RD 188, %2, 31744
CPY 40, 168
MM 168, 0, v, 80, 144
s_waitcnt vmcnt(16)
v_cmp_le_u32 vcc, s53, v208
CPY 44, 172
MM 172, 0, v, 80, 152
CHKF 2
s_cbranch_scc0 .Lslow%=_G_0_2f
.Lback%=_G_0_2f:
CPY 60, 188
MM 188, 0, v, 88, 152
s_add_u32 s40, s40, 2621440
CPY 56, 184
MM 184, 0, v, 88, 144
CPY 52, 180
MM 180, 0, v, 88, 136
CPY 48, 176
MM 176, 0, v, 88, 128
s_branch .Ljoin%=_0_2
.Lnotf%=_0_2:
s_waitcnt lgkmcnt(0)
RD 160, %2, 24576
RD 164, %2, 25600
RD 168, %2, 26624
RD 172, %2, 27648
RD 176, %2, 28672
RD 180, %2, 29696
RD 184, %2, 30720
RD 188, %2, 31744
s_waitcnt vmcnt(16)
v_cmp_le_u32 vcc, s53, v208
CHKF 2
s_cbranch_scc0 .Lslow%=_G_0_2s
.Lback%=_G_0_2s:
s_add_u32 s40, s40, 2621440
s_branch .Ljoin%=_0_2
.Lspec%=_0_3:
s_bitcmp1_b32 s46, 15
s_cbranch_scc0 .Lnotf%=_0_3
CPY 48, 240
s_waitcnt lgkmcnt(0)
MM 240, 0, v, 88, 160
s_waitcnt vmcnt(8)
s_barrier
CPY 52, 244
MM 244, 0, v, 88, 168
RD 128, %2, 32768
RD 132, %2, 33792
SETW 65536, 42
CPY 56, 248
MM 248, 0, v, 88, 176
LDW 0
CPY 60, 252
MM 252, 0, v, 88, 184
LDW 1024
CPY 44, 236
MM 236, 0, v, 80, 184
RD 136, %2, 34816
RD 140, %2, 35840
SETW 73728, 43
CPY 40, 232
MM 232, 0, v, 80, 176
LDW 0
CPY 36, 228
MM 228, 0, v, 80, 168
LDW 1024
CPY 32, 224
MM 224, 0, v, 80, 160
RD 144, %2, 36864
RD 148, %2, 37888
CPY 16, 208
MM 208, 0, v, 72, 160
RD 152, %2, 38912
SETW 81920, 44
CPY 20, 212
MM 212, 0, v, 72, 168
LDW 0
CPY 24, 216
MM 216, 0, v, 72, 176
LDW 1024
CPY 28, 220
MM 220, 0, v, 72, 184
RD 156, %2, 39936
SETW 90112, 45
CPY 12, 204
MM 204, 0, v, 64, 184
LDW 0
CPY 8, 200
MM 200, 0, v, 64, 176
LDW 1024
CPY 4, 196
MM 196, 0, v, 64, 168
CPY 0, 192
MM 192, 0, v, 64, 160
s_branch .Ljoin%=_0_3
.Lnotf%=_0_3:
s_waitcnt lgkmcnt(0)
s_waitcnt vmcnt(8)
s_barrier
RD 128, %2, 32768
RD 132, %2, 33792
SETW 65536, 42
LDW 0
LDW 1024
RD 136, %2, 34816
RD 140, %2, 35840
SETW 73728, 43
LDW 0
LDW 1024
RD 144, %2, 36864
RD 148, %2, 37888
RD 152, %2, 38912
SETW 81920, 44
LDW 0
LDW 1024
RD 156, %2, 39936
SETW 90112, 45
LDW 0
LDW 1024
s_branch .Ljoin%=_0_3
.Lspec%=_1_0:
s_waitcnt vmcnt(14) lgkmcnt(0)
RD 160, %2, 40960
s_add_u32 s50, s40, s48
SIL 0, 0, 0
SIL 0, 0, 1
LDX v, 64, 0
RD 164, %2, 41984
SIL 0, 0, 2
SIL 0, 0, 3
RD 168, %2, 43008
SIL 0, 0, 4
s_waitcnt vmcnt(13)
RD 172, %2, 44032
SIL 0, 0, 5
LDX v, 68, 1024
RD 176, %2, 45056
SIL 0, 0, 6
SIL 0, 0, 7
RD 180, %2, 46080
SIL 0, 1, 0
s_waitcnt vmcnt(12)
RD 184, %2, 47104
SIL 0, 1, 1
LDX v, 72, 2048
RD 188, %2, 48128
SIL 0, 1, 2
SIL 0, 1, 3
SIL 0, 1, 4
s_waitcnt vmcnt(11)
SIL 0, 1, 5
LDX v, 76, 3072
SIL 0, 1, 6
SIL 0, 1, 7
s_branch .Ljoin%=_1_0
.Lspec%=_1_1:
s_waitcnt lgkmcnt(0)
RD 128, %2, 49152
s_add_u32 s50, s50, 4096
SIL 0, 2, 0
LDX v, 80, 0
RD 132, %2, 50176
SIL 0, 2, 1
SIL 0, 2, 2
RD 136, %2, 51200
SIL 0, 2, 3
SIL 0, 2, 4
RD 140, %2, 52224
STADR 0
LDX v, 84, 1024
RD 144, %2, 53248
SIL 0, 2, 5
SIL 0, 2, 6
RD 148, %2, 54272
SIL 0, 2, 7
RD 152, %2, 55296
LDX v, 88, 2048
RD 156, %2, 56320
SIL 1, 0, 0
SIL 1, 0, 1
buffer_store_dword v214, off, %6, s72 sc1
SIL 1, 0, 2
SIL 1, 0, 3
LDX v, 92, 3072
SIL 1, 0, 4
SIL 1, 0, 5
s_branch .Ljoin%=_1_1
.Lspec%=_1_2:
s_waitcnt lgkmcnt(0)
RD 160, %2, 57344
SIL 1, 0, 6
SIL 1, 0, 7
STORE 0
RD 164, %2, 58368
SIL 1, 1, 0
SIL 1, 1, 1
SIL 1, 1, 2
RD 168, %2, 59392
SIL 1, 1, 3
RD 172, %2, 60416
SIL 1, 1, 4
RD 176, %2, 61440
SIL 1, 1, 5
RD 180, %2, 62464
SIL 1, 1, 6
RD 184, %2, 63488
SIL 1, 1, 7
RD 188, %2, 64512
SIL 1, 2, 0
SIL 1, 2, 1
CHKF 3
s_cbranch_scc0 .Lslow%=_G_1_2s
.Lback%=_G_1_2s:
SIL 1, 2, 2
s_add_u32 s40, s40, 2621440
SIL 1, 2, 3
SIL 1, 2, 4
buffer_load_dword v208, v206, %6, s49 offen sc0 sc1
s_branch .Ljoin%=_1_2
.Lspec%=_1_3:
s_waitcnt lgkmcnt(0)
s_waitcnt vmcnt(11)
s_barrier
RD 128, v211, 0
RD 132, v211, 1024
SETW 98304, 42
STADR 1
LDW 0
LDW 1024
RD 136, v211, 2048
RD 140, v211, 3072
SETW 106496, 43
SIL 1, 2, 5
LDW 0
LDW 1024
RD 144, v211, 4096
RD 148, v211, 5120
SIL 1, 2, 6
SIL 1, 2, 7
RD 152, v211, 6144
SETW 114688, 44
LDW 0
LDW 1024
RD 156, v211, 7168
SETW 122880, 45
SIL 2, 0, 0
SIL 2, 0, 1
LDW 0
LDW 1024
SIL 2, 0, 2
SIL 2, 0, 3
SIL 2, 0, 4
SIL 2, 0, 5
SIL 2, 0, 6
s_branch .Ljoin%=_1_3
.Lspec%=_2_0:
s_waitcnt vmcnt(17) lgkmcnt(0)
RD 160, v211, 8192
s_add_u32 s50, s40, s48
SIL 2, 0, 7
SIL 2, 1, 0
LDX v, 96, 0
RD 164, v211, 9216
SIL 2, 1, 1
SIL 2, 1, 2
RD 168, v211, 10240
SIL 2, 1, 3
SIL 2, 1, 4
SIL 2, 1, 5
s_waitcnt vmcnt(16)
RD 172, v211, 11264
SIL 2, 1, 6
SIL 2, 1, 7
SIL 2, 2, 0
LDX v, 100, 1024
RD 176, v211, 12288
SIL 2, 2, 1
SIL 2, 2, 2
RD 180, v211, 13312
SIL 2, 2, 3
SIL 2, 2, 4
s_waitcnt vmcnt(15)
RD 184, v211, 14336
STADR 2
LDX v, 104, 2048
RD 188, v211, 15360
SIL 2, 2, 5
SIL 2, 2, 6
STORE 1
s_waitcnt vmcnt(14)
SIL 2, 2, 7
LDX v, 108, 3072
v_add_u32 v210, 64, v210
SIL 3, 0, 0
SIL 3, 0, 1
s_branch .Ljoin%=_2_0
.Lspec%=_2_1:
s_waitcnt lgkmcnt(0)
RD 128, v211, 16384
s_add_u32 s50, s50, 4096
SIL 3, 0, 2
SIL 3, 0, 3
LDX v, 112, 0
RD 132, v211, 17408
SIL 3, 0, 4
SIL 3, 0, 5
RD 136, v211, 18432
SIL 3, 0, 6
RD 140, v211, 19456
SIL 3, 0, 7
LDX v, 116, 1024
RD 144, v211, 20480
SIL 3, 1, 0
SIL 3, 1, 1
RD 148, v211, 21504
SIL 3, 1, 2
RD 152, v211, 22528
SIL 3, 1, 3
LDX v, 120, 2048
RD 156, v211, 23552
SIL 3, 1, 4
SIL 3, 1, 5
STORE 2
SIL 3, 1, 6
SIL 3, 1, 7
LDX v, 124, 3072
SIL 3, 2, 0
SIL 3, 2, 1
s_branch .Ljoin%=_2_1
.Lspec%=_2_2:
s_waitcnt lgkmcnt(0)
RD 160, v211, 24576
SIL 3, 2, 2
SIL 3, 2, 3
SIL 3, 2, 4
RD 164, v211, 25600
STADR 3
SIL 3, 2, 5
RD 168, v211, 26624
SIL 3, 2, 6
RD 172, v211, 27648
SIL 3, 2, 7
RD 176, v211, 28672
RD 180, v211, 29696
SIL 4, 0, 0
RD 184, v211, 30720
SIL 4, 0, 1
RD 188, v211, 31744
SIL 4, 0, 2
s_waitcnt vmcnt(18)
v_cmp_le_u32 vcc, s53, v208
SIL 4, 0, 3
CHKF 4
s_cbranch_scc0 .Lslow%=_G_2_2s
.Lback%=_G_2_2s:
SIL 4, 0, 4
s_add_u32 s40, s40, 2621440
SIL 4, 0, 5
STORE 3
SIL 4, 0, 6
SIL 4, 0, 7
s_branch .Ljoin%=_2_2
.Lspec%=_2_3:
s_waitcnt lgkmcnt(0)
s_waitcnt vmcnt(11)
s_barrier
RD 128, v211, 32768
RD 132, v211, 33792
SETW 0, 42
LDW 0
LDW 1024
RD 136, v211, 34816
RD 140, v211, 35840
SETW 8192, 43
LDW 0
LDW 1024
RD 144, v211, 36864
RD 148, v211, 37888
SIL 4, 1, 0
RD 152, v211, 38912
SETW 16384, 44
SIL 4, 1, 1
LDW 0
LDW 1024
RD 156, v211, 39936
SETW 24576, 45
SIL 4, 1, 2
LDW 0
LDW 1024
SIL 4, 1, 3
SIL 4, 1, 4
SIL 4, 1, 5
SIL 4, 1, 6
s_branch .Ljoin%=_2_3
.Lspec%=_3_0:
s_waitcnt vmcnt(17) lgkmcnt(0)
RD 160, v211, 40960
s_add_u32 s50, s40, s48
SIL 4, 1, 7
SIL 4, 2, 0
LDX v, 64, 0
RD 164, v211, 41984
SIL 4, 2, 1
SIL 4, 2, 2
RD 168, v211, 43008
SIL 4, 2, 3
SIL 4, 2, 4
STADR 4
s_waitcnt vmcnt(15)
RD 172, v211, 44032
SIL 4, 2, 5
SIL 4, 2, 6
SIL 4, 2, 7
LDX v, 68, 1024
RD 176, v211, 45056
SIL 5, 0, 0
SIL 5, 0, 1
RD 180, v211, 46080
SIL 5, 0, 2
SIL 5, 0, 3
SIL 5, 0, 4
SIL 5, 0, 5
s_waitcnt vmcnt(14)
RD 184, v211, 47104
SIL 5, 0, 6
SIL 5, 0, 7
LDX v, 72, 2048
RD 188, v211, 48128
SIL 5, 1, 0
SIL 5, 1, 1
STORE 4
s_waitcnt vmcnt(13)
SIL 5, 1, 2
SIL 5, 1, 3
LDX v, 76, 3072
ds_read_b32 v204, v210
ds_read_b32 v205, v210 offset:32
SIL 5, 1, 4
SIL 5, 1, 5
s_branch .Ljoin%=_3_0
.Lspec%=_3_1:
s_waitcnt lgkmcnt(2)
RD 128, v211, 49152
s_add_u32 s50, s50, 4096
SIL 5, 1, 6
SIL 5, 1, 7
LDX v, 80, 0
RD 132, v211, 50176
SIL 5, 2, 0
SIL 5, 2, 1
RD 136, v211, 51200
SIL 5, 2, 2
RD 140, v211, 52224
SIL 5, 2, 3
LDX v, 84, 1024
RD 144, v211, 53248
SIL 5, 2, 4
STADR 5
RD 148, v211, 54272
SIL 5, 2, 5
RD 152, v211, 55296
SIL 5, 2, 6
LDX v, 88, 2048
RD 156, v211, 56320
SIL 5, 2, 7
STORE 5
SIL 6, 0, 0
SIL 6, 0, 1
LDX v, 92, 3072
SIL 6, 0, 2
SIL 6, 0, 3
s_branch .Ljoin%=_3_1
.Lspec%=_3_2:
s_waitcnt lgkmcnt(0)
RD 160, v211, 57344
SIL 6, 0, 4
SIL 6, 0, 5
SIL 6, 0, 6
RD 164, v211, 58368
SIL 6, 0, 7
SIL 6, 1, 0
RD 168, v211, 59392
SIL 6, 1, 1
RD 172, v211, 60416
SIL 6, 1, 2
RD 176, v211, 61440
SIL 6, 1, 3
RD 180, v211, 62464
SIL 6, 1, 4
RD 184, v211, 63488
SIL 6, 1, 5
RD 188, v211, 64512
SIL 6, 1, 6
SIL 6, 1, 7
CHKF 5
s_cbranch_scc0 .Lslow%=_G_3_2s
.Lback%=_G_3_2s:
SIL 6, 2, 0
s_add_u32 s40, s40, 2621440
SIL 6, 2, 1
SIL 6, 2, 2
buffer_load_dword v208, v206, %6, s49 offen sc0 sc1
s_branch .Ljoin%=_3_2
.Lspec%=_3_3:
s_waitcnt lgkmcnt(0)
s_waitcnt vmcnt(11)
s_barrier
RD 128, %2, 0
RD 132, %2, 1024
SETW 32768, 42
LDW 0
LDW 1024
RD 136, %2, 2048
RD 140, %2, 3072
SETW 40960, 43
LDW 0
LDW 1024
RD 144, %2, 4096
RD 148, %2, 5120
SIL 6, 2, 3
SIL 6, 2, 4
RD 152, %2, 6144
SETW 49152, 44
STADR 6
SIL 6, 2, 5
LDW 0
LDW 1024
RD 156, %2, 7168
SETW 57344, 45
SIL 6, 2, 6
SIL 6, 2, 7
LDW 0
LDW 1024
SIL 7, 0, 0
SIL 7, 0, 1
SIL 7, 0, 2
s_branch .Ljoin%=_3_3
.Lspec%=_4_0:
s_waitcnt vmcnt(17) lgkmcnt(0)
RD 160, %2, 8192
s_add_u32 s50, s40, s48
SIL 7, 0, 3
SIL 7, 0, 4
SIL 7, 0, 5
LDX v, 96, 0
RD 164, %2, 9216
SIL 7, 0, 6
SIL 7, 0, 7
SIL 7, 1, 0
RD 168, %2, 10240
SIL 7, 1, 1
SIL 7, 1, 2
s_waitcnt vmcnt(15)
RD 172, %2, 11264
SIL 7, 1, 3
SIL 7, 1, 4
SIL 7, 1, 5
LDX v, 100, 1024
RD 176, %2, 12288
SIL 7, 1, 6
SIL 7, 1, 7
SIL 7, 2, 0
RD 180, %2, 13312
SIL 7, 2, 1
SIL 7, 2, 2
s_waitcnt vmcnt(14)
RD 184, %2, 14336
SIL 7, 2, 3
LDX v, 104, 2048
RD 188, %2, 15360
SIL 7, 2, 4
STADR 7
STORE 6
s_waitcnt vmcnt(13)
SIL 7, 2, 5
SIL 7, 2, 6
LDX v, 108, 3072
v_add_u32 v207, v204, %5
SIL 7, 2, 7
s_branch .Ljoin%=_4_0
.Lspec%=_4_1:
s_waitcnt lgkmcnt(0)
RD 128, %2, 16384
s_add_u32 s50, s50, 4096
SIL 8, 0, 0
SIL 8, 0, 1
LDX v, 112, 0
RD 132, %2, 17408
SIL 8, 0, 2
SIL 8, 0, 3
RD 136, %2, 18432
SIL 8, 0, 4
RD 140, %2, 19456
SIL 8, 0, 5
LDX v, 116, 1024
RD 144, %2, 20480
SIL 8, 0, 6
SIL 8, 0, 7
RD 148, %2, 21504
SIL 8, 1, 0
RD 152, %2, 22528
SIL 8, 1, 1
LDX v, 120, 2048
RD 156, %2, 23552
SIL 8, 1, 2
SIL 8, 1, 3
STORE 7
SIL 8, 1, 4
SIL 8, 1, 5
LDX v, 124, 3072
SIL 8, 1, 6
SIL 8, 1, 7
s_branch .Ljoin%=_4_1
.Lspec%=_4_2:
s_waitcnt lgkmcnt(0)
RD 160, %2, 24576
SIL 8, 2, 0
SIL 8, 2, 1
SIL 8, 2, 2
RD 164, %2, 25600
SIL 8, 2, 3
SIL 8, 2, 4
RD 168, %2, 26624
STADR 8
RD 172, %2, 27648
SIL 8, 2, 5
RD 176, %2, 28672
SIL 8, 2, 6
RD 180, %2, 29696
SIL 8, 2, 7
RD 184, %2, 30720
RD 188, %2, 31744
SIL 9, 0, 0
s_waitcnt vmcnt(18)
v_cmp_le_u32 vcc, s53, v208
SIL 9, 0, 1
CHKF 6
s_cbranch_scc0 .Lslow%=_G_4_2s
.Lback%=_G_4_2s:
SIL 9, 0, 2
s_add_u32 s40, s40, 2621440
SIL 9, 0, 3
STORE 8
SIL 9, 0, 4
SIL 9, 0, 5
s_branch .Ljoin%=_4_2
.Lspec%=_4_3:
s_waitcnt lgkmcnt(0)
s_waitcnt vmcnt(11)
s_barrier
RD 128, %2, 32768
RD 132, %2, 33792
SETW 65536, 42
SIL 9, 0, 6
LDW 0
LDW 1024
RD 136, %2, 34816
RD 140, %2, 35840
SETW 73728, 43
SIL 9, 0, 7
LDW 0
LDW 1024
RD 144, %2, 36864
RD 148, %2, 37888
SIL 9, 1, 0
RD 152, %2, 38912
SETW 81920, 44
SIL 9, 1, 1
LDW 0
LDW 1024
RD 156, %2, 39936
SETW 90112, 45
SIL 9, 1, 2
LDW 0
LDW 1024
SIL 9, 1, 3
SIL 9, 1, 4
SIL 9, 1, 5
SIL 9, 1, 6
s_branch .Ljoin%=_4_3
.Lspec%=_5_0:
s_waitcnt vmcnt(17) lgkmcnt(0)
RD 160, %2, 40960
s_add_u32 s50, s40, s48
SIL 9, 1, 7
SIL 9, 2, 0
LDX v, 64, 0
RD 164, %2, 41984
SIL 9, 2, 1
SIL 9, 2, 2
RD 168, %2, 43008
SIL 9, 2, 3
SIL 9, 2, 4
STADR 9
s_waitcnt vmcnt(15)
RD 172, %2, 44032
SIL 9, 2, 5
SIL 9, 2, 6
SIL 9, 2, 7
LDX v, 68, 1024
RD 176, %2, 45056
SIL 10, 0, 0
SIL 10, 0, 1
RD 180, %2, 46080
SIL 10, 0, 2
SIL 10, 0, 3
SIL 10, 0, 4
s_waitcnt vmcnt(14)
RD 184, %2, 47104
SIL 10, 0, 5
LDX v, 72, 2048
RD 188, %2, 48128
SIL 10, 0, 6
SIL 10, 0, 7
STORE 9
s_waitcnt vmcnt(13)
SIL 10, 1, 0
SIL 10, 1, 1
LDX v, 76, 3072
SIL 10, 1, 2
SIL 10, 1, 3
s_branch .Ljoin%=_5_0
.Lspec%=_5_1:
s_waitcnt lgkmcnt(0)
RD 128, %2, 49152
s_add_u32 s50, s50, 4096
SIL 10, 1, 4
SIL 10, 1, 5
LDX v, 80, 0
RD 132, %2, 50176
SIL 10, 1, 6
SIL 10, 1, 7
RD 136, %2, 51200
SIL 10, 2, 0
RD 140, %2, 52224
SIL 10, 2, 1
LDX v, 84, 1024
RD 144, %2, 53248
SIL 10, 2, 2
SIL 10, 2, 3
RD 148, %2, 54272
SIL 10, 2, 4
RD 152, %2, 55296
STADR 10
LDX v, 88, 2048
RD 156, %2, 56320
SIL 10, 2, 5
SIL 10, 2, 6
SIL 10, 2, 7
LDX v, 92, 3072
SIL 11, 0, 0
SIL 11, 0, 1
s_branch .Ljoin%=_5_1
.Lspec%=_5_2:
s_waitcnt lgkmcnt(0)
RD 160, %2, 57344
SIL 11, 0, 2
SIL 11, 0, 3
STORE 10
RD 164, %2, 58368
SIL 11, 0, 4
SIL 11, 0, 5
SIL 11, 0, 6
RD 168, %2, 59392
SIL 11, 0, 7
RD 172, %2, 60416
SIL 11, 1, 0
RD 176, %2, 61440
SIL 11, 1, 1
RD 180, %2, 62464
SIL 11, 1, 2
RD 184, %2, 63488
SIL 11, 1, 3
RD 188, %2, 64512
SIL 11, 1, 4
SIL 11, 1, 5
CHKF 7
s_cbranch_scc0 .Lslow%=_G_5_2s
.Lback%=_G_5_2s:
SIL 11, 1, 6
s_add_u32 s40, s40, 2621440
SIL 11, 1, 7
.Lentry%=:
s_add_u32 s49, s49, 10240
s_cmp_eq_u32 s49, 40960
s_cselect_b32 s49, 0, s49
SIL 11, 2, 0
v_mov_b32 v206, v207
buffer_load_dword v208, v206, %6, s49 offen sc0 sc1
s_branch .Ljoin%=_5_2
.Lspec%=_5_3:
s_waitcnt lgkmcnt(0)
s_waitcnt vmcnt(11)
s_barrier
RD 128, v211, 0
RD 132, v211, 1024
SETW 98304, 42
LDW 0
LDW 1024
RD 136, v211, 2048
RD 140, v211, 3072
SETW 106496, 43
LDW 0
LDW 1024
RD 144, v211, 4096
RD 148, v211, 5120
SIL 11, 2, 1
RD 152, v211, 6144
SETW 114688, 44
SIL 11, 2, 2
LDW 0
LDW 1024
RD 156, v211, 7168
SETW 122880, 45
SIL 11, 2, 3
LDW 0
LDW 1024
SIL 11, 2, 4
STADR 11
SIL 11, 2, 5
SIL 11, 2, 6
SIL 11, 2, 7
s_branch .Ljoin%=_5_3
.Lspec%=_6_0:
s_waitcnt vmcnt(17) lgkmcnt(0)
RD 160, v211, 8192
s_add_u32 s50, s40, s48
SIL 12, 0, 0
LDX v, 96, 0
RD 164, v211, 9216
SIL 12, 0, 1
SIL 12, 0, 2
SIL 12, 0, 3
SIL 12, 0, 4
RD 168, v211, 10240
SIL 12, 0, 5
SIL 12, 0, 6
SIL 12, 0, 7
s_waitcnt vmcnt(15)
RD 172, v211, 11264
SIL 12, 1, 0
SIL 12, 1, 1
LDX v, 100, 1024
RD 176, v211, 12288
SIL 12, 1, 2
SIL 12, 1, 3
RD 180, v211, 13312
SIL 12, 1, 4
SIL 12, 1, 5
SIL 12, 1, 6
SIL 12, 1, 7
s_waitcnt vmcnt(14)
RD 184, v211, 14336
SIL 12, 2, 0
SIL 12, 2, 1
LDX v, 104, 2048
RD 188, v211, 15360
SIL 12, 2, 2
SIL 12, 2, 3
STORE 11
s_waitcnt vmcnt(14)
SIL 12, 2, 4
STADR 12
LDX v, 108, 3072
SIL 12, 2, 5
SIL 12, 2, 6
s_branch .Ljoin%=_6_0
.Lspec%=_6_1:
s_waitcnt lgkmcnt(0)
RD 128, v211, 16384
s_add_u32 s50, s50, 4096
SIL 12, 2, 7
LDX v, 112, 0
RD 132, v211, 17408
SIL 13, 0, 0
SIL 13, 0, 1
RD 136, v211, 18432
SIL 13, 0, 2
RD 140, v211, 19456
SIL 13, 0, 3
LDX v, 116, 1024
RD 144, v211, 20480
SIL 13, 0, 4
SIL 13, 0, 5
RD 148, v211, 21504
SIL 13, 0, 6
RD 152, v211, 22528
SIL 13, 0, 7
LDX v, 120, 2048
RD 156, v211, 23552
SIL 13, 1, 0
SIL 13, 1, 1
v_readlane_b32 s42, v205, 0
v_readlane_b32 s43, v205, 1
SIL 13, 1, 2
v_readlane_b32 s44, v205, 2
v_readlane_b32 s45, v205, 3
SIL 13, 1, 3
LDX v, 124, 3072
v_readlane_b32 s48, v205, 6
SIL 13, 1, 4
SIL 13, 1, 5
s_branch .Ljoin%=_6_1
.Lspec%=_6_2:
s_waitcnt lgkmcnt(0)
RD 160, v211, 24576
SIL 13, 1, 6
SIL 13, 1, 7
STORE 12
RD 164, v211, 25600
SIL 13, 2, 0
SIL 13, 2, 1
SIL 13, 2, 2
RD 168, v211, 26624
SIL 13, 2, 3
RD 172, v211, 27648
SIL 13, 2, 4
RD 176, v211, 28672
STADR 13
RD 180, v211, 29696
SIL 13, 2, 5
RD 184, v211, 30720
SIL 13, 2, 6
RD 188, v211, 31744
SIL 13, 2, 7
s_waitcnt vmcnt(18)
v_cmp_le_u32 vcc, s53, v208
CHKF 0
s_cbranch_scc0 .Lslow%=_G_6_2s
.Lback%=_G_6_2s:
SIL 14, 0, 0
s_add_u32 s40, s40, 2621440
s_cmp_eq_u32 s40, s60
s_cselect_b32 s40, %13, s40
SIL 14, 0, 1
STORE 13
SIL 14, 0, 2
SIL 14, 0, 3
s_branch .Ljoin%=_6_2
.Lspec%=_6_3:
s_waitcnt lgkmcnt(0)
s_waitcnt vmcnt(11)
s_barrier
RD 128, v211, 32768
RD 132, v211, 33792
SETW 0, 42
SIL 14, 0, 4
LDW 0
LDW 1024
RD 136, v211, 34816
RD 140, v211, 35840
SETW 8192, 43
SIL 14, 0, 5
LDW 0
LDW 1024
RD 144, v211, 36864
RD 148, v211, 37888
SIL 14, 0, 6
SIL 14, 0, 7
RD 152, v211, 38912
SETW 16384, 44
SIL 14, 1, 0
LDW 0
LDW 1024
RD 156, v211, 39936
SETW 24576, 45
SIL 14, 1, 1
LDW 0
LDW 1024
SIL 14, 1, 2
SIL 14, 1, 3
SIL 14, 1, 4
s_branch .Ljoin%=_6_3
.Lspec%=_7_0:
s_waitcnt vmcnt(17) lgkmcnt(0)
RD 160, v211, 40960
s_add_u32 s50, s40, s48
SIL 14, 1, 5
SIL 14, 1, 6
SIL 14, 1, 7
LDX v, 64, 0
RD 164, v211, 41984
SIL 14, 2, 0
SIL 14, 2, 1
RD 168, v211, 43008
SIL 14, 2, 2
SIL 14, 2, 3
s_waitcnt vmcnt(15)
RD 172, v211, 44032
SIL 14, 2, 4
STADR 14
SIL 14, 2, 5
SIL 14, 2, 6
LDX v, 68, 1024
RD 176, v211, 45056
SIL 14, 2, 7
SIL 15, 0, 0
RD 180, v211, 46080
SIL 15, 0, 1
SIL 15, 0, 2
s_waitcnt vmcnt(14)
RD 184, v211, 47104
SIL 15, 0, 3
LDX v, 72, 2048
RD 188, v211, 48128
SIL 15, 0, 4
SIL 15, 0, 5
STORE 14
s_waitcnt vmcnt(14)
SIL 15, 0, 6
SIL 15, 0, 7
LDX v, 76, 3072
v_readlane_b32 s62, v205, 4
v_readlane_b32 s63, v205, 5
SIL 15, 1, 0
s_branch .Ljoin%=_7_0
.Lspec%=_7_1:
s_waitcnt lgkmcnt(0)
RD 128, v211, 49152
s_add_u32 s50, s50, 4096
SIL 15, 1, 1
LDX v, 80, 0
RD 132, v211, 50176
SIL 15, 1, 2
SIL 15, 1, 3
RD 136, v211, 51200
SIL 15, 1, 4
SIL 15, 1, 5
SIL 15, 1, 6
RD 140, v211, 52224
SIL 15, 1, 7
LDX v, 84, 1024
RD 144, v211, 53248
SIL 15, 2, 0
SIL 15, 2, 1
RD 148, v211, 54272
SIL 15, 2, 2
RD 152, v211, 55296
SIL 15, 2, 3
LDX v, 88, 2048
RD 156, v211, 56320
SIL 15, 2, 4
STADR 15
SIL 15, 2, 5
SIL 15, 2, 6
LDX v, 92, 3072
v_readlane_b32 s72, v205, 7
SIL 15, 2, 7
s_branch .Ljoin%=_7_1
.Lspec%=_7_2:
s_waitcnt lgkmcnt(0)
RD 160, v211, 57344
s_bitcmp1_b32 s62, 4
s_cselect_b32 s42, 0x80000000, s42
s_bitcmp1_b32 s62, 5
s_cselect_b32 s43, 0x80000000, s43
RD 164, v211, 58368
s_bitcmp1_b32 s62, 6
s_cselect_b32 s44, 0x80000000, s44
s_bitcmp1_b32 s62, 7
s_cselect_b32 s45, 0x80000000, s45
RD 168, v211, 59392
RD 172, v211, 60416
RD 176, v211, 61440
RD 180, v211, 62464
RD 184, v211, 63488
RD 188, v211, 64512
STORE 15
CHKF 1
s_cbranch_scc0 .Lslow%=_G_7_2s
.Lback%=_G_7_2s:
s_add_u32 s40, s40, 2621440
buffer_load_dword v208, v206, %6, s49 offen sc0 sc1
s_branch .Ljoin%=_7_2
.Lspec%=_7_3:
s_waitcnt lgkmcnt(0)
s_waitcnt vmcnt(11)
s_barrier
RD 128, %2, 0
RD 132, %2, 1024
SETW 32768, 42
LDW 0
LDW 1024
RD 136, %2, 2048
RD 140, %2, 3072
SETW 40960, 43
LDW 0
LDW 1024
RD 144, %2, 4096
RD 148, %2, 5120
RD 152, %2, 6144
SETW 49152, 44
LDW 0
LDW 1024
RD 156, %2, 7168
SETW 57344, 45
LDW 0
LDW 1024
s_mov_b64 s[46:47], s[62:63]
s_sub_u32 s54, s54, 1
s_branch .Ljoin%=_7_3
  .p2align 6
.Ltop%=:
s_bitcmp1_b32 s46, 0
s_cbranch_scc1 .Lspec%=_0_0
s_waitcnt vmcnt(17) lgkmcnt(0)
MM 0, 1, v, 64, 128
RD 160, %2, 8192
s_add_u32 s50, s40, s48
MM 4, 1, v, 64, 136
LDX v, 96, 0
MM 8, 1, v, 64, 144
RD 164, %2, 9216
MM 12, 1, v, 64, 152
RD 168, %2, 10240
s_waitcnt vmcnt(15)
MM 28, 1, v, 72, 152
RD 172, %2, 11264
MM 24, 1, v, 72, 144
LDX v, 100, 1024
MM 20, 1, v, 72, 136
RD 176, %2, 12288
MM 16, 1, v, 72, 128
RD 180, %2, 13312
s_waitcnt vmcnt(14)
MM 32, 1, v, 80, 128
RD 184, %2, 14336
MM 36, 1, v, 80, 136
LDX v, 104, 2048
MM 40, 1, v, 80, 144
RD 188, %2, 15360
MM 44, 1, v, 80, 152
s_waitcnt vmcnt(13)
MM 60, 1, v, 88, 152
MM 56, 1, v, 88, 144
LDX v, 108, 3072
MM 52, 1, v, 88, 136
MM 48, 1, v, 88, 128
.Ljoin%=_0_0:
s_bitcmp1_b32 s46, 1
s_cbranch_scc1 .Lspec%=_0_1
s_waitcnt lgkmcnt(0)
MM 112, 1, v, 88, 160
RD 128, %2, 16384
s_add_u32 s50, s50, 4096
MM 116, 1, v, 88, 168
LDX v, 112, 0
MM 120, 1, v, 88, 176
RD 132, %2, 17408
MM 124, 1, v, 88, 184
RD 136, %2, 18432
MM 108, 1, v, 80, 184
RD 140, %2, 19456
MM 104, 1, v, 80, 176
LDX v, 116, 1024
MM 100, 1, v, 80, 168
RD 144, %2, 20480
MM 96, 1, v, 80, 160
RD 148, %2, 21504
MM 80, 1, v, 72, 160
RD 152, %2, 22528
MM 84, 1, v, 72, 168
LDX v, 120, 2048
MM 88, 1, v, 72, 176
RD 156, %2, 23552
MM 92, 1, v, 72, 184
s_add_u32 s72, s72, s66
MM 76, 1, v, 64, 184
MM 72, 1, v, 64, 176
LDX v, 124, 3072
MM 68, 1, v, 64, 168
MM 64, 1, v, 64, 160
.Ljoin%=_0_1:
s_bitcmp1_b32 s46, 2
s_cbranch_scc1 .Lspec%=_0_2
s_waitcnt lgkmcnt(0)
MM 128, 1, v, 64, 128
RD 160, %2, 24576
MM 132, 1, v, 64, 136
MM 136, 1, v, 64, 144
RD 164, %2, 25600
MM 140, 1, v, 64, 152
MM 156, 1, v, 72, 152
RD 168, %2, 26624
MM 152, 1, v, 72, 144
RD 172, %2, 27648
MM 148, 1, v, 72, 136
RD 176, %2, 28672
MM 144, 1, v, 72, 128
RD 180, %2, 29696
MM 160, 1, v, 80, 128
RD 184, %2, 30720
MM 164, 1, v, 80, 136
RD 188, %2, 31744
MM 168, 1, v, 80, 144
s_waitcnt vmcnt(16)
v_cmp_le_u32 vcc, s53, v208
MM 172, 1, v, 80, 152
CHKF 2
s_cbranch_scc0 .Lslow%=_G_0_2n
.Lback%=_G_0_2n:
MM 188, 1, v, 88, 152
s_add_u32 s40, s40, 2621440
MM 184, 1, v, 88, 144
MM 180, 1, v, 88, 136
MM 176, 1, v, 88, 128
.Ljoin%=_0_2:
s_bitcmp1_b32 s46, 3
s_cbranch_scc1 .Lspec%=_0_3
s_waitcnt lgkmcnt(0)
MM 240, 1, v, 88, 160
s_waitcnt vmcnt(8)
s_barrier
MM 244, 1, v, 88, 168
RD 128, %2, 32768
RD 132, %2, 33792
SETW 65536, 42
MM 248, 1, v, 88, 176
LDW 0
MM 252, 1, v, 88, 184
LDW 1024
MM 236, 1, v, 80, 184
RD 136, %2, 34816
RD 140, %2, 35840
SETW 73728, 43
MM 232, 1, v, 80, 176
LDW 0
MM 228, 1, v, 80, 168
LDW 1024
MM 224, 1, v, 80, 160
RD 144, %2, 36864
RD 148, %2, 37888
MM 208, 1, v, 72, 160
RD 152, %2, 38912
SETW 81920, 44
MM 212, 1, v, 72, 168
LDW 0
MM 216, 1, v, 72, 176
LDW 1024
MM 220, 1, v, 72, 184
RD 156, %2, 39936
SETW 90112, 45
MM 204, 1, v, 64, 184
LDW 0
MM 200, 1, v, 64, 176
LDW 1024
MM 196, 1, v, 64, 168
MM 192, 1, v, 64, 160
.Ljoin%=_0_3:
s_bitcmp1_b32 s46, 4
s_cbranch_scc1 .Lspec%=_1_0
s_waitcnt vmcnt(14) lgkmcnt(0)
MM 0, 1, v, 96, 128
RD 160, %2, 40960
s_add_u32 s50, s40, s48
SIL 0, 0, 0
SIL 0, 0, 1
MM 4, 1, v, 96, 136
LDX v, 64, 0
MM 8, 1, v, 96, 144
RD 164, %2, 41984
SIL 0, 0, 2
SIL 0, 0, 3
MM 12, 1, v, 96, 152
RD 168, %2, 43008
SIL 0, 0, 4
s_waitcnt vmcnt(13)
MM 28, 1, v, 104, 152
RD 172, %2, 44032
SIL 0, 0, 5
MM 24, 1, v, 104, 144
LDX v, 68, 1024
MM 20, 1, v, 104, 136
RD 176, %2, 45056
SIL 0, 0, 6
SIL 0, 0, 7
MM 16, 1, v, 104, 128
RD 180, %2, 46080
SIL 0, 1, 0
s_waitcnt vmcnt(12)
MM 32, 1, v, 112, 128
RD 184, %2, 47104
SIL 0, 1, 1
MM 36, 1, v, 112, 136
LDX v, 72, 2048
MM 40, 1, v, 112, 144
RD 188, %2, 48128
SIL 0, 1, 2
SIL 0, 1, 3
MM 44, 1, v, 112, 152
SIL 0, 1, 4
s_waitcnt vmcnt(11)
MM 60, 1, v, 120, 152
SIL 0, 1, 5
MM 56, 1, v, 120, 144
LDX v, 76, 3072
MM 52, 1, v, 120, 136
SIL 0, 1, 6
SIL 0, 1, 7
MM 48, 1, v, 120, 128
.Ljoin%=_1_0:
s_bitcmp1_b32 s46, 5
s_cbranch_scc1 .Lspec%=_1_1
s_waitcnt lgkmcnt(0)
MM 112, 1, v, 120, 160
RD 128, %2, 49152
s_add_u32 s50, s50, 4096
SIL 0, 2, 0
MM 116, 1, v, 120, 168
LDX v, 80, 0
MM 120, 1, v, 120, 176
RD 132, %2, 50176
SIL 0, 2, 1
SIL 0, 2, 2
MM 124, 1, v, 120, 184
RD 136, %2, 51200
SIL 0, 2, 3
SIL 0, 2, 4
MM 108, 1, v, 112, 184
RD 140, %2, 52224
STADR 0
MM 104, 1, v, 112, 176
LDX v, 84, 1024
MM 100, 1, v, 112, 168
RD 144, %2, 53248
SIL 0, 2, 5
SIL 0, 2, 6
MM 96, 1, v, 112, 160
RD 148, %2, 54272
SIL 0, 2, 7
MM 80, 1, v, 104, 160
RD 152, %2, 55296
DUP 0, 0
MM 84, 1, v, 104, 168
LDX v, 88, 2048
MM 88, 1, v, 104, 176
RD 156, %2, 56320
SIL 1, 0, 0
SIL 1, 0, 1
MM 92, 1, v, 104, 184
buffer_store_dword v214, off, %6, s72 sc1
MM 76, 1, v, 96, 184
SIL 1, 0, 2
SIL 1, 0, 3
MM 72, 1, v, 96, 176
LDX v, 92, 3072
MM 68, 1, v, 96, 168
SIL 1, 0, 4
SIL 1, 0, 5
MM 64, 1, v, 96, 160
.Ljoin%=_1_1:
s_bitcmp1_b32 s46, 6
s_cbranch_scc1 .Lspec%=_1_2
s_waitcnt lgkmcnt(0)
MM 128, 1, v, 96, 128
RD 160, %2, 57344
SIL 1, 0, 6
SIL 1, 0, 7
MM 132, 1, v, 96, 136
STORE 0
MM 136, 1, v, 96, 144
RD 164, %2, 58368
SIL 1, 1, 0
SIL 1, 1, 1
MM 140, 1, v, 96, 152
SIL 1, 1, 2
MM 156, 1, v, 104, 152
RD 168, %2, 59392
SIL 1, 1, 3
MM 152, 1, v, 104, 144
RD 172, %2, 60416
SIL 1, 1, 4
MM 148, 1, v, 104, 136
RD 176, %2, 61440
SIL 1, 1, 5
MM 144, 1, v, 104, 128
RD 180, %2, 62464
SIL 1, 1, 6
MM 160, 1, v, 112, 128
RD 184, %2, 63488
SIL 1, 1, 7
MM 164, 1, v, 112, 136
RD 188, %2, 64512
SIL 1, 2, 0
MM 168, 1, v, 112, 144
SIL 1, 2, 1
MM 172, 1, v, 112, 152
CHKF 3
s_cbranch_scc0 .Lslow%=_G_1_2n
.Lback%=_G_1_2n:
SIL 1, 2, 2
MM 188, 1, v, 120, 152
s_add_u32 s40, s40, 2621440
SIL 1, 2, 3
MM 184, 1, v, 120, 144
SIL 1, 2, 4
MM 180, 1, v, 120, 136
buffer_load_dword v208, v206, %6, s49 offen sc0 sc1
MM 176, 1, v, 120, 128
.Ljoin%=_1_2:
s_bitcmp1_b32 s46, 7
s_cbranch_scc1 .Lspec%=_1_3
s_waitcnt lgkmcnt(0)
MM 240, 1, v, 120, 160
s_waitcnt vmcnt(11)
s_barrier
MM 244, 1, v, 120, 168
RD 128, v211, 0
RD 132, v211, 1024
SETW 98304, 42
STADR 1
MM 248, 1, v, 120, 176
LDW 0
MM 252, 1, v, 120, 184
LDW 1024
MM 236, 1, v, 112, 184
RD 136, v211, 2048
RD 140, v211, 3072
SETW 106496, 43
SIL 1, 2, 5
MM 232, 1, v, 112, 176
LDW 0
MM 228, 1, v, 112, 168
LDW 1024
MM 224, 1, v, 112, 160
RD 144, v211, 4096
RD 148, v211, 5120
SIL 1, 2, 6
SIL 1, 2, 7
MM 208, 1, v, 104, 160
RD 152, v211, 6144
SETW 114688, 44
DUP 1, 1
MM 212, 1, v, 104, 168
LDW 0
MM 216, 1, v, 104, 176
LDW 1024
MM 220, 1, v, 104, 184
RD 156, v211, 7168
SETW 122880, 45
SIL 2, 0, 0
SIL 2, 0, 1
MM 204, 1, v, 96, 184
LDW 0
MM 200, 1, v, 96, 176
LDW 1024
MM 196, 1, v, 96, 168
SIL 2, 0, 2
SIL 2, 0, 3
SIL 2, 0, 4
SIL 2, 0, 5
SIL 2, 0, 6
MM 192, 1, v, 96, 160
.Ljoin%=_1_3:
s_bitcmp1_b32 s46, 4
s_cbranch_scc1 .Lspec%=_2_0
s_waitcnt vmcnt(17) lgkmcnt(0)
MM 0, 1, v, 64, 128
RD 160, v211, 8192
s_add_u32 s50, s40, s48
SIL 2, 0, 7
SIL 2, 1, 0
MM 4, 1, v, 64, 136
LDX v, 96, 0
MM 8, 1, v, 64, 144
RD 164, v211, 9216
SIL 2, 1, 1
SIL 2, 1, 2
MM 12, 1, v, 64, 152
RD 168, v211, 10240
SIL 2, 1, 3
SIL 2, 1, 4
SIL 2, 1, 5
s_waitcnt vmcnt(16)
MM 28, 1, v, 72, 152
RD 172, v211, 11264
SIL 2, 1, 6
SIL 2, 1, 7
SIL 2, 2, 0
MM 24, 1, v, 72, 144
LDX v, 100, 1024
MM 20, 1, v, 72, 136
RD 176, v211, 12288
SIL 2, 2, 1
SIL 2, 2, 2
MM 16, 1, v, 72, 128
RD 180, v211, 13312
SIL 2, 2, 3
SIL 2, 2, 4
s_waitcnt vmcnt(15)
MM 32, 1, v, 80, 128
RD 184, v211, 14336
STADR 2
MM 36, 1, v, 80, 136
LDX v, 104, 2048
MM 40, 1, v, 80, 144
RD 188, v211, 15360
SIL 2, 2, 5
SIL 2, 2, 6
MM 44, 1, v, 80, 152
STORE 1
s_waitcnt vmcnt(14)
MM 60, 1, v, 88, 152
SIL 2, 2, 7
DUP 2, 2
MM 56, 1, v, 88, 144
LDX v, 108, 3072
MM 52, 1, v, 88, 136
v_add_u32 v210, 64, v210
SIL 3, 0, 0
SIL 3, 0, 1
MM 48, 1, v, 88, 128
.Ljoin%=_2_0:
s_bitcmp1_b32 s46, 5
s_cbranch_scc1 .Lspec%=_2_1
s_waitcnt lgkmcnt(0)
MM 112, 1, v, 88, 160
RD 128, v211, 16384
s_add_u32 s50, s50, 4096
SIL 3, 0, 2
SIL 3, 0, 3
MM 116, 1, v, 88, 168
LDX v, 112, 0
MM 120, 1, v, 88, 176
RD 132, v211, 17408
SIL 3, 0, 4
SIL 3, 0, 5
MM 124, 1, v, 88, 184
RD 136, v211, 18432
SIL 3, 0, 6
MM 108, 1, v, 80, 184
RD 140, v211, 19456
SIL 3, 0, 7
MM 104, 1, v, 80, 176
LDX v, 116, 1024
MM 100, 1, v, 80, 168
RD 144, v211, 20480
SIL 3, 1, 0
SIL 3, 1, 1
MM 96, 1, v, 80, 160
RD 148, v211, 21504
SIL 3, 1, 2
MM 80, 1, v, 72, 160
RD 152, v211, 22528
SIL 3, 1, 3
MM 84, 1, v, 72, 168
LDX v, 120, 2048
MM 88, 1, v, 72, 176
RD 156, v211, 23552
SIL 3, 1, 4
SIL 3, 1, 5
MM 92, 1, v, 72, 184
STORE 2
MM 76, 1, v, 64, 184
SIL 3, 1, 6
SIL 3, 1, 7
MM 72, 1, v, 64, 176
LDX v, 124, 3072
MM 68, 1, v, 64, 168
SIL 3, 2, 0
SIL 3, 2, 1
MM 64, 1, v, 64, 160
.Ljoin%=_2_1:
s_bitcmp1_b32 s46, 6
s_cbranch_scc1 .Lspec%=_2_2
s_waitcnt lgkmcnt(0)
MM 128, 1, v, 64, 128
RD 160, v211, 24576
SIL 3, 2, 2
SIL 3, 2, 3
MM 132, 1, v, 64, 136
SIL 3, 2, 4
MM 136, 1, v, 64, 144
RD 164, v211, 25600
STADR 3
MM 140, 1, v, 64, 152
SIL 3, 2, 5
MM 156, 1, v, 72, 152
RD 168, v211, 26624
SIL 3, 2, 6
MM 152, 1, v, 72, 144
RD 172, v211, 27648
SIL 3, 2, 7
MM 148, 1, v, 72, 136
RD 176, v211, 28672
DUP 3, 3
MM 144, 1, v, 72, 128
RD 180, v211, 29696
SIL 4, 0, 0
MM 160, 1, v, 80, 128
RD 184, v211, 30720
SIL 4, 0, 1
MM 164, 1, v, 80, 136
RD 188, v211, 31744
SIL 4, 0, 2
MM 168, 1, v, 80, 144
s_waitcnt vmcnt(18)
v_cmp_le_u32 vcc, s53, v208
SIL 4, 0, 3
MM 172, 1, v, 80, 152
CHKF 4
s_cbranch_scc0 .Lslow%=_G_2_2n
.Lback%=_G_2_2n:
SIL 4, 0, 4
MM 188, 1, v, 88, 152
s_add_u32 s40, s40, 2621440
SIL 4, 0, 5
MM 184, 1, v, 88, 144
STORE 3
MM 180, 1, v, 88, 136
SIL 4, 0, 6
SIL 4, 0, 7
MM 176, 1, v, 88, 128
.Ljoin%=_2_2:
s_bitcmp1_b32 s46, 7
s_cbranch_scc1 .Lspec%=_2_3
s_waitcnt lgkmcnt(0)
MM 240, 1, v, 88, 160
s_waitcnt vmcnt(11)
s_barrier
MM 244, 1, v, 88, 168
RD 128, v211, 32768
RD 132, v211, 33792
SETW 0, 42
MM 248, 1, v, 88, 176
LDW 0
MM 252, 1, v, 88, 184
LDW 1024
MM 236, 1, v, 80, 184
RD 136, v211, 34816
RD 140, v211, 35840
SETW 8192, 43
MM 232, 1, v, 80, 176
LDW 0
MM 228, 1, v, 80, 168
LDW 1024
MM 224, 1, v, 80, 160
RD 144, v211, 36864
RD 148, v211, 37888
SIL 4, 1, 0
MM 208, 1, v, 72, 160
RD 152, v211, 38912
SETW 16384, 44
SIL 4, 1, 1
MM 212, 1, v, 72, 168
LDW 0
MM 216, 1, v, 72, 176
LDW 1024
MM 220, 1, v, 72, 184
RD 156, v211, 39936
SETW 24576, 45
SIL 4, 1, 2
MM 204, 1, v, 64, 184
LDW 0
MM 200, 1, v, 64, 176
LDW 1024
MM 196, 1, v, 64, 168
SIL 4, 1, 3
SIL 4, 1, 4
SIL 4, 1, 5
SIL 4, 1, 6
MM 192, 1, v, 64, 160
.Ljoin%=_2_3:
s_bitcmp1_b32 s46, 4
s_cbranch_scc1 .Lspec%=_3_0
s_waitcnt vmcnt(17) lgkmcnt(0)
MM 0, 1, v, 96, 128
RD 160, v211, 40960
s_add_u32 s50, s40, s48
SIL 4, 1, 7
SIL 4, 2, 0
MM 4, 1, v, 96, 136
LDX v, 64, 0
MM 8, 1, v, 96, 144
RD 164, v211, 41984
SIL 4, 2, 1
SIL 4, 2, 2
MM 12, 1, v, 96, 152
RD 168, v211, 43008
SIL 4, 2, 3
SIL 4, 2, 4
STADR 4
s_waitcnt vmcnt(15)
MM 28, 1, v, 104, 152
RD 172, v211, 44032
SIL 4, 2, 5
SIL 4, 2, 6
SIL 4, 2, 7
MM 24, 1, v, 104, 144
LDX v, 68, 1024
MM 20, 1, v, 104, 136
RD 176, v211, 45056
DUP 4, 4
SIL 5, 0, 0
SIL 5, 0, 1
MM 16, 1, v, 104, 128
RD 180, v211, 46080
SIL 5, 0, 2
SIL 5, 0, 3
SIL 5, 0, 4
SIL 5, 0, 5
s_waitcnt vmcnt(14)
MM 32, 1, v, 112, 128
RD 184, v211, 47104
SIL 5, 0, 6
SIL 5, 0, 7
MM 36, 1, v, 112, 136
LDX v, 72, 2048
MM 40, 1, v, 112, 144
RD 188, v211, 48128
SIL 5, 1, 0
SIL 5, 1, 1
MM 44, 1, v, 112, 152
STORE 4
s_waitcnt vmcnt(13)
MM 60, 1, v, 120, 152
SIL 5, 1, 2
SIL 5, 1, 3
MM 56, 1, v, 120, 144
LDX v, 76, 3072
MM 52, 1, v, 120, 136
ds_read_b32 v204, v210
ds_read_b32 v205, v210 offset:32
SIL 5, 1, 4
SIL 5, 1, 5
MM 48, 1, v, 120, 128
.Ljoin%=_3_0:
s_bitcmp1_b32 s46, 5
s_cbranch_scc1 .Lspec%=_3_1
s_waitcnt lgkmcnt(2)
MM 112, 1, v, 120, 160
RD 128, v211, 49152
s_add_u32 s50, s50, 4096
SIL 5, 1, 6
SIL 5, 1, 7
MM 116, 1, v, 120, 168
LDX v, 80, 0
MM 120, 1, v, 120, 176
RD 132, v211, 50176
SIL 5, 2, 0
SIL 5, 2, 1
MM 124, 1, v, 120, 184
RD 136, v211, 51200
SIL 5, 2, 2
MM 108, 1, v, 112, 184
RD 140, v211, 52224
SIL 5, 2, 3
MM 104, 1, v, 112, 176
LDX v, 84, 1024
MM 100, 1, v, 112, 168
RD 144, v211, 53248
SIL 5, 2, 4
STADR 5
MM 96, 1, v, 112, 160
RD 148, v211, 54272
SIL 5, 2, 5
MM 80, 1, v, 104, 160
RD 152, v211, 55296
SIL 5, 2, 6
MM 84, 1, v, 104, 168
LDX v, 88, 2048
MM 88, 1, v, 104, 176
RD 156, v211, 56320
SIL 5, 2, 7
DUP 5, 5
MM 92, 1, v, 104, 184
STORE 5
MM 76, 1, v, 96, 184
SIL 6, 0, 0
SIL 6, 0, 1
MM 72, 1, v, 96, 176
LDX v, 92, 3072
MM 68, 1, v, 96, 168
SIL 6, 0, 2
SIL 6, 0, 3
MM 64, 1, v, 96, 160
.Ljoin%=_3_1:
s_bitcmp1_b32 s46, 6
s_cbranch_scc1 .Lspec%=_3_2
s_waitcnt lgkmcnt(0)
MM 128, 1, v, 96, 128
RD 160, v211, 57344
SIL 6, 0, 4
SIL 6, 0, 5
MM 132, 1, v, 96, 136
SIL 6, 0, 6
MM 136, 1, v, 96, 144
RD 164, v211, 58368
SIL 6, 0, 7
MM 140, 1, v, 96, 152
SIL 6, 1, 0
MM 156, 1, v, 104, 152
RD 168, v211, 59392
SIL 6, 1, 1
MM 152, 1, v, 104, 144
RD 172, v211, 60416
SIL 6, 1, 2
MM 148, 1, v, 104, 136
RD 176, v211, 61440
SIL 6, 1, 3
MM 144, 1, v, 104, 128
RD 180, v211, 62464
SIL 6, 1, 4
MM 160, 1, v, 112, 128
RD 184, v211, 63488
SIL 6, 1, 5
MM 164, 1, v, 112, 136
RD 188, v211, 64512
SIL 6, 1, 6
MM 168, 1, v, 112, 144
SIL 6, 1, 7
MM 172, 1, v, 112, 152
CHKF 5
s_cbranch_scc0 .Lslow%=_G_3_2n
.Lback%=_G_3_2n:
SIL 6, 2, 0
MM 188, 1, v, 120, 152
s_add_u32 s40, s40, 2621440
SIL 6, 2, 1
MM 184, 1, v, 120, 144
SIL 6, 2, 2
MM 180, 1, v, 120, 136
buffer_load_dword v208, v206, %6, s49 offen sc0 sc1
MM 176, 1, v, 120, 128
.Ljoin%=_3_2:
s_bitcmp1_b32 s46, 7
s_cbranch_scc1 .Lspec%=_3_3
s_waitcnt lgkmcnt(0)
MM 240, 1, v, 120, 160
s_waitcnt vmcnt(11)
s_barrier
MM 244, 1, v, 120, 168
RD 128, %2, 0
RD 132, %2, 1024
SETW 32768, 42
MM 248, 1, v, 120, 176
LDW 0
MM 252, 1, v, 120, 184
LDW 1024
MM 236, 1, v, 112, 184
RD 136, %2, 2048
RD 140, %2, 3072
SETW 40960, 43
MM 232, 1, v, 112, 176
LDW 0
MM 228, 1, v, 112, 168
LDW 1024
MM 224, 1, v, 112, 160
RD 144, %2, 4096
RD 148, %2, 5120
SIL 6, 2, 3
SIL 6, 2, 4
MM 208, 1, v, 104, 160
RD 152, %2, 6144
SETW 49152, 44
STADR 6
SIL 6, 2, 5
MM 212, 1, v, 104, 168
LDW 0
MM 216, 1, v, 104, 176
LDW 1024
MM 220, 1, v, 104, 184
RD 156, %2, 7168
SETW 57344, 45
SIL 6, 2, 6
SIL 6, 2, 7
MM 204, 1, v, 96, 184
LDW 0
MM 200, 1, v, 96, 176
LDW 1024
MM 196, 1, v, 96, 168
DUP 6, 6
SIL 7, 0, 0
SIL 7, 0, 1
SIL 7, 0, 2
MM 192, 1, v, 96, 160
.Ljoin%=_3_3:
s_bitcmp1_b32 s46, 4
s_cbranch_scc1 .Lspec%=_4_0
s_waitcnt vmcnt(17) lgkmcnt(0)
MM 0, 1, v, 64, 128
RD 160, %2, 8192
s_add_u32 s50, s40, s48
SIL 7, 0, 3
SIL 7, 0, 4
SIL 7, 0, 5
MM 4, 1, v, 64, 136
LDX v, 96, 0
MM 8, 1, v, 64, 144
RD 164, %2, 9216
SIL 7, 0, 6
SIL 7, 0, 7
SIL 7, 1, 0
MM 12, 1, v, 64, 152
RD 168, %2, 10240
SIL 7, 1, 1
SIL 7, 1, 2
s_waitcnt vmcnt(15)
MM 28, 1, v, 72, 152
RD 172, %2, 11264
SIL 7, 1, 3
SIL 7, 1, 4
SIL 7, 1, 5
MM 24, 1, v, 72, 144
LDX v, 100, 1024
MM 20, 1, v, 72, 136
RD 176, %2, 12288
SIL 7, 1, 6
SIL 7, 1, 7
SIL 7, 2, 0
MM 16, 1, v, 72, 128
RD 180, %2, 13312
SIL 7, 2, 1
SIL 7, 2, 2
s_waitcnt vmcnt(14)
MM 32, 1, v, 80, 128
RD 184, %2, 14336
SIL 7, 2, 3
MM 36, 1, v, 80, 136
LDX v, 104, 2048
MM 40, 1, v, 80, 144
RD 188, %2, 15360
SIL 7, 2, 4
STADR 7
MM 44, 1, v, 80, 152
STORE 6
s_waitcnt vmcnt(13)
MM 60, 1, v, 88, 152
SIL 7, 2, 5
SIL 7, 2, 6
MM 56, 1, v, 88, 144
LDX v, 108, 3072
MM 52, 1, v, 88, 136
v_add_u32 v207, v204, %5
SIL 7, 2, 7
DUP 7, 7
MM 48, 1, v, 88, 128
.Ljoin%=_4_0:
s_bitcmp1_b32 s46, 5
s_cbranch_scc1 .Lspec%=_4_1
s_waitcnt lgkmcnt(0)
MM 112, 1, v, 88, 160
RD 128, %2, 16384
s_add_u32 s50, s50, 4096
SIL 8, 0, 0
SIL 8, 0, 1
MM 116, 1, v, 88, 168
LDX v, 112, 0
MM 120, 1, v, 88, 176
RD 132, %2, 17408
SIL 8, 0, 2
SIL 8, 0, 3
MM 124, 1, v, 88, 184
RD 136, %2, 18432
SIL 8, 0, 4
MM 108, 1, v, 80, 184
RD 140, %2, 19456
SIL 8, 0, 5
MM 104, 1, v, 80, 176
LDX v, 116, 1024
MM 100, 1, v, 80, 168
RD 144, %2, 20480
SIL 8, 0, 6
SIL 8, 0, 7
MM 96, 1, v, 80, 160
RD 148, %2, 21504
SIL 8, 1, 0
MM 80, 1, v, 72, 160
RD 152, %2, 22528
SIL 8, 1, 1
MM 84, 1, v, 72, 168
LDX v, 120, 2048
MM 88, 1, v, 72, 176
RD 156, %2, 23552
SIL 8, 1, 2
SIL 8, 1, 3
MM 92, 1, v, 72, 184
STORE 7
MM 76, 1, v, 64, 184
SIL 8, 1, 4
SIL 8, 1, 5
MM 72, 1, v, 64, 176
LDX v, 124, 3072
MM 68, 1, v, 64, 168
SIL 8, 1, 6
SIL 8, 1, 7
MM 64, 1, v, 64, 160
.Ljoin%=_4_1:
s_bitcmp1_b32 s46, 6
s_cbranch_scc1 .Lspec%=_4_2
s_waitcnt lgkmcnt(0)
MM 128, 1, v, 64, 128
RD 160, %2, 24576
SIL 8, 2, 0
SIL 8, 2, 1
MM 132, 1, v, 64, 136
SIL 8, 2, 2
MM 136, 1, v, 64, 144
RD 164, %2, 25600
SIL 8, 2, 3
MM 140, 1, v, 64, 152
SIL 8, 2, 4
MM 156, 1, v, 72, 152
RD 168, %2, 26624
STADR 8
MM 152, 1, v, 72, 144
RD 172, %2, 27648
SIL 8, 2, 5
MM 148, 1, v, 72, 136
RD 176, %2, 28672
SIL 8, 2, 6
MM 144, 1, v, 72, 128
RD 180, %2, 29696
SIL 8, 2, 7
MM 160, 1, v, 80, 128
RD 184, %2, 30720
DUP 8, 8
MM 164, 1, v, 80, 136
RD 188, %2, 31744
SIL 9, 0, 0
MM 168, 1, v, 80, 144
s_waitcnt vmcnt(18)
v_cmp_le_u32 vcc, s53, v208
SIL 9, 0, 1
MM 172, 1, v, 80, 152
CHKF 6
s_cbranch_scc0 .Lslow%=_G_4_2n
.Lback%=_G_4_2n:
SIL 9, 0, 2
MM 188, 1, v, 88, 152
s_add_u32 s40, s40, 2621440
SIL 9, 0, 3
MM 184, 1, v, 88, 144
STORE 8
MM 180, 1, v, 88, 136
SIL 9, 0, 4
SIL 9, 0, 5
MM 176, 1, v, 88, 128
.Ljoin%=_4_2:
s_bitcmp1_b32 s46, 7
s_cbranch_scc1 .Lspec%=_4_3
s_waitcnt lgkmcnt(0)
MM 240, 1, v, 88, 160
s_waitcnt vmcnt(11)
s_barrier
MM 244, 1, v, 88, 168
RD 128, %2, 32768
RD 132, %2, 33792
SETW 65536, 42
SIL 9, 0, 6
MM 248, 1, v, 88, 176
LDW 0
MM 252, 1, v, 88, 184
LDW 1024
MM 236, 1, v, 80, 184
RD 136, %2, 34816
RD 140, %2, 35840
SETW 73728, 43
SIL 9, 0, 7
MM 232, 1, v, 80, 176
LDW 0
MM 228, 1, v, 80, 168
LDW 1024
MM 224, 1, v, 80, 160
RD 144, %2, 36864
RD 148, %2, 37888
SIL 9, 1, 0
MM 208, 1, v, 72, 160
RD 152, %2, 38912
SETW 81920, 44
SIL 9, 1, 1
MM 212, 1, v, 72, 168
LDW 0
MM 216, 1, v, 72, 176
LDW 1024
MM 220, 1, v, 72, 184
RD 156, %2, 39936
SETW 90112, 45
SIL 9, 1, 2
MM 204, 1, v, 64, 184
LDW 0
MM 200, 1, v, 64, 176
LDW 1024
MM 196, 1, v, 64, 168
SIL 9, 1, 3
SIL 9, 1, 4
SIL 9, 1, 5
SIL 9, 1, 6
MM 192, 1, v, 64, 160
.Ljoin%=_4_3:
s_bitcmp1_b32 s46, 4
s_cbranch_scc1 .Lspec%=_5_0
s_waitcnt vmcnt(17) lgkmcnt(0)
MM 0, 1, v, 96, 128
RD 160, %2, 40960
s_add_u32 s50, s40, s48
SIL 9, 1, 7
SIL 9, 2, 0
MM 4, 1, v, 96, 136
LDX v, 64, 0
MM 8, 1, v, 96, 144
RD 164, %2, 41984
SIL 9, 2, 1
SIL 9, 2, 2
MM 12, 1, v, 96, 152
RD 168, %2, 43008
SIL 9, 2, 3
SIL 9, 2, 4
STADR 9
s_waitcnt vmcnt(15)
MM 28, 1, v, 104, 152
RD 172, %2, 44032
SIL 9, 2, 5
SIL 9, 2, 6
SIL 9, 2, 7
MM 24, 1, v, 104, 144
LDX v, 68, 1024
MM 20, 1, v, 104, 136
RD 176, %2, 45056
DUP 9, 9
SIL 10, 0, 0
SIL 10, 0, 1
MM 16, 1, v, 104, 128
RD 180, %2, 46080
SIL 10, 0, 2
SIL 10, 0, 3
SIL 10, 0, 4
s_waitcnt vmcnt(14)
MM 32, 1, v, 112, 128
RD 184, %2, 47104
SIL 10, 0, 5
MM 36, 1, v, 112, 136
LDX v, 72, 2048
MM 40, 1, v, 112, 144
RD 188, %2, 48128
SIL 10, 0, 6
SIL 10, 0, 7
MM 44, 1, v, 112, 152
STORE 9
s_waitcnt vmcnt(13)
MM 60, 1, v, 120, 152
SIL 10, 1, 0
SIL 10, 1, 1
MM 56, 1, v, 120, 144
LDX v, 76, 3072
MM 52, 1, v, 120, 136
SIL 10, 1, 2
SIL 10, 1, 3
MM 48, 1, v, 120, 128
.Ljoin%=_5_0:
s_bitcmp1_b32 s46, 5
s_cbranch_scc1 .Lspec%=_5_1
s_waitcnt lgkmcnt(0)
MM 112, 1, v, 120, 160
RD 128, %2, 49152
s_add_u32 s50, s50, 4096
SIL 10, 1, 4
SIL 10, 1, 5
MM 116, 1, v, 120, 168
LDX v, 80, 0
MM 120, 1, v, 120, 176
RD 132, %2, 50176
SIL 10, 1, 6
SIL 10, 1, 7
MM 124, 1, v, 120, 184
RD 136, %2, 51200
SIL 10, 2, 0
MM 108, 1, v, 112, 184
RD 140, %2, 52224
SIL 10, 2, 1
MM 104, 1, v, 112, 176
LDX v, 84, 1024
MM 100, 1, v, 112, 168
RD 144, %2, 53248
SIL 10, 2, 2
SIL 10, 2, 3
MM 96, 1, v, 112, 160
RD 148, %2, 54272
SIL 10, 2, 4
MM 80, 1, v, 104, 160
RD 152, %2, 55296
STADR 10
MM 84, 1, v, 104, 168
LDX v, 88, 2048
MM 88, 1, v, 104, 176
RD 156, %2, 56320
SIL 10, 2, 5
SIL 10, 2, 6
MM 92, 1, v, 104, 184
SIL 10, 2, 7
MM 76, 1, v, 96, 184
DUP 10, 10
MM 72, 1, v, 96, 176
LDX v, 92, 3072
MM 68, 1, v, 96, 168
SIL 11, 0, 0
SIL 11, 0, 1
MM 64, 1, v, 96, 160
.Ljoin%=_5_1:
s_bitcmp1_b32 s46, 6
s_cbranch_scc1 .Lspec%=_5_2
s_waitcnt lgkmcnt(0)
MM 128, 1, v, 96, 128
RD 160, %2, 57344
SIL 11, 0, 2
SIL 11, 0, 3
MM 132, 1, v, 96, 136
STORE 10
MM 136, 1, v, 96, 144
RD 164, %2, 58368
SIL 11, 0, 4
SIL 11, 0, 5
MM 140, 1, v, 96, 152
SIL 11, 0, 6
MM 156, 1, v, 104, 152
RD 168, %2, 59392
SIL 11, 0, 7
MM 152, 1, v, 104, 144
RD 172, %2, 60416
SIL 11, 1, 0
MM 148, 1, v, 104, 136
RD 176, %2, 61440
SIL 11, 1, 1
MM 144, 1, v, 104, 128
RD 180, %2, 62464
SIL 11, 1, 2
MM 160, 1, v, 112, 128
RD 184, %2, 63488
SIL 11, 1, 3
MM 164, 1, v, 112, 136
RD 188, %2, 64512
SIL 11, 1, 4
MM 168, 1, v, 112, 144
SIL 11, 1, 5
MM 172, 1, v, 112, 152
CHKF 7
s_cbranch_scc0 .Lslow%=_G_5_2n
.Lback%=_G_5_2n:
SIL 11, 1, 6
MM 188, 1, v, 120, 152
s_add_u32 s40, s40, 2621440
SIL 11, 1, 7
MM 184, 1, v, 120, 144
s_add_u32 s49, s49, 10240
s_cmp_eq_u32 s49, 40960
s_cselect_b32 s49, 0, s49
SIL 11, 2, 0
MM 180, 1, v, 120, 136
v_mov_b32 v206, v207
buffer_load_dword v208, v206, %6, s49 offen sc0 sc1
MM 176, 1, v, 120, 128
.Ljoin%=_5_2:
s_bitcmp1_b32 s46, 7
s_cbranch_scc1 .Lspec%=_5_3
s_waitcnt lgkmcnt(0)
MM 240, 1, v, 120, 160
s_waitcnt vmcnt(11)
s_barrier
MM 244, 1, v, 120, 168
RD 128, v211, 0
RD 132, v211, 1024
SETW 98304, 42
MM 248, 1, v, 120, 176
LDW 0
MM 252, 1, v, 120, 184
LDW 1024
MM 236, 1, v, 112, 184
RD 136, v211, 2048
RD 140, v211, 3072
SETW 106496, 43
MM 232, 1, v, 112, 176
LDW 0
MM 228, 1, v, 112, 168
LDW 1024
MM 224, 1, v, 112, 160
RD 144, v211, 4096
RD 148, v211, 5120
SIL 11, 2, 1
MM 208, 1, v, 104, 160
RD 152, v211, 6144
SETW 114688, 44
SIL 11, 2, 2
MM 212, 1, v, 104, 168
LDW 0
MM 216, 1, v, 104, 176
LDW 1024
MM 220, 1, v, 104, 184
RD 156, v211, 7168
SETW 122880, 45
SIL 11, 2, 3
MM 204, 1, v, 96, 184
LDW 0
MM 200, 1, v, 96, 176
LDW 1024
MM 196, 1, v, 96, 168
SIL 11, 2, 4
STADR 11
SIL 11, 2, 5
SIL 11, 2, 6
SIL 11, 2, 7
MM 192, 1, v, 96, 160
.Ljoin%=_5_3:
s_bitcmp1_b32 s46, 4
s_cbranch_scc1 .Lspec%=_6_0
s_waitcnt vmcnt(17) lgkmcnt(0)
MM 0, 1, v, 64, 128
RD 160, v211, 8192
s_add_u32 s50, s40, s48
DUP 11, 11
SIL 12, 0, 0
MM 4, 1, v, 64, 136
LDX v, 96, 0
MM 8, 1, v, 64, 144
RD 164, v211, 9216
SIL 12, 0, 1
SIL 12, 0, 2
SIL 12, 0, 3
SIL 12, 0, 4
MM 12, 1, v, 64, 152
RD 168, v211, 10240
SIL 12, 0, 5
SIL 12, 0, 6
SIL 12, 0, 7
s_waitcnt vmcnt(15)
MM 28, 1, v, 72, 152
RD 172, v211, 11264
SIL 12, 1, 0
SIL 12, 1, 1
MM 24, 1, v, 72, 144
LDX v, 100, 1024
MM 20, 1, v, 72, 136
RD 176, v211, 12288
SIL 12, 1, 2
SIL 12, 1, 3
MM 16, 1, v, 72, 128
RD 180, v211, 13312
SIL 12, 1, 4
SIL 12, 1, 5
SIL 12, 1, 6
SIL 12, 1, 7
s_waitcnt vmcnt(14)
MM 32, 1, v, 80, 128
RD 184, v211, 14336
SIL 12, 2, 0
SIL 12, 2, 1
MM 36, 1, v, 80, 136
LDX v, 104, 2048
MM 40, 1, v, 80, 144
RD 188, v211, 15360
SIL 12, 2, 2
SIL 12, 2, 3
MM 44, 1, v, 80, 152
STORE 11
s_waitcnt vmcnt(14)
MM 60, 1, v, 88, 152
SIL 12, 2, 4
STADR 12
MM 56, 1, v, 88, 144
LDX v, 108, 3072
MM 52, 1, v, 88, 136
SIL 12, 2, 5
SIL 12, 2, 6
MM 48, 1, v, 88, 128
.Ljoin%=_6_0:
s_bitcmp1_b32 s46, 5
s_cbranch_scc1 .Lspec%=_6_1
s_waitcnt lgkmcnt(0)
MM 112, 1, v, 88, 160
RD 128, v211, 16384
s_add_u32 s50, s50, 4096
SIL 12, 2, 7
DUP 12, 12
MM 116, 1, v, 88, 168
LDX v, 112, 0
MM 120, 1, v, 88, 176
RD 132, v211, 17408
SIL 13, 0, 0
SIL 13, 0, 1
MM 124, 1, v, 88, 184
RD 136, v211, 18432
SIL 13, 0, 2
MM 108, 1, v, 80, 184
RD 140, v211, 19456
SIL 13, 0, 3
MM 104, 1, v, 80, 176
LDX v, 116, 1024
MM 100, 1, v, 80, 168
RD 144, v211, 20480
SIL 13, 0, 4
SIL 13, 0, 5
MM 96, 1, v, 80, 160
RD 148, v211, 21504
SIL 13, 0, 6
MM 80, 1, v, 72, 160
RD 152, v211, 22528
SIL 13, 0, 7
MM 84, 1, v, 72, 168
LDX v, 120, 2048
MM 88, 1, v, 72, 176
RD 156, v211, 23552
SIL 13, 1, 0
SIL 13, 1, 1
MM 92, 1, v, 72, 184
v_readlane_b32 s42, v205, 0
v_readlane_b32 s43, v205, 1
SIL 13, 1, 2
MM 76, 1, v, 64, 184
v_readlane_b32 s44, v205, 2
v_readlane_b32 s45, v205, 3
SIL 13, 1, 3
MM 72, 1, v, 64, 176
LDX v, 124, 3072
MM 68, 1, v, 64, 168
v_readlane_b32 s48, v205, 6
SIL 13, 1, 4
SIL 13, 1, 5
MM 64, 1, v, 64, 160
.Ljoin%=_6_1:
s_bitcmp1_b32 s46, 6
s_cbranch_scc1 .Lspec%=_6_2
s_waitcnt lgkmcnt(0)
MM 128, 1, v, 64, 128
RD 160, v211, 24576
SIL 13, 1, 6
SIL 13, 1, 7
MM 132, 1, v, 64, 136
STORE 12
MM 136, 1, v, 64, 144
RD 164, v211, 25600
SIL 13, 2, 0
SIL 13, 2, 1
MM 140, 1, v, 64, 152
SIL 13, 2, 2
MM 156, 1, v, 72, 152
RD 168, v211, 26624
SIL 13, 2, 3
MM 152, 1, v, 72, 144
RD 172, v211, 27648
SIL 13, 2, 4
MM 148, 1, v, 72, 136
RD 176, v211, 28672
STADR 13
MM 144, 1, v, 72, 128
RD 180, v211, 29696
SIL 13, 2, 5
MM 160, 1, v, 80, 128
RD 184, v211, 30720
SIL 13, 2, 6
MM 164, 1, v, 80, 136
RD 188, v211, 31744
SIL 13, 2, 7
MM 168, 1, v, 80, 144
s_waitcnt vmcnt(18)
v_cmp_le_u32 vcc, s53, v208
DUP 13, 13
MM 172, 1, v, 80, 152
CHKF 0
s_cbranch_scc0 .Lslow%=_G_6_2n
.Lback%=_G_6_2n:
SIL 14, 0, 0
MM 188, 1, v, 88, 152
s_add_u32 s40, s40, 2621440
s_cmp_eq_u32 s40, s60
s_cselect_b32 s40, %13, s40
SIL 14, 0, 1
MM 184, 1, v, 88, 144
STORE 13
MM 180, 1, v, 88, 136
SIL 14, 0, 2
SIL 14, 0, 3
MM 176, 1, v, 88, 128
.Ljoin%=_6_2:
s_bitcmp1_b32 s46, 7
s_cbranch_scc1 .Lspec%=_6_3
s_waitcnt lgkmcnt(0)
MM 240, 1, v, 88, 160
s_waitcnt vmcnt(11)
s_barrier
MM 244, 1, v, 88, 168
RD 128, v211, 32768
RD 132, v211, 33792
SETW 0, 42
SIL 14, 0, 4
MM 248, 1, v, 88, 176
LDW 0
MM 252, 1, v, 88, 184
LDW 1024
MM 236, 1, v, 80, 184
RD 136, v211, 34816
RD 140, v211, 35840
SETW 8192, 43
SIL 14, 0, 5
MM 232, 1, v, 80, 176
LDW 0
MM 228, 1, v, 80, 168
LDW 1024
MM 224, 1, v, 80, 160
RD 144, v211, 36864
RD 148, v211, 37888
SIL 14, 0, 6
SIL 14, 0, 7
MM 208, 1, v, 72, 160
RD 152, v211, 38912
SETW 16384, 44
SIL 14, 1, 0
MM 212, 1, v, 72, 168
LDW 0
MM 216, 1, v, 72, 176
LDW 1024
MM 220, 1, v, 72, 184
RD 156, v211, 39936
SETW 24576, 45
SIL 14, 1, 1
MM 204, 1, v, 64, 184
LDW 0
MM 200, 1, v, 64, 176
LDW 1024
MM 196, 1, v, 64, 168
SIL 14, 1, 2
SIL 14, 1, 3
SIL 14, 1, 4
MM 192, 1, v, 64, 160
.Ljoin%=_6_3:
s_bitcmp1_b32 s46, 4
s_cbranch_scc1 .Lspec%=_7_0
s_waitcnt vmcnt(17) lgkmcnt(0)
MM 0, 1, v, 96, 128
RD 160, v211, 40960
s_add_u32 s50, s40, s48
SIL 14, 1, 5
SIL 14, 1, 6
SIL 14, 1, 7
MM 4, 1, v, 96, 136
LDX v, 64, 0
MM 8, 1, v, 96, 144
RD 164, v211, 41984
SIL 14, 2, 0
SIL 14, 2, 1
MM 12, 1, v, 96, 152
RD 168, v211, 43008
SIL 14, 2, 2
SIL 14, 2, 3
s_waitcnt vmcnt(15)
MM 28, 1, v, 104, 152
RD 172, v211, 44032
SIL 14, 2, 4
STADR 14
SIL 14, 2, 5
SIL 14, 2, 6
MM 24, 1, v, 104, 144
LDX v, 68, 1024
MM 20, 1, v, 104, 136
RD 176, v211, 45056
SIL 14, 2, 7
DUP 14, 14
SIL 15, 0, 0
MM 16, 1, v, 104, 128
RD 180, v211, 46080
SIL 15, 0, 1
SIL 15, 0, 2
s_waitcnt vmcnt(14)
MM 32, 1, v, 112, 128
RD 184, v211, 47104
SIL 15, 0, 3
MM 36, 1, v, 112, 136
LDX v, 72, 2048
MM 40, 1, v, 112, 144
RD 188, v211, 48128
SIL 15, 0, 4
SIL 15, 0, 5
MM 44, 1, v, 112, 152
STORE 14
s_waitcnt vmcnt(14)
MM 60, 1, v, 120, 152
SIL 15, 0, 6
SIL 15, 0, 7
MM 56, 1, v, 120, 144
LDX v, 76, 3072
MM 52, 1, v, 120, 136
v_readlane_b32 s62, v205, 4
v_readlane_b32 s63, v205, 5
SIL 15, 1, 0
MM 48, 1, v, 120, 128
.Ljoin%=_7_0:
s_bitcmp1_b32 s46, 5
s_cbranch_scc1 .Lspec%=_7_1
s_waitcnt lgkmcnt(0)
MM 112, 1, v, 120, 160
RD 128, v211, 49152
s_add_u32 s50, s50, 4096
SIL 15, 1, 1
MM 116, 1, v, 120, 168
LDX v, 80, 0
MM 120, 1, v, 120, 176
RD 132, v211, 50176
SIL 15, 1, 2
SIL 15, 1, 3
MM 124, 1, v, 120, 184
RD 136, v211, 51200
SIL 15, 1, 4
SIL 15, 1, 5
SIL 15, 1, 6
MM 108, 1, v, 112, 184
RD 140, v211, 52224
SIL 15, 1, 7
MM 104, 1, v, 112, 176
LDX v, 84, 1024
MM 100, 1, v, 112, 168
RD 144, v211, 53248
SIL 15, 2, 0
SIL 15, 2, 1
MM 96, 1, v, 112, 160
RD 148, v211, 54272
SIL 15, 2, 2
MM 80, 1, v, 104, 160
RD 152, v211, 55296
SIL 15, 2, 3
MM 84, 1, v, 104, 168
LDX v, 88, 2048
MM 88, 1, v, 104, 176
RD 156, v211, 56320
SIL 15, 2, 4
STADR 15
MM 92, 1, v, 104, 184
SIL 15, 2, 5
MM 76, 1, v, 96, 184
SIL 15, 2, 6
MM 72, 1, v, 96, 176
LDX v, 92, 3072
MM 68, 1, v, 96, 168
v_readlane_b32 s72, v205, 7
SIL 15, 2, 7
DUP 15, 15
MM 64, 1, v, 96, 160
.Ljoin%=_7_1:
s_bitcmp1_b32 s46, 6
s_cbranch_scc1 .Lspec%=_7_2
s_waitcnt lgkmcnt(0)
MM 128, 1, v, 96, 128
RD 160, v211, 57344
MM 132, 1, v, 96, 136
s_bitcmp1_b32 s62, 4
s_cselect_b32 s42, 0x80000000, s42
s_bitcmp1_b32 s62, 5
s_cselect_b32 s43, 0x80000000, s43
MM 136, 1, v, 96, 144
RD 164, v211, 58368
MM 140, 1, v, 96, 152
s_bitcmp1_b32 s62, 6
s_cselect_b32 s44, 0x80000000, s44
s_bitcmp1_b32 s62, 7
s_cselect_b32 s45, 0x80000000, s45
MM 156, 1, v, 104, 152
RD 168, v211, 59392
MM 152, 1, v, 104, 144
RD 172, v211, 60416
MM 148, 1, v, 104, 136
RD 176, v211, 61440
MM 144, 1, v, 104, 128
RD 180, v211, 62464
MM 160, 1, v, 112, 128
RD 184, v211, 63488
MM 164, 1, v, 112, 136
RD 188, v211, 64512
MM 168, 1, v, 112, 144
STORE 15
MM 172, 1, v, 112, 152
CHKF 1
s_cbranch_scc0 .Lslow%=_G_7_2n
.Lback%=_G_7_2n:
MM 188, 1, v, 120, 152
s_add_u32 s40, s40, 2621440
MM 184, 1, v, 120, 144
MM 180, 1, v, 120, 136
buffer_load_dword v208, v206, %6, s49 offen sc0 sc1
MM 176, 1, v, 120, 128
.Ljoin%=_7_2:
s_bitcmp1_b32 s46, 7
s_cbranch_scc1 .Lspec%=_7_3
s_waitcnt lgkmcnt(0)
MM 240, 1, v, 120, 160
s_waitcnt vmcnt(11)
s_barrier
MM 244, 1, v, 120, 168
RD 128, %2, 0
RD 132, %2, 1024
SETW 32768, 42
MM 248, 1, v, 120, 176
LDW 0
MM 252, 1, v, 120, 184
LDW 1024
MM 236, 1, v, 112, 184
RD 136, %2, 2048
RD 140, %2, 3072
SETW 40960, 43
MM 232, 1, v, 112, 176
LDW 0
MM 228, 1, v, 112, 168
LDW 1024
MM 224, 1, v, 112, 160
RD 144, %2, 4096
RD 148, %2, 5120
MM 208, 1, v, 104, 160
RD 152, %2, 6144
SETW 49152, 44
MM 212, 1, v, 104, 168
LDW 0
MM 216, 1, v, 104, 176
LDW 1024
MM 220, 1, v, 104, 184
RD 156, %2, 7168
SETW 57344, 45
MM 204, 1, v, 96, 184
LDW 0
MM 200, 1, v, 96, 176
LDW 1024
s_mov_b64 s[46:47], s[62:63]
MM 196, 1, v, 96, 168
s_sub_u32 s54, s54, 1
MM 192, 1, v, 96, 160
.Ljoin%=_7_3:
s_cbranch_scc0 .Ltop%=
s_waitcnt vmcnt(17) lgkmcnt(0)
MM 0, 1, v, 64, 128
RD 160, %2, 8192
s_add_u32 s50, s40, s48
MM 4, 1, v, 64, 136
LDX v, 96, 0
MM 8, 1, v, 64, 144
RD 164, %2, 9216
MM 12, 1, v, 64, 152
RD 168, %2, 10240
LDX v, 100, 1024
s_waitcnt vmcnt(16)
MM 28, 1, v, 72, 152
RD 172, %2, 11264
v_accvgpr_read_b32 v0, a192
v_accvgpr_read_b32 v4, a196
v_accvgpr_read_b32 v8, a200
v_accvgpr_read_b32 v12, a204
MM 24, 1, v, 72, 144
LDX v, 104, 2048
MM 20, 1, v, 72, 136
RD 176, %2, 12288
SIL 0, 0, 0
SIL 0, 0, 1
SIL 0, 0, 2
SIL 0, 0, 3
MM 16, 1, v, 72, 128
RD 180, %2, 13312
LDX v, 108, 3072
s_waitcnt vmcnt(16)
MM 32, 1, v, 80, 128
RD 184, %2, 14336
s_add_u32 s50, s50, 4096
SIL 0, 0, 4
SIL 0, 0, 5
SIL 0, 0, 6
MM 36, 1, v, 80, 136
LDX v, 112, 0
MM 40, 1, v, 80, 144
RD 188, %2, 15360
SIL 0, 0, 7
SIL 0, 1, 0
MM 44, 1, v, 80, 152
LDX v, 116, 1024
s_waitcnt vmcnt(16)
MM 60, 1, v, 88, 152
SIL 0, 1, 1
SIL 0, 1, 2
MM 56, 1, v, 88, 144
LDX v, 120, 2048
MM 52, 1, v, 88, 136
SIL 0, 1, 3
SIL 0, 1, 4
SIL 0, 1, 5
SIL 0, 1, 6
MM 48, 1, v, 88, 128
LDX v, 124, 3072
s_waitcnt lgkmcnt(0)
MM 112, 1, v, 88, 160
RD 128, %2, 16384
SIL 0, 1, 7
SIL 0, 2, 0
MM 116, 1, v, 88, 168
SIL 0, 2, 1
SIL 0, 2, 2
MM 120, 1, v, 88, 176
RD 132, %2, 17408
s_waitcnt vmcnt(16)
v_cmp_le_u32 vcc, s53, v208
SIL 0, 2, 3
MM 124, 1, v, 88, 184
RD 136, %2, 18432
CHKF 2
s_cbranch_scc0 .Lslow%=_T3_0_2n
.Lback%=_T3_0_2n:
SIL 0, 2, 4
MM 108, 1, v, 80, 184
RD 140, %2, 19456
s_add_u32 s40, s40, 2621440
STADR 0
SIL 0, 2, 5
SIL 0, 2, 6
MM 104, 1, v, 80, 176
SIL 0, 2, 7
v_accvgpr_read_b32 v1, a193
MM 100, 1, v, 80, 168
RD 144, %2, 20480
STORE 0
MM 96, 1, v, 80, 160
RD 148, %2, 21504
v_accvgpr_read_b32 v5, a197
v_accvgpr_read_b32 v9, a201
MM 80, 1, v, 72, 160
RD 152, %2, 22528
SETW 65536, 42
v_accvgpr_read_b32 v13, a205
MM 84, 1, v, 72, 168
LDW 0
MM 88, 1, v, 72, 176
RD 156, %2, 23552
SIL 1, 0, 0
SIL 1, 0, 1
SIL 1, 0, 2
MM 92, 1, v, 72, 184
LDW 1024
MM 76, 1, v, 64, 184
SETW 73728, 43
SIL 1, 0, 3
SIL 1, 0, 4
MM 72, 1, v, 64, 176
LDW 0
MM 68, 1, v, 64, 168
SIL 1, 0, 5
SIL 1, 0, 6
SIL 1, 0, 7
MM 64, 1, v, 64, 160
LDW 1024
s_waitcnt lgkmcnt(0)
MM 128, 1, v, 64, 128
s_waitcnt vmcnt(15)
s_barrier
MM 132, 1, v, 64, 136
RD 160, %2, 32768
RD 164, %2, 33792
SETW 81920, 44
MM 136, 1, v, 64, 144
LDW 0
MM 140, 1, v, 64, 152
SIL 1, 1, 0
SIL 1, 1, 1
MM 156, 1, v, 72, 152
RD 168, %2, 34816
RD 172, %2, 35840
LDW 1024
MM 152, 1, v, 72, 144
SIL 1, 1, 2
SIL 1, 1, 3
SIL 1, 1, 4
MM 148, 1, v, 72, 136
SIL 1, 1, 5
SIL 1, 1, 6
SIL 1, 1, 7
SIL 1, 2, 0
MM 144, 1, v, 72, 128
RD 176, %2, 36864
RD 180, %2, 37888
SIL 1, 2, 1
MM 160, 1, v, 80, 128
RD 184, %2, 38912
SIL 1, 2, 2
SIL 1, 2, 3
MM 164, 1, v, 80, 136
SIL 1, 2, 4
STADR 1
MM 168, 1, v, 80, 144
SIL 1, 2, 5
MM 172, 1, v, 80, 152
RD 188, %2, 39936
SIL 1, 2, 6
SIL 1, 2, 7
MM 188, 1, v, 88, 152
STORE 1
MM 184, 1, v, 88, 144
v_accvgpr_read_b32 v2, a194
v_accvgpr_read_b32 v6, a198
MM 180, 1, v, 88, 136
v_accvgpr_read_b32 v10, a202
MM 176, 1, v, 88, 128
v_accvgpr_read_b32 v14, a206
SIL 2, 0, 0
s_waitcnt vmcnt(14) lgkmcnt(0)
MM 0, 1, v, 96, 160
RD 128, %2, 40960
s_add_u32 s50, s40, s48
SIL 2, 0, 1
MM 4, 1, v, 96, 168
LDX v, 64, 0
MM 8, 1, v, 96, 176
RD 132, %2, 41984
SIL 2, 0, 2
SIL 2, 0, 3
SIL 2, 0, 4
MM 12, 1, v, 96, 184
RD 136, %2, 43008
LDX v, 68, 1024
s_waitcnt vmcnt(14)
MM 28, 1, v, 104, 184
RD 140, %2, 44032
SIL 2, 0, 5
SIL 2, 0, 6
MM 24, 1, v, 104, 176
LDX v, 72, 2048
MM 20, 1, v, 104, 168
RD 144, %2, 45056
SIL 2, 0, 7
SIL 2, 1, 0
MM 16, 1, v, 104, 160
RD 148, %2, 46080
LDX v, 76, 3072
s_waitcnt vmcnt(14)
MM 32, 1, v, 112, 160
RD 152, %2, 47104
s_add_u32 s50, s50, 4096
SIL 2, 1, 1
MM 36, 1, v, 112, 168
LDX v, 80, 0
MM 40, 1, v, 112, 176
RD 156, %2, 48128
SIL 2, 1, 2
SIL 2, 1, 3
MM 44, 1, v, 112, 184
LDX v, 84, 1024
s_waitcnt vmcnt(14)
MM 60, 1, v, 120, 184
SIL 2, 1, 4
SIL 2, 1, 5
SIL 2, 1, 6
SIL 2, 1, 7
MM 56, 1, v, 120, 176
LDX v, 88, 2048
MM 52, 1, v, 120, 168
SIL 2, 2, 0
SIL 2, 2, 1
MM 48, 1, v, 120, 160
LDX v, 92, 3072
s_waitcnt lgkmcnt(0)
MM 112, 1, v, 120, 128
RD 160, %2, 49152
SIL 2, 2, 2
SIL 2, 2, 3
MM 116, 1, v, 120, 136
SIL 2, 2, 4
STADR 2
SIL 2, 2, 5
SIL 2, 2, 6
MM 120, 1, v, 120, 144
RD 164, %2, 50176
SIL 2, 2, 7
MM 124, 1, v, 120, 152
RD 168, %2, 51200
CHKF 3
s_cbranch_scc0 .Lslow%=_T3_1_2n
.Lback%=_T3_1_2n:
v_accvgpr_read_b32 v3, a195
MM 108, 1, v, 112, 152
RD 172, %2, 52224
s_add_u32 s40, s40, 2621440
v_accvgpr_read_b32 v7, a199
MM 104, 1, v, 112, 144
STORE 2
MM 100, 1, v, 112, 136
RD 176, %2, 53248
buffer_load_dword v208, v206, %6, s49 offen sc0 sc1
MM 96, 1, v, 112, 128
RD 180, %2, 54272
v_accvgpr_read_b32 v11, a203
v_accvgpr_read_b32 v15, a207
SIL 3, 0, 0
SIL 3, 0, 1
MM 80, 1, v, 104, 128
RD 184, %2, 55296
SETW 98304, 42
SIL 3, 0, 2
SIL 3, 0, 3
MM 84, 1, v, 104, 136
LDW 0
MM 88, 1, v, 104, 144
RD 188, %2, 56320
SIL 3, 0, 4
SIL 3, 0, 5
MM 92, 1, v, 104, 152
LDW 1024
MM 76, 1, v, 96, 152
SETW 106496, 43
SIL 3, 0, 6
SIL 3, 0, 7
MM 72, 1, v, 96, 144
LDW 0
MM 68, 1, v, 96, 136
SIL 3, 1, 0
SIL 3, 1, 1
MM 64, 1, v, 96, 128
LDW 1024
s_waitcnt lgkmcnt(0)
MM 128, 1, v, 96, 160
s_waitcnt vmcnt(15)
s_barrier
MM 132, 1, v, 96, 168
RD 128, v211, 0
RD 132, v211, 1024
SETW 114688, 44
MM 136, 1, v, 96, 176
LDW 0
MM 140, 1, v, 96, 184
SIL 3, 1, 2
SIL 3, 1, 3
SIL 3, 1, 4
MM 156, 1, v, 104, 184
RD 136, v211, 2048
RD 140, v211, 3072
LDW 1024
MM 152, 1, v, 104, 176
SIL 3, 1, 5
SIL 3, 1, 6
SIL 3, 1, 7
SIL 3, 2, 0
MM 148, 1, v, 104, 168
SIL 3, 2, 1
SIL 3, 2, 2
MM 144, 1, v, 104, 160
RD 144, v211, 4096
RD 148, v211, 5120
SIL 3, 2, 3
SIL 3, 2, 4
MM 160, 1, v, 112, 160
RD 152, v211, 6144
STADR 3
SIL 3, 2, 5
SIL 3, 2, 6
MM 164, 1, v, 112, 168
SIL 3, 2, 7
MM 168, 1, v, 112, 176
STORE 3
MM 172, 1, v, 112, 184
RD 156, v211, 7168
v_accvgpr_read_b32 v16, a208
v_accvgpr_read_b32 v20, a212
v_accvgpr_read_b32 v24, a216
MM 188, 1, v, 120, 184
v_accvgpr_read_b32 v28, a220
MM 184, 1, v, 120, 176
SIL 4, 0, 0
MM 180, 1, v, 120, 168
SIL 4, 0, 1
SIL 4, 0, 2
MM 176, 1, v, 120, 160
SIL 4, 0, 3
s_waitcnt vmcnt(15) lgkmcnt(0)
MM 0, 1, v, 64, 128
RD 160, v211, 8192
s_add_u32 s50, s40, s48
SIL 4, 0, 4
MM 4, 1, v, 64, 136
LDX v, 96, 0
MM 8, 1, v, 64, 144
RD 164, v211, 9216
SIL 4, 0, 5
SIL 4, 0, 6
SIL 4, 0, 7
MM 12, 1, v, 64, 152
RD 168, v211, 10240
LDX v, 100, 1024
s_waitcnt vmcnt(15)
MM 28, 1, v, 72, 152
RD 172, v211, 11264
SIL 4, 1, 0
SIL 4, 1, 1
MM 24, 1, v, 72, 144
LDX v, 104, 2048
MM 20, 1, v, 72, 136
RD 176, v211, 12288
SIL 4, 1, 2
SIL 4, 1, 3
MM 16, 1, v, 72, 128
RD 180, v211, 13312
LDX v, 108, 3072
s_waitcnt vmcnt(15)
MM 32, 1, v, 80, 128
RD 184, v211, 14336
s_add_u32 s50, s50, 4096
SIL 4, 1, 4
SIL 4, 1, 5
SIL 4, 1, 6
MM 36, 1, v, 80, 136
LDX v, 112, 0
MM 40, 1, v, 80, 144
RD 188, v211, 15360
SIL 4, 1, 7
SIL 4, 2, 0
MM 44, 1, v, 80, 152
LDX v, 116, 1024
s_waitcnt vmcnt(15)
MM 60, 1, v, 88, 152
SIL 4, 2, 1
SIL 4, 2, 2
MM 56, 1, v, 88, 144
LDX v, 120, 2048
MM 52, 1, v, 88, 136
v_add_u32 v210, 64, v210
SIL 4, 2, 3
SIL 4, 2, 4
STADR 4
MM 48, 1, v, 88, 128
LDX v, 124, 3072
s_waitcnt lgkmcnt(0)
MM 112, 1, v, 88, 160
RD 128, v211, 16384
SIL 4, 2, 5
SIL 4, 2, 6
SIL 4, 2, 7
v_accvgpr_read_b32 v17, a209
MM 116, 1, v, 88, 168
STORE 4
MM 120, 1, v, 88, 176
RD 132, v211, 17408
s_waitcnt vmcnt(16)
v_cmp_le_u32 vcc, s53, v208
v_accvgpr_read_b32 v21, a213
v_accvgpr_read_b32 v25, a217
MM 124, 1, v, 88, 184
RD 136, v211, 18432
CHKF 4
s_cbranch_scc0 .Lslow%=_T3_2_2n
.Lback%=_T3_2_2n:
v_accvgpr_read_b32 v29, a221
MM 108, 1, v, 80, 184
RD 140, v211, 19456
s_add_u32 s40, s40, 2621440
SIL 5, 0, 0
SIL 5, 0, 1
MM 104, 1, v, 80, 176
SIL 5, 0, 2
MM 100, 1, v, 80, 168
RD 144, v211, 20480
SIL 5, 0, 3
MM 96, 1, v, 80, 160
RD 148, v211, 21504
SIL 5, 0, 4
MM 80, 1, v, 72, 160
RD 152, v211, 22528
SETW 0, 42
SIL 5, 0, 5
SIL 5, 0, 6
MM 84, 1, v, 72, 168
LDW 0
MM 88, 1, v, 72, 176
RD 156, v211, 23552
SIL 5, 0, 7
SIL 5, 1, 0
MM 92, 1, v, 72, 184
LDW 1024
MM 76, 1, v, 64, 184
SETW 8192, 43
SIL 5, 1, 1
MM 72, 1, v, 64, 176
LDW 0
MM 68, 1, v, 64, 168
SIL 5, 1, 2
SIL 5, 1, 3
SIL 5, 1, 4
MM 64, 1, v, 64, 160
LDW 1024
s_waitcnt lgkmcnt(0)
MM 128, 1, v, 64, 128
s_waitcnt vmcnt(14)
s_barrier
MM 132, 1, v, 64, 136
RD 160, v211, 32768
RD 164, v211, 33792
SETW 16384, 44
SIL 5, 1, 5
MM 136, 1, v, 64, 144
LDW 0
MM 140, 1, v, 64, 152
SIL 5, 1, 6
SIL 5, 1, 7
SIL 5, 2, 0
MM 156, 1, v, 72, 152
RD 168, v211, 34816
RD 172, v211, 35840
LDW 1024
MM 152, 1, v, 72, 144
SIL 5, 2, 1
SIL 5, 2, 2
MM 148, 1, v, 72, 136
SIL 5, 2, 3
SIL 5, 2, 4
STADR 5
SIL 5, 2, 5
MM 144, 1, v, 72, 128
RD 176, v211, 36864
RD 180, v211, 37888
SIL 5, 2, 6
SIL 5, 2, 7
v_accvgpr_read_b32 v18, a210
MM 160, 1, v, 80, 128
RD 184, v211, 38912
STORE 5
MM 164, 1, v, 80, 136
v_accvgpr_read_b32 v22, a214
v_accvgpr_read_b32 v26, a218
MM 168, 1, v, 80, 144
v_accvgpr_read_b32 v30, a222
SIL 6, 0, 0
MM 172, 1, v, 80, 152
RD 188, v211, 39936
SIL 6, 0, 1
MM 188, 1, v, 88, 152
SIL 6, 0, 2
MM 184, 1, v, 88, 144
SIL 6, 0, 3
SIL 6, 0, 4
MM 180, 1, v, 88, 136
SIL 6, 0, 5
MM 176, 1, v, 88, 128
SIL 6, 0, 6
s_waitcnt vmcnt(14) lgkmcnt(0)
MM 0, 1, v, 96, 160
RD 128, v211, 40960
s_add_u32 s50, s40, s48
SIL 6, 0, 7
MM 4, 1, v, 96, 168
LDX v, 64, 0
MM 8, 1, v, 96, 176
RD 132, v211, 41984
SIL 6, 1, 0
SIL 6, 1, 1
MM 12, 1, v, 96, 184
RD 136, v211, 43008
LDX v, 68, 1024
s_waitcnt vmcnt(14)
MM 28, 1, v, 104, 184
RD 140, v211, 44032
SIL 6, 1, 2
SIL 6, 1, 3
MM 24, 1, v, 104, 176
LDX v, 72, 2048
MM 20, 1, v, 104, 168
RD 144, v211, 45056
SIL 6, 1, 4
SIL 6, 1, 5
SIL 6, 1, 6
SIL 6, 1, 7
MM 16, 1, v, 104, 160
RD 148, v211, 46080
LDX v, 76, 3072
s_waitcnt vmcnt(14)
MM 32, 1, v, 112, 160
RD 152, v211, 47104
s_add_u32 s50, s50, 4096
SIL 6, 2, 0
MM 36, 1, v, 112, 168
LDX v, 80, 0
MM 40, 1, v, 112, 176
RD 156, v211, 48128
SIL 6, 2, 1
SIL 6, 2, 2
MM 44, 1, v, 112, 184
LDX v, 84, 1024
s_waitcnt vmcnt(14)
MM 60, 1, v, 120, 184
SIL 6, 2, 3
SIL 6, 2, 4
STADR 6
SIL 6, 2, 5
MM 56, 1, v, 120, 176
LDX v, 88, 2048
MM 52, 1, v, 120, 168
ds_read_b32 v204, v210
ds_read_b32 v205, v210 offset:32
SIL 6, 2, 6
SIL 6, 2, 7
v_accvgpr_read_b32 v19, a211
MM 48, 1, v, 120, 160
LDX v, 92, 3072
s_waitcnt lgkmcnt(2)
MM 112, 1, v, 120, 128
RD 160, v211, 49152
STORE 6
MM 116, 1, v, 120, 136
v_accvgpr_read_b32 v23, a215
v_accvgpr_read_b32 v27, a219
v_accvgpr_read_b32 v31, a223
SIL 7, 0, 0
MM 120, 1, v, 120, 144
RD 164, v211, 50176
SIL 7, 0, 1
MM 124, 1, v, 120, 152
RD 168, v211, 51200
CHKF 5
s_cbranch_scc0 .Lslow%=_T3_3_2n
.Lback%=_T3_3_2n:
SIL 7, 0, 2
MM 108, 1, v, 112, 152
RD 172, v211, 52224
s_add_u32 s40, s40, 2621440
SIL 7, 0, 3
SIL 7, 0, 4
MM 104, 1, v, 112, 144
SIL 7, 0, 5
MM 100, 1, v, 112, 136
RD 176, v211, 53248
SIL 7, 0, 6
MM 96, 1, v, 112, 128
RD 180, v211, 54272
SIL 7, 0, 7
SIL 7, 1, 0
MM 80, 1, v, 104, 128
RD 184, v211, 55296
SETW 32768, 42
SIL 7, 1, 1
MM 84, 1, v, 104, 136
LDW 0
MM 88, 1, v, 104, 144
RD 188, v211, 56320
SIL 7, 1, 2
SIL 7, 1, 3
MM 92, 1, v, 104, 152
LDW 1024
MM 76, 1, v, 96, 152
SETW 40960, 43
SIL 7, 1, 4
SIL 7, 1, 5
SIL 7, 1, 6
MM 72, 1, v, 96, 144
LDW 0
MM 68, 1, v, 96, 136
SIL 7, 1, 7
SIL 7, 2, 0
SIL 7, 2, 1
MM 64, 1, v, 96, 128
LDW 1024
s_waitcnt lgkmcnt(0)
MM 128, 1, v, 96, 160
s_waitcnt vmcnt(14)
s_barrier
MM 132, 1, v, 96, 168
RD 128, %2, 0
RD 132, %2, 1024
SETW 49152, 44
MM 136, 1, v, 96, 176
LDW 0
MM 140, 1, v, 96, 184
SIL 7, 2, 2
SIL 7, 2, 3
SIL 7, 2, 4
MM 156, 1, v, 104, 184
RD 136, %2, 2048
RD 140, %2, 3072
LDW 1024
MM 152, 1, v, 104, 176
STADR 7
SIL 7, 2, 5
SIL 7, 2, 6
SIL 7, 2, 7
v_accvgpr_read_b32 v32, a224
MM 148, 1, v, 104, 168
STORE 7
MM 144, 1, v, 104, 160
RD 144, %2, 4096
RD 148, %2, 5120
v_accvgpr_read_b32 v36, a228
v_accvgpr_read_b32 v40, a232
v_accvgpr_read_b32 v44, a236
MM 160, 1, v, 112, 160
RD 152, %2, 6144
SIL 8, 0, 0
MM 164, 1, v, 112, 168
SIL 8, 0, 1
SIL 8, 0, 2
MM 168, 1, v, 112, 176
SIL 8, 0, 3
MM 172, 1, v, 112, 184
RD 156, %2, 7168
SIL 8, 0, 4
MM 188, 1, v, 120, 184
SIL 8, 0, 5
MM 184, 1, v, 120, 176
SIL 8, 0, 6
SIL 8, 0, 7
MM 180, 1, v, 120, 168
SIL 8, 1, 0
MM 176, 1, v, 120, 160
SIL 8, 1, 1
s_waitcnt vmcnt(14) lgkmcnt(0)
MM 0, 1, v, 64, 128
RD 160, %2, 8192
s_add_u32 s50, s40, s48
SIL 8, 1, 2
MM 4, 1, v, 64, 136
LDX v, 96, 0
MM 8, 1, v, 64, 144
RD 164, %2, 9216
SIL 8, 1, 3
SIL 8, 1, 4
SIL 8, 1, 5
MM 12, 1, v, 64, 152
RD 168, %2, 10240
LDX v, 100, 1024
s_waitcnt vmcnt(14)
MM 28, 1, v, 72, 152
RD 172, %2, 11264
SIL 8, 1, 6
SIL 8, 1, 7
SIL 8, 2, 0
MM 24, 1, v, 72, 144
LDX v, 104, 2048
MM 20, 1, v, 72, 136
RD 176, %2, 12288
SIL 8, 2, 1
SIL 8, 2, 2
MM 16, 1, v, 72, 128
RD 180, %2, 13312
LDX v, 108, 3072
s_waitcnt vmcnt(14)
MM 32, 1, v, 80, 128
RD 184, %2, 14336
s_add_u32 s50, s50, 4096
SIL 8, 2, 3
SIL 8, 2, 4
MM 36, 1, v, 80, 136
LDX v, 112, 0
MM 40, 1, v, 80, 144
RD 188, %2, 15360
STADR 8
SIL 8, 2, 5
SIL 8, 2, 6
MM 44, 1, v, 80, 152
LDX v, 116, 1024
s_waitcnt vmcnt(14)
MM 60, 1, v, 88, 152
SIL 8, 2, 7
v_accvgpr_read_b32 v33, a225
v_accvgpr_read_b32 v37, a229
MM 56, 1, v, 88, 144
LDX v, 120, 2048
MM 52, 1, v, 88, 136
v_add_u32 v207, v204, %5
STORE 8
MM 48, 1, v, 88, 128
LDX v, 124, 3072
s_waitcnt lgkmcnt(0)
MM 112, 1, v, 88, 160
RD 128, %2, 16384
v_accvgpr_read_b32 v41, a233
v_accvgpr_read_b32 v45, a237
SIL 9, 0, 0
SIL 9, 0, 1
MM 116, 1, v, 88, 168
SIL 9, 0, 2
SIL 9, 0, 3
MM 120, 1, v, 88, 176
RD 132, %2, 17408
SIL 9, 0, 4
MM 124, 1, v, 88, 184
RD 136, %2, 18432
CHKF 6
s_cbranch_scc0 .Lslow%=_T3_4_2n
.Lback%=_T3_4_2n:
SIL 9, 0, 5
MM 108, 1, v, 80, 184
RD 140, %2, 19456
s_add_u32 s40, s40, 2621440
SIL 9, 0, 6
SIL 9, 0, 7
MM 104, 1, v, 80, 176
SIL 9, 1, 0
MM 100, 1, v, 80, 168
RD 144, %2, 20480
SIL 9, 1, 1
SIL 9, 1, 2
MM 96, 1, v, 80, 160
RD 148, %2, 21504
SIL 9, 1, 3
MM 80, 1, v, 72, 160
RD 152, %2, 22528
SETW 65536, 42
SIL 9, 1, 4
MM 84, 1, v, 72, 168
LDW 0
MM 88, 1, v, 72, 176
RD 156, %2, 23552
SIL 9, 1, 5
SIL 9, 1, 6
SIL 9, 1, 7
MM 92, 1, v, 72, 184
LDW 1024
MM 76, 1, v, 64, 184
SETW 73728, 43
SIL 9, 2, 0
MM 72, 1, v, 64, 176
LDW 0
MM 68, 1, v, 64, 168
SIL 9, 2, 1
SIL 9, 2, 2
MM 64, 1, v, 64, 160
LDW 1024
s_waitcnt lgkmcnt(0)
MM 128, 1, v, 64, 128
s_waitcnt vmcnt(14)
s_barrier
MM 132, 1, v, 64, 136
RD 160, %2, 32768
RD 164, %2, 33792
SETW 81920, 44
MM 136, 1, v, 64, 144
LDW 0
MM 140, 1, v, 64, 152
SIL 9, 2, 3
SIL 9, 2, 4
STADR 9
SIL 9, 2, 5
MM 156, 1, v, 72, 152
RD 168, %2, 34816
RD 172, %2, 35840
LDW 1024
MM 152, 1, v, 72, 144
SIL 9, 2, 6
SIL 9, 2, 7
v_accvgpr_read_b32 v34, a226
v_accvgpr_read_b32 v38, a230
v_accvgpr_read_b32 v42, a234
MM 148, 1, v, 72, 136
STORE 9
MM 144, 1, v, 72, 128
RD 176, %2, 36864
RD 180, %2, 37888
v_accvgpr_read_b32 v46, a238
SIL 10, 0, 0
SIL 10, 0, 1
MM 160, 1, v, 80, 128
RD 184, %2, 38912
SIL 10, 0, 2
SIL 10, 0, 3
SIL 10, 0, 4
MM 164, 1, v, 80, 136
SIL 10, 0, 5
MM 168, 1, v, 80, 144
SIL 10, 0, 6
MM 172, 1, v, 80, 152
RD 188, %2, 39936
SIL 10, 0, 7
MM 188, 1, v, 88, 152
SIL 10, 1, 0
SIL 10, 1, 1
MM 184, 1, v, 88, 144
SIL 10, 1, 2
MM 180, 1, v, 88, 136
SIL 10, 1, 3
MM 176, 1, v, 88, 128
SIL 10, 1, 4
s_waitcnt vmcnt(14) lgkmcnt(0)
MM 0, 1, v, 96, 160
RD 128, %2, 40960
s_add_u32 s50, s40, s48
SIL 10, 1, 5
SIL 10, 1, 6
MM 4, 1, v, 96, 168
LDX v, 64, 0
MM 8, 1, v, 96, 176
RD 132, %2, 41984
SIL 10, 1, 7
SIL 10, 2, 0
MM 12, 1, v, 96, 184
RD 136, %2, 43008
LDX v, 68, 1024
s_waitcnt vmcnt(14)
MM 28, 1, v, 104, 184
RD 140, %2, 44032
SIL 10, 2, 1
SIL 10, 2, 2
MM 24, 1, v, 104, 176
LDX v, 72, 2048
MM 20, 1, v, 104, 168
RD 144, %2, 45056
SIL 10, 2, 3
SIL 10, 2, 4
STADR 10
MM 16, 1, v, 104, 160
RD 148, %2, 46080
LDX v, 76, 3072
s_waitcnt vmcnt(14)
MM 32, 1, v, 112, 160
RD 152, %2, 47104
s_add_u32 s50, s50, 4096
SIL 10, 2, 5
SIL 10, 2, 6
SIL 10, 2, 7
MM 36, 1, v, 112, 168
LDX v, 80, 0
MM 40, 1, v, 112, 176
RD 156, %2, 48128
STORE 10
MM 44, 1, v, 112, 184
LDX v, 84, 1024
s_waitcnt vmcnt(14)
MM 60, 1, v, 120, 184
v_accvgpr_read_b32 v35, a227
v_accvgpr_read_b32 v39, a231
v_accvgpr_read_b32 v43, a235
v_accvgpr_read_b32 v47, a239
SIL 11, 0, 0
MM 56, 1, v, 120, 176
LDX v, 88, 2048
MM 52, 1, v, 120, 168
SIL 11, 0, 1
SIL 11, 0, 2
SIL 11, 0, 3
MM 48, 1, v, 120, 160
LDX v, 92, 3072
s_waitcnt lgkmcnt(0)
MM 112, 1, v, 120, 128
RD 160, %2, 49152
SIL 11, 0, 4
SIL 11, 0, 5
MM 116, 1, v, 120, 136
SIL 11, 0, 6
MM 120, 1, v, 120, 144
RD 164, %2, 50176
SIL 11, 0, 7
SIL 11, 1, 0
MM 124, 1, v, 120, 152
RD 168, %2, 51200
CHKF 7
s_cbranch_scc0 .Lslow%=_T3_5_2n
.Lback%=_T3_5_2n:
MM 108, 1, v, 112, 152
RD 172, %2, 52224
s_add_u32 s40, s40, 2621440
SIL 11, 1, 1
MM 104, 1, v, 112, 144
s_add_u32 s49, s49, 10240
s_cmp_eq_u32 s49, 40960
s_cselect_b32 s49, 0, s49
SIL 11, 1, 2
MM 100, 1, v, 112, 136
RD 176, %2, 53248
v_mov_b32 v206, v207
buffer_load_dword v208, v206, %6, s49 offen sc0 sc1
MM 96, 1, v, 112, 128
RD 180, %2, 54272
SIL 11, 1, 3
SIL 11, 1, 4
SIL 11, 1, 5
MM 80, 1, v, 104, 128
RD 184, %2, 55296
SETW 98304, 42
SIL 11, 1, 6
SIL 11, 1, 7
MM 84, 1, v, 104, 136
LDW 0
MM 88, 1, v, 104, 144
RD 188, %2, 56320
SIL 11, 2, 0
SIL 11, 2, 1
MM 92, 1, v, 104, 152
LDW 1024
MM 76, 1, v, 96, 152
SETW 106496, 43
SIL 11, 2, 2
MM 72, 1, v, 96, 144
LDW 0
MM 68, 1, v, 96, 136
SIL 11, 2, 3
SIL 11, 2, 4
STADR 11
SIL 11, 2, 5
MM 64, 1, v, 96, 128
LDW 1024
s_waitcnt lgkmcnt(0)
MM 128, 1, v, 96, 160
s_waitcnt vmcnt(15)
s_barrier
MM 132, 1, v, 96, 168
RD 128, v211, 0
RD 132, v211, 1024
SETW 114688, 44
SIL 11, 2, 6
MM 136, 1, v, 96, 176
LDW 0
MM 140, 1, v, 96, 184
SIL 11, 2, 7
v_accvgpr_read_b32 v48, a240
v_accvgpr_read_b32 v52, a244
v_accvgpr_read_b32 v56, a248
v_accvgpr_read_b32 v60, a252
MM 156, 1, v, 104, 184
RD 136, v211, 2048
RD 140, v211, 3072
LDW 1024
MM 152, 1, v, 104, 176
STORE 11
MM 148, 1, v, 104, 168
SIL 12, 0, 0
SIL 12, 0, 1
SIL 12, 0, 2
SIL 12, 0, 3
SIL 12, 0, 4
MM 144, 1, v, 104, 160
RD 144, v211, 4096
RD 148, v211, 5120
SIL 12, 0, 5
MM 160, 1, v, 112, 160
RD 152, v211, 6144
SIL 12, 0, 6
SIL 12, 0, 7
MM 164, 1, v, 112, 168
SIL 12, 1, 0
MM 168, 1, v, 112, 176
SIL 12, 1, 1
MM 172, 1, v, 112, 184
RD 156, v211, 7168
SIL 12, 1, 2
SIL 12, 1, 3
MM 188, 1, v, 120, 184
SIL 12, 1, 4
MM 184, 1, v, 120, 176
SIL 12, 1, 5
MM 180, 1, v, 120, 168
SIL 12, 1, 6
MM 176, 1, v, 120, 160
SIL 12, 1, 7
SIL 12, 2, 0
s_waitcnt vmcnt(15) lgkmcnt(0)
MM 0, 1, v, 64, 128
RD 160, v211, 8192
s_add_u32 s50, s40, s48
SIL 12, 2, 1
MM 4, 1, v, 64, 136
LDX v, 96, 0
MM 8, 1, v, 64, 144
RD 164, v211, 9216
SIL 12, 2, 2
SIL 12, 2, 3
MM 12, 1, v, 64, 152
RD 168, v211, 10240
LDX v, 100, 1024
s_waitcnt vmcnt(15)
MM 28, 1, v, 72, 152
RD 172, v211, 11264
SIL 12, 2, 4
STADR 12
SIL 12, 2, 5
MM 24, 1, v, 72, 144
LDX v, 104, 2048
MM 20, 1, v, 72, 136
RD 176, v211, 12288
SIL 12, 2, 6
SIL 12, 2, 7
v_accvgpr_read_b32 v49, a241
MM 16, 1, v, 72, 128
RD 180, v211, 13312
LDX v, 108, 3072
s_waitcnt vmcnt(14)
MM 32, 1, v, 80, 128
RD 184, v211, 14336
s_add_u32 s50, s50, 4096
v_accvgpr_read_b32 v53, a245
v_accvgpr_read_b32 v57, a249
MM 36, 1, v, 80, 136
LDX v, 112, 0
MM 40, 1, v, 80, 144
RD 188, v211, 15360
STORE 12
MM 44, 1, v, 80, 152
LDX v, 116, 1024
s_waitcnt vmcnt(15)
MM 60, 1, v, 88, 152
v_accvgpr_read_b32 v61, a253
SIL 13, 0, 0
SIL 13, 0, 1
SIL 13, 0, 2
SIL 13, 0, 3
MM 56, 1, v, 88, 144
LDX v, 120, 2048
MM 52, 1, v, 88, 136
SIL 13, 0, 4
SIL 13, 0, 5
SIL 13, 0, 6
MM 48, 1, v, 88, 128
LDX v, 124, 3072
s_waitcnt lgkmcnt(0)
MM 112, 1, v, 88, 160
RD 128, v211, 16384
SIL 13, 0, 7
SIL 13, 1, 0
MM 116, 1, v, 88, 168
SIL 13, 1, 1
SIL 13, 1, 2
MM 120, 1, v, 88, 176
RD 132, v211, 17408
s_waitcnt vmcnt(16)
v_cmp_le_u32 vcc, s53, v208
SIL 13, 1, 3
MM 124, 1, v, 88, 184
RD 136, v211, 18432
CHKF 0
s_cbranch_scc0 .Lslow%=_T3_6_2n
.Lback%=_T3_6_2n:
SIL 13, 1, 4
MM 108, 1, v, 80, 184
RD 140, v211, 19456
s_add_u32 s40, s40, 2621440
s_cmp_eq_u32 s40, s60
s_cselect_b32 s40, %13, s40
SIL 13, 1, 5
MM 104, 1, v, 80, 176
SIL 13, 1, 6
SIL 13, 1, 7
MM 100, 1, v, 80, 168
RD 144, v211, 20480
SIL 13, 2, 0
MM 96, 1, v, 80, 160
RD 148, v211, 21504
SIL 13, 2, 1
MM 80, 1, v, 72, 160
RD 152, v211, 22528
SETW 0, 42
SIL 13, 2, 2
MM 84, 1, v, 72, 168
LDW 0
MM 88, 1, v, 72, 176
RD 156, v211, 23552
SIL 13, 2, 3
SIL 13, 2, 4
STADR 13
MM 92, 1, v, 72, 184
LDW 1024
MM 76, 1, v, 64, 184
SETW 8192, 43
SIL 13, 2, 5
SIL 13, 2, 6
SIL 13, 2, 7
MM 72, 1, v, 64, 176
LDW 0
MM 68, 1, v, 64, 168
STORE 13
MM 64, 1, v, 64, 160
LDW 1024
s_waitcnt lgkmcnt(0)
MM 128, 1, v, 64, 128
s_waitcnt vmcnt(15)
s_barrier
MM 132, 1, v, 64, 136
RD 160, v211, 32768
RD 164, v211, 33792
v_accvgpr_read_b32 v50, a242
v_accvgpr_read_b32 v54, a246
v_accvgpr_read_b32 v58, a250
MM 136, 1, v, 64, 144
v_accvgpr_read_b32 v62, a254
SIL 14, 0, 0
SIL 14, 0, 1
SIL 14, 0, 2
MM 140, 1, v, 64, 152
SIL 14, 0, 3
SIL 14, 0, 4
MM 156, 1, v, 72, 152
RD 168, v211, 34816
RD 172, v211, 35840
SIL 14, 0, 5
MM 152, 1, v, 72, 144
SIL 14, 0, 6
MM 148, 1, v, 72, 136
SIL 14, 0, 7
MM 144, 1, v, 72, 128
RD 176, v211, 36864
RD 180, v211, 37888
SIL 14, 1, 0
MM 160, 1, v, 80, 128
RD 184, v211, 38912
SIL 14, 1, 1
SIL 14, 1, 2
MM 164, 1, v, 80, 136
SIL 14, 1, 3
MM 168, 1, v, 80, 144
SIL 14, 1, 4
MM 172, 1, v, 80, 152
RD 188, v211, 39936
SIL 14, 1, 5
SIL 14, 1, 6
MM 188, 1, v, 88, 152
SIL 14, 1, 7
MM 184, 1, v, 88, 144
SIL 14, 2, 0
MM 180, 1, v, 88, 136
SIL 14, 2, 1
SIL 14, 2, 2
MM 176, 1, v, 88, 128
SIL 14, 2, 3
s_waitcnt vmcnt(12) lgkmcnt(0)
MM 0, 1, v, 96, 160
RD 128, v211, 40960
s_add_u32 s50, s40, s48
SIL 14, 2, 4
MM 4, 1, v, 96, 168
LDX v, 64, 0
MM 8, 1, v, 96, 176
RD 132, v211, 41984
STADR 14
SIL 14, 2, 5
SIL 14, 2, 6
MM 12, 1, v, 96, 184
RD 136, v211, 43008
LDX v, 68, 1024
s_waitcnt vmcnt(12)
MM 28, 1, v, 104, 184
RD 140, v211, 44032
SIL 14, 2, 7
v_accvgpr_read_b32 v51, a243
MM 24, 1, v, 104, 176
LDX v, 72, 2048
MM 20, 1, v, 104, 168
RD 144, v211, 45056
STORE 14
MM 16, 1, v, 104, 160
RD 148, v211, 46080
LDX v, 76, 3072
s_waitcnt vmcnt(12)
MM 32, 1, v, 112, 160
RD 152, v211, 47104
s_add_u32 s50, s50, 4096
v_accvgpr_read_b32 v55, a247
v_accvgpr_read_b32 v59, a251
v_accvgpr_read_b32 v63, a255
MM 36, 1, v, 112, 168
LDX v, 80, 0
MM 40, 1, v, 112, 176
RD 156, v211, 48128
SIL 15, 0, 0
SIL 15, 0, 1
SIL 15, 0, 2
SIL 15, 0, 3
MM 44, 1, v, 112, 184
LDX v, 84, 1024
s_waitcnt vmcnt(12)
MM 60, 1, v, 120, 184
SIL 15, 0, 4
SIL 15, 0, 5
SIL 15, 0, 6
MM 56, 1, v, 120, 176
LDX v, 88, 2048
MM 52, 1, v, 120, 168
v_readlane_b32 s62, v205, 4
v_readlane_b32 s63, v205, 5
SIL 15, 0, 7
SIL 15, 1, 0
MM 48, 1, v, 120, 160
LDX v, 92, 3072
s_waitcnt lgkmcnt(0)
MM 112, 1, v, 120, 128
RD 160, v211, 49152
SIL 15, 1, 1
SIL 15, 1, 2
MM 116, 1, v, 120, 136
SIL 15, 1, 3
SIL 15, 1, 4
SIL 15, 1, 5
MM 120, 1, v, 120, 144
RD 164, v211, 50176
SIL 15, 1, 6
MM 124, 1, v, 120, 152
RD 168, v211, 51200
CHKF 1
s_cbranch_scc0 .Lslow%=_T3_7_2n
.Lback%=_T3_7_2n:
SIL 15, 1, 7
MM 108, 1, v, 112, 152
RD 172, v211, 52224
s_add_u32 s40, s40, 2621440
SIL 15, 2, 0
MM 104, 1, v, 112, 144
SIL 15, 2, 1
SIL 15, 2, 2
MM 100, 1, v, 112, 136
RD 176, v211, 53248
SIL 15, 2, 3
MM 96, 1, v, 112, 128
RD 180, v211, 54272
SIL 15, 2, 4
MM 80, 1, v, 104, 128
RD 184, v211, 55296
SETW 32768, 42
STADR 15
SIL 15, 2, 5
MM 84, 1, v, 104, 136
LDW 0
MM 88, 1, v, 104, 144
RD 188, v211, 56320
SIL 15, 2, 6
SIL 15, 2, 7
MM 92, 1, v, 104, 152
LDW 1024
MM 76, 1, v, 96, 152
SETW 40960, 43
MM 72, 1, v, 96, 144
LDW 0
MM 68, 1, v, 96, 136
STORE 15
MM 64, 1, v, 96, 128
LDW 1024
s_waitcnt lgkmcnt(0)
MM 128, 1, v, 96, 160
s_waitcnt vmcnt(14)
s_barrier
MM 132, 1, v, 96, 168
RD 128, %2, 0
RD 132, %2, 1024
MM 136, 1, v, 96, 176
MM 140, 1, v, 96, 184
MM 156, 1, v, 104, 184
RD 136, %2, 2048
RD 140, %2, 3072
MM 152, 1, v, 104, 176
MM 148, 1, v, 104, 168
MM 144, 1, v, 104, 160
RD 144, %2, 4096
RD 148, %2, 5120
MM 160, 1, v, 112, 160
RD 152, %2, 6144
MM 164, 1, v, 112, 168
MM 168, 1, v, 112, 176
MM 172, 1, v, 112, 184
RD 156, %2, 7168
MM 188, 1, v, 120, 184
MM 184, 1, v, 120, 176
MM 180, 1, v, 120, 168
MM 176, 1, v, 120, 160
s_mov_b64 s[46:47], s[62:63]
s_waitcnt vmcnt(12) lgkmcnt(0)
MM 0, 1, v, 64, 128
RD 160, %2, 8192
s_add_u32 s50, s40, s48
MM 4, 1, v, 64, 136
LDX v, 96, 0
MM 8, 1, v, 64, 144
RD 164, %2, 9216
LDX v, 100, 1024
MM 12, 1, v, 64, 152
RD 168, %2, 10240
LDX v, 104, 2048
s_waitcnt vmcnt(12)
MM 28, 1, v, 72, 152
RD 172, %2, 11264
LDX v, 108, 3072
s_add_u32 s50, s50, 4096
MM 24, 1, v, 72, 144
LDX v, 112, 0
v_accvgpr_read_b32 v0, a128
v_accvgpr_read_b32 v4, a132
MM 20, 1, v, 72, 136
RD 176, %2, 12288
LDX v, 116, 1024
v_accvgpr_read_b32 v8, a136
MM 16, 1, v, 72, 128
RD 180, %2, 13312
LDX v, 120, 2048
v_accvgpr_read_b32 v12, a140
s_waitcnt vmcnt(14)
MM 32, 1, v, 80, 128
RD 184, %2, 14336
LDX v, 124, 3072
CHKF 2
s_cbranch_scc0 .Lslow%=_T2_0_2n
.Lback%=_T2_0_2n:
MM 36, 1, v, 80, 136
s_add_u32 s40, s40, 2621440
SIL 0, 0, 0
SIL 0, 0, 1
SIL 0, 0, 2
SIL 0, 0, 3
SIL 0, 0, 4
SIL 0, 0, 5
MM 40, 1, v, 80, 144
RD 188, %2, 15360
SETW 65536, 42
SIL 0, 0, 6
SIL 0, 0, 7
SIL 0, 1, 0
MM 44, 1, v, 80, 152
LDW 0
SIL 0, 1, 1
s_waitcnt vmcnt(14)
MM 60, 1, v, 88, 152
LDW 1024
SIL 0, 1, 2
MM 56, 1, v, 88, 144
SETW 73728, 43
SIL 0, 1, 3
SIL 0, 1, 4
SIL 0, 1, 5
SIL 0, 1, 6
MM 52, 1, v, 88, 136
LDW 0
SIL 0, 1, 7
MM 48, 1, v, 88, 128
LDW 1024
SIL 0, 2, 0
s_waitcnt lgkmcnt(0)
MM 112, 1, v, 88, 160
s_waitcnt vmcnt(12)
s_barrier
MM 116, 1, v, 88, 168
RD 128, %2, 32768
RD 132, %2, 33792
s_add_u32 s50, s40, s48
SIL 0, 2, 1
SIL 0, 2, 2
MM 120, 1, v, 88, 176
LDX a, 192, 0
SIL 0, 2, 3
MM 124, 1, v, 88, 184
SIL 0, 2, 4
STADR 0
SIL 0, 2, 5
SIL 0, 2, 6
SIL 0, 2, 7
v_accvgpr_read_b32 v1, a129
v_accvgpr_read_b32 v5, a133
MM 108, 1, v, 80, 184
RD 136, %2, 34816
RD 140, %2, 35840
LDX a, 196, 1024
MM 104, 1, v, 80, 176
STORE 0
MM 100, 1, v, 80, 168
LDX a, 200, 2048
v_accvgpr_read_b32 v9, a137
v_accvgpr_read_b32 v13, a141
MM 96, 1, v, 80, 160
RD 144, %2, 36864
RD 148, %2, 37888
SIL 1, 0, 0
SIL 1, 0, 1
SIL 1, 0, 2
SIL 1, 0, 3
SIL 1, 0, 4
MM 80, 1, v, 72, 160
RD 152, %2, 38912
LDX a, 204, 3072
SIL 1, 0, 5
MM 84, 1, v, 72, 168
s_add_u32 s50, s50, 4096
SIL 1, 0, 6
SIL 1, 0, 7
SIL 1, 1, 0
SIL 1, 1, 1
MM 88, 1, v, 72, 176
LDX a, 208, 0
SIL 1, 1, 2
MM 92, 1, v, 72, 184
RD 156, %2, 39936
SIL 1, 1, 3
SIL 1, 1, 4
SIL 1, 1, 5
SIL 1, 1, 6
SIL 1, 1, 7
MM 76, 1, v, 64, 184
LDX a, 212, 1024
SIL 1, 2, 0
MM 72, 1, v, 64, 176
LDX a, 216, 2048
SIL 1, 2, 1
MM 68, 1, v, 64, 168
LDX a, 220, 3072
SIL 1, 2, 2
MM 64, 1, v, 64, 160
SIL 1, 2, 3
SIL 1, 2, 4
STADR 1
SIL 1, 2, 5
SIL 1, 2, 6
SIL 1, 2, 7
s_waitcnt vmcnt(19) lgkmcnt(0)
MM 0, 1, v, 96, 128
RD 160, %2, 40960
CHKF 3
s_cbranch_scc0 .Lslow%=_T2_1_2n
.Lback%=_T2_1_2n:
v_accvgpr_read_b32 v2, a130
v_accvgpr_read_b32 v6, a134
v_accvgpr_read_b32 v10, a138
MM 4, 1, v, 96, 136
s_add_u32 s40, s40, 2621440
STORE 1
MM 8, 1, v, 96, 144
RD 164, %2, 41984
v_accvgpr_read_b32 v14, a142
SIL 2, 0, 0
SIL 2, 0, 1
SIL 2, 0, 2
SIL 2, 0, 3
SIL 2, 0, 4
MM 12, 1, v, 96, 152
RD 168, %2, 43008
buffer_load_dword v208, v206, %6, s49 offen sc0 sc1
SIL 2, 0, 5
s_waitcnt vmcnt(19)
MM 28, 1, v, 104, 152
RD 172, %2, 44032
SETW 98304, 42
SIL 2, 0, 6
SIL 2, 0, 7
SIL 2, 1, 0
MM 24, 1, v, 104, 144
LDW 0
SIL 2, 1, 1
MM 20, 1, v, 104, 136
RD 176, %2, 45056
SIL 2, 1, 2
SIL 2, 1, 3
SIL 2, 1, 4
SIL 2, 1, 5
MM 16, 1, v, 104, 128
RD 180, %2, 46080
LDW 1024
SIL 2, 1, 6
s_waitcnt vmcnt(19)
MM 32, 1, v, 112, 128
RD 184, %2, 47104
SETW 106496, 43
SIL 2, 1, 7
SIL 2, 2, 0
MM 36, 1, v, 112, 136
LDW 0
SIL 2, 2, 1
MM 40, 1, v, 112, 144
RD 188, %2, 48128
SIL 2, 2, 2
SIL 2, 2, 3
SIL 2, 2, 4
MM 44, 1, v, 112, 152
LDW 1024
STADR 2
SIL 2, 2, 5
s_waitcnt vmcnt(19)
MM 60, 1, v, 120, 152
s_add_u32 s50, s40, s48
SIL 2, 2, 6
SIL 2, 2, 7
MM 56, 1, v, 120, 144
LDX a, 224, 0
v_accvgpr_read_b32 v3, a131
v_accvgpr_read_b32 v7, a135
MM 52, 1, v, 120, 136
STORE 2
MM 48, 1, v, 120, 128
LDX a, 228, 1024
v_accvgpr_read_b32 v11, a139
v_accvgpr_read_b32 v15, a143
s_waitcnt lgkmcnt(0)
MM 112, 1, v, 120, 160
s_waitcnt vmcnt(18)
s_barrier
SIL 3, 0, 0
MM 116, 1, v, 120, 168
RD 128, v211, 0
RD 132, v211, 1024
SIL 3, 0, 1
SIL 3, 0, 2
SIL 3, 0, 3
SIL 3, 0, 4
SIL 3, 0, 5
MM 120, 1, v, 120, 176
LDX a, 232, 2048
SIL 3, 0, 6
MM 124, 1, v, 120, 184
SIL 3, 0, 7
SIL 3, 1, 0
MM 108, 1, v, 112, 184
RD 136, v211, 2048
RD 140, v211, 3072
LDX a, 236, 3072
MM 104, 1, v, 112, 176
s_add_u32 s50, s50, 4096
SIL 3, 1, 1
SIL 3, 1, 2
SIL 3, 1, 3
MM 100, 1, v, 112, 168
LDX a, 240, 0
SIL 3, 1, 4
SIL 3, 1, 5
MM 96, 1, v, 112, 160
RD 144, v211, 4096
RD 148, v211, 5120
SIL 3, 1, 6
SIL 3, 1, 7
SIL 3, 2, 0
MM 80, 1, v, 104, 160
RD 152, v211, 6144
LDX a, 244, 1024
MM 84, 1, v, 104, 168
SIL 3, 2, 1
SIL 3, 2, 2
SIL 3, 2, 3
SIL 3, 2, 4
MM 88, 1, v, 104, 176
LDX a, 248, 2048
STADR 3
SIL 3, 2, 5
MM 92, 1, v, 104, 184
RD 156, v211, 7168
SIL 3, 2, 6
SIL 3, 2, 7
MM 76, 1, v, 96, 184
LDX a, 252, 3072
v_accvgpr_read_b32 v16, a144
MM 72, 1, v, 96, 176
STORE 3
MM 68, 1, v, 96, 168
s_waitcnt vmcnt(14)
v_cmp_le_u32 vcc, s53, v208
v_accvgpr_read_b32 v20, a148
v_accvgpr_read_b32 v24, a152
v_accvgpr_read_b32 v28, a156
SIL 4, 0, 0
MM 64, 1, v, 96, 160
SIL 4, 0, 1
SIL 4, 0, 2
s_waitcnt lgkmcnt(0)
MM 0, 1, a, 192, 128
RD 160, v211, 8192
CHKF 4
s_cbranch_scc0 .Lslow%=_T2_2_2n
.Lback%=_T2_2_2n:
SIL 4, 0, 3
SIL 4, 0, 4
MM 4, 1, a, 192, 136
s_add_u32 s40, s40, 2621440
SIL 4, 0, 5
SIL 4, 0, 6
MM 8, 1, a, 192, 144
RD 164, v211, 9216
SIL 4, 0, 7
SIL 4, 1, 0
MM 12, 1, a, 192, 152
RD 168, v211, 10240
SIL 4, 1, 1
SIL 4, 1, 2
MM 28, 1, a, 200, 152
RD 172, v211, 11264
SETW 0, 42
SIL 4, 1, 3
SIL 4, 1, 4
MM 24, 1, a, 200, 144
LDW 0
SIL 4, 1, 5
SIL 4, 1, 6
MM 20, 1, a, 200, 136
RD 176, v211, 12288
SIL 4, 1, 7
SIL 4, 2, 0
MM 16, 1, a, 200, 128
RD 180, v211, 13312
LDW 1024
MM 32, 1, a, 208, 128
RD 184, v211, 14336
SETW 8192, 43
SIL 4, 2, 1
SIL 4, 2, 2
MM 36, 1, a, 208, 136
LDW 0
SIL 4, 2, 3
MM 40, 1, a, 208, 144
RD 188, v211, 15360
SIL 4, 2, 4
STADR 4
SIL 4, 2, 5
SIL 4, 2, 6
MM 44, 1, a, 208, 152
LDW 1024
SIL 4, 2, 7
v_accvgpr_read_b32 v17, a145
MM 60, 1, a, 216, 152
s_add_u32 s50, s40, s48
STORE 4
MM 56, 1, a, 216, 144
LDX v, 64, 0
v_accvgpr_read_b32 v21, a149
v_accvgpr_read_b32 v25, a153
MM 52, 1, a, 216, 136
v_add_u32 v210, 64, v210
v_accvgpr_read_b32 v29, a157
SIL 5, 0, 0
SIL 5, 0, 1
SIL 5, 0, 2
MM 48, 1, a, 216, 128
LDX v, 68, 1024
SIL 5, 0, 3
SIL 5, 0, 4
s_waitcnt lgkmcnt(0)
MM 112, 1, a, 216, 160
s_waitcnt vmcnt(17)
s_barrier
SIL 5, 0, 5
MM 116, 1, a, 216, 168
RD 128, v211, 32768
RD 132, v211, 33792
SIL 5, 0, 6
SIL 5, 0, 7
MM 120, 1, a, 216, 176
LDX v, 72, 2048
SIL 5, 1, 0
MM 124, 1, a, 216, 184
SIL 5, 1, 1
SIL 5, 1, 2
SIL 5, 1, 3
MM 108, 1, a, 208, 184
RD 136, v211, 34816
RD 140, v211, 35840
LDX v, 76, 3072
MM 104, 1, a, 208, 176
s_add_u32 s50, s50, 4096
SIL 5, 1, 4
SIL 5, 1, 5
SIL 5, 1, 6
SIL 5, 1, 7
MM 100, 1, a, 208, 168
LDX v, 80, 0
SIL 5, 2, 0
MM 96, 1, a, 208, 160
RD 144, v211, 36864
RD 148, v211, 37888
SIL 5, 2, 1
SIL 5, 2, 2
MM 80, 1, a, 200, 160
RD 152, v211, 38912
LDX v, 84, 1024
MM 84, 1, a, 200, 168
SIL 5, 2, 3
SIL 5, 2, 4
STADR 5
SIL 5, 2, 5
SIL 5, 2, 6
MM 88, 1, a, 200, 176
LDX v, 88, 2048
SIL 5, 2, 7
v_accvgpr_read_b32 v18, a146
MM 92, 1, a, 200, 184
RD 156, v211, 39936
STORE 5
MM 76, 1, a, 192, 184
LDX v, 92, 3072
v_accvgpr_read_b32 v22, a150
v_accvgpr_read_b32 v26, a154
MM 72, 1, a, 192, 176
v_accvgpr_read_b32 v30, a158
SIL 6, 0, 0
SIL 6, 0, 1
MM 68, 1, a, 192, 168
SIL 6, 0, 2
SIL 6, 0, 3
MM 64, 1, a, 192, 160
SIL 6, 0, 4
SIL 6, 0, 5
s_waitcnt vmcnt(21) lgkmcnt(0)
MM 0, 1, a, 224, 128
RD 160, v211, 40960
CHKF 5
s_cbranch_scc0 .Lslow%=_T2_3_2n
.Lback%=_T2_3_2n:
SIL 6, 0, 6
SIL 6, 0, 7
MM 4, 1, a, 224, 136
s_add_u32 s40, s40, 2621440
SIL 6, 1, 0
SIL 6, 1, 1
MM 8, 1, a, 224, 144
RD 164, v211, 41984
SIL 6, 1, 2
SIL 6, 1, 3
MM 12, 1, a, 224, 152
RD 168, v211, 43008
SIL 6, 1, 4
SIL 6, 1, 5
s_waitcnt vmcnt(19)
MM 28, 1, a, 232, 152
RD 172, v211, 44032
SETW 32768, 42
SIL 6, 1, 6
SIL 6, 1, 7
MM 24, 1, a, 232, 144
LDW 0
SIL 6, 2, 0
MM 20, 1, a, 232, 136
RD 176, v211, 45056
SIL 6, 2, 1
SIL 6, 2, 2
MM 16, 1, a, 232, 128
RD 180, v211, 46080
LDW 1024
s_waitcnt vmcnt(19)
MM 32, 1, a, 240, 128
RD 184, v211, 47104
SETW 40960, 43
SIL 6, 2, 3
SIL 6, 2, 4
STADR 6
MM 36, 1, a, 240, 136
LDW 0
SIL 6, 2, 5
SIL 6, 2, 6
MM 40, 1, a, 240, 144
RD 188, v211, 48128
SIL 6, 2, 7
v_accvgpr_read_b32 v19, a147
v_accvgpr_read_b32 v23, a151
MM 44, 1, a, 240, 152
LDW 1024
v_accvgpr_read_b32 v27, a155
v_accvgpr_read_b32 v31, a159
s_waitcnt vmcnt(19)
MM 60, 1, a, 248, 152
s_add_u32 s50, s40, s48
STORE 6
MM 56, 1, a, 248, 144
LDX v, 96, 0
SIL 7, 0, 0
SIL 7, 0, 1
MM 52, 1, a, 248, 136
ds_read_b32 v204, v210
ds_read_b32 v205, v210 offset:32
SIL 7, 0, 2
SIL 7, 0, 3
SIL 7, 0, 4
SIL 7, 0, 5
MM 48, 1, a, 248, 128
LDX v, 100, 1024
SIL 7, 0, 6
s_waitcnt lgkmcnt(2)
MM 112, 1, a, 248, 160
s_waitcnt vmcnt(17)
s_barrier
SIL 7, 0, 7
MM 116, 1, a, 248, 168
RD 128, %2, 0
RD 132, %2, 1024
SIL 7, 1, 0
SIL 7, 1, 1
MM 120, 1, a, 248, 176
LDX v, 104, 2048
SIL 7, 1, 2
MM 124, 1, a, 248, 184
SIL 7, 1, 3
SIL 7, 1, 4
SIL 7, 1, 5
SIL 7, 1, 6
MM 108, 1, a, 240, 184
RD 136, %2, 2048
RD 140, %2, 3072
LDX v, 108, 3072
MM 104, 1, a, 240, 176
s_add_u32 s50, s50, 4096
SIL 7, 1, 7
SIL 7, 2, 0
SIL 7, 2, 1
MM 100, 1, a, 240, 168
LDX v, 112, 0
SIL 7, 2, 2
MM 96, 1, a, 240, 160
RD 144, %2, 4096
RD 148, %2, 5120
SIL 7, 2, 3
SIL 7, 2, 4
STADR 7
SIL 7, 2, 5
MM 80, 1, a, 232, 160
RD 152, %2, 6144
LDX v, 116, 1024
SIL 7, 2, 6
MM 84, 1, a, 232, 168
SIL 7, 2, 7
v_accvgpr_read_b32 v32, a160
MM 88, 1, a, 232, 176
LDX v, 120, 2048
v_accvgpr_read_b32 v36, a164
v_accvgpr_read_b32 v40, a168
MM 92, 1, a, 232, 184
RD 156, %2, 7168
STORE 7
MM 76, 1, a, 224, 184
LDX v, 124, 3072
v_accvgpr_read_b32 v44, a172
SIL 8, 0, 0
MM 72, 1, a, 224, 176
SIL 8, 0, 1
SIL 8, 0, 2
SIL 8, 0, 3
SIL 8, 0, 4
MM 68, 1, a, 224, 168
SIL 8, 0, 5
SIL 8, 0, 6
MM 64, 1, a, 224, 160
SIL 8, 0, 7
SIL 8, 1, 0
s_waitcnt vmcnt(21) lgkmcnt(0)
MM 0, 1, v, 64, 128
RD 160, %2, 8192
CHKF 6
s_cbranch_scc0 .Lslow%=_T2_4_2n
.Lback%=_T2_4_2n:
SIL 8, 1, 1
MM 4, 1, v, 64, 136
s_add_u32 s40, s40, 2621440
SIL 8, 1, 2
SIL 8, 1, 3
SIL 8, 1, 4
MM 8, 1, v, 64, 144
RD 164, %2, 9216
SIL 8, 1, 5
SIL 8, 1, 6
MM 12, 1, v, 64, 152
RD 168, %2, 10240
SIL 8, 1, 7
SIL 8, 2, 0
s_waitcnt vmcnt(19)
MM 28, 1, v, 72, 152
RD 172, %2, 11264
SETW 65536, 42
SIL 8, 2, 1
MM 24, 1, v, 72, 144
LDW 0
SIL 8, 2, 2
MM 20, 1, v, 72, 136
RD 176, %2, 12288
SIL 8, 2, 3
SIL 8, 2, 4
STADR 8
MM 16, 1, v, 72, 128
RD 180, %2, 13312
LDW 1024
SIL 8, 2, 5
s_waitcnt vmcnt(19)
MM 32, 1, v, 80, 128
RD 184, %2, 14336
SETW 73728, 43
SIL 8, 2, 6
SIL 8, 2, 7
v_accvgpr_read_b32 v33, a161
MM 36, 1, v, 80, 136
LDW 0
v_accvgpr_read_b32 v37, a165
v_accvgpr_read_b32 v41, a169
MM 40, 1, v, 80, 144
RD 188, %2, 15360
STORE 8
MM 44, 1, v, 80, 152
LDW 1024
v_accvgpr_read_b32 v45, a173
SIL 9, 0, 0
s_waitcnt vmcnt(19)
MM 60, 1, v, 88, 152
s_add_u32 s50, s40, s48
SIL 9, 0, 1
SIL 9, 0, 2
SIL 9, 0, 3
SIL 9, 0, 4
MM 56, 1, v, 88, 144
LDX a, 192, 0
SIL 9, 0, 5
SIL 9, 0, 6
MM 52, 1, v, 88, 136
v_add_u32 v207, v204, %5
SIL 9, 0, 7
MM 48, 1, v, 88, 128
LDX a, 196, 1024
SIL 9, 1, 0
s_waitcnt lgkmcnt(0)
MM 112, 1, v, 88, 160
s_waitcnt vmcnt(17)
s_barrier
MM 116, 1, v, 88, 168
RD 128, %2, 32768
RD 132, %2, 33792
SIL 9, 1, 1
SIL 9, 1, 2
MM 120, 1, v, 88, 176
LDX a, 200, 2048
SIL 9, 1, 3
MM 124, 1, v, 88, 184
SIL 9, 1, 4
SIL 9, 1, 5
SIL 9, 1, 6
SIL 9, 1, 7
SIL 9, 2, 0
MM 108, 1, v, 80, 184
RD 136, %2, 34816
RD 140, %2, 35840
LDX a, 204, 3072
MM 104, 1, v, 80, 176
s_add_u32 s50, s50, 4096
SIL 9, 2, 1
SIL 9, 2, 2
SIL 9, 2, 3
MM 100, 1, v, 80, 168
LDX a, 208, 0
SIL 9, 2, 4
STADR 9
MM 96, 1, v, 80, 160
RD 144, %2, 36864
RD 148, %2, 37888
SIL 9, 2, 5
SIL 9, 2, 6
SIL 9, 2, 7
v_accvgpr_read_b32 v34, a162
MM 80, 1, v, 72, 160
RD 152, %2, 38912
LDX a, 212, 1024
v_accvgpr_read_b32 v38, a166
MM 84, 1, v, 72, 168
STORE 9
MM 88, 1, v, 72, 176
LDX a, 216, 2048
v_accvgpr_read_b32 v42, a170
v_accvgpr_read_b32 v46, a174
MM 92, 1, v, 72, 184
RD 156, %2, 39936
SIL 10, 0, 0
SIL 10, 0, 1
SIL 10, 0, 2
SIL 10, 0, 3
MM 76, 1, v, 64, 184
LDX a, 220, 3072
SIL 10, 0, 4
SIL 10, 0, 5
MM 72, 1, v, 64, 176
SIL 10, 0, 6
SIL 10, 0, 7
MM 68, 1, v, 64, 168
SIL 10, 1, 0
SIL 10, 1, 1
MM 64, 1, v, 64, 160
SIL 10, 1, 2
SIL 10, 1, 3
s_waitcnt vmcnt(21) lgkmcnt(0)
MM 0, 1, v, 96, 128
RD 160, %2, 40960
CHKF 7
s_cbranch_scc0 .Lslow%=_T2_5_2n
.Lback%=_T2_5_2n:
SIL 10, 1, 4
SIL 10, 1, 5
MM 4, 1, v, 96, 136
s_add_u32 s40, s40, 2621440
SIL 10, 1, 6
SIL 10, 1, 7
MM 8, 1, v, 96, 144
RD 164, %2, 41984
s_add_u32 s49, s49, 10240
s_cmp_eq_u32 s49, 40960
s_cselect_b32 s49, 0, s49
SIL 10, 2, 0
MM 12, 1, v, 96, 152
RD 168, %2, 43008
v_mov_b32 v206, v207
buffer_load_dword v208, v206, %6, s49 offen sc0 sc1
s_waitcnt vmcnt(20)
MM 28, 1, v, 104, 152
RD 172, %2, 44032
SETW 98304, 42
SIL 10, 2, 1
SIL 10, 2, 2
MM 24, 1, v, 104, 144
LDW 0
SIL 10, 2, 3
MM 20, 1, v, 104, 136
RD 176, %2, 45056
SIL 10, 2, 4
STADR 10
SIL 10, 2, 5
SIL 10, 2, 6
SIL 10, 2, 7
MM 16, 1, v, 104, 128
RD 180, %2, 46080
LDW 1024
v_accvgpr_read_b32 v35, a163
s_waitcnt vmcnt(20)
MM 32, 1, v, 112, 128
RD 184, %2, 47104
SETW 106496, 43
v_accvgpr_read_b32 v39, a167
v_accvgpr_read_b32 v43, a171
v_accvgpr_read_b32 v47, a175
MM 36, 1, v, 112, 136
LDW 0
SIL 11, 0, 0
SIL 11, 0, 1
MM 40, 1, v, 112, 144
RD 188, %2, 48128
STORE 10
MM 44, 1, v, 112, 152
LDW 1024
SIL 11, 0, 2
SIL 11, 0, 3
s_waitcnt vmcnt(20)
MM 60, 1, v, 120, 152
s_add_u32 s50, s40, s48
SIL 11, 0, 4
SIL 11, 0, 5
SIL 11, 0, 6
MM 56, 1, v, 120, 144
LDX a, 224, 0
SIL 11, 0, 7
MM 52, 1, v, 120, 136
SIL 11, 1, 0
SIL 11, 1, 1
SIL 11, 1, 2
MM 48, 1, v, 120, 128
LDX a, 228, 1024
SIL 11, 1, 3
s_waitcnt lgkmcnt(0)
MM 112, 1, v, 120, 160
s_waitcnt vmcnt(17)
s_barrier
SIL 11, 1, 4
MM 116, 1, v, 120, 168
RD 128, v211, 0
RD 132, v211, 1024
SIL 11, 1, 5
SIL 11, 1, 6
SIL 11, 1, 7
SIL 11, 2, 0
MM 120, 1, v, 120, 176
LDX a, 232, 2048
SIL 11, 2, 1
MM 124, 1, v, 120, 184
SIL 11, 2, 2
SIL 11, 2, 3
SIL 11, 2, 4
MM 108, 1, v, 112, 184
RD 136, v211, 2048
RD 140, v211, 3072
LDX a, 236, 3072
MM 104, 1, v, 112, 176
s_add_u32 s50, s50, 4096
STADR 11
SIL 11, 2, 5
SIL 11, 2, 6
SIL 11, 2, 7
MM 100, 1, v, 112, 168
LDX a, 240, 0
v_accvgpr_read_b32 v48, a176
MM 96, 1, v, 112, 160
RD 144, v211, 4096
RD 148, v211, 5120
STORE 11
MM 80, 1, v, 104, 160
RD 152, v211, 6144
LDX a, 244, 1024
v_accvgpr_read_b32 v52, a180
MM 84, 1, v, 104, 168
v_accvgpr_read_b32 v56, a184
v_accvgpr_read_b32 v60, a188
SIL 12, 0, 0
SIL 12, 0, 1
SIL 12, 0, 2
MM 88, 1, v, 104, 176
LDX a, 248, 2048
SIL 12, 0, 3
SIL 12, 0, 4
MM 92, 1, v, 104, 184
RD 156, v211, 7168
SIL 12, 0, 5
SIL 12, 0, 6
MM 76, 1, v, 96, 184
LDX a, 252, 3072
SIL 12, 0, 7
MM 72, 1, v, 96, 176
SIL 12, 1, 0
SIL 12, 1, 1
SIL 12, 1, 2
MM 68, 1, v, 96, 168
s_waitcnt vmcnt(14)
v_cmp_le_u32 vcc, s53, v208
SIL 12, 1, 3
SIL 12, 1, 4
MM 64, 1, v, 96, 160
SIL 12, 1, 5
SIL 12, 1, 6
s_waitcnt lgkmcnt(0)
MM 0, 1, a, 192, 128
RD 160, v211, 8192
CHKF 0
s_cbranch_scc0 .Lslow%=_T2_6_2n
.Lback%=_T2_6_2n:
SIL 12, 1, 7
SIL 12, 2, 0
MM 4, 1, a, 192, 136
s_add_u32 s40, s40, 2621440
s_cmp_eq_u32 s40, s60
s_cselect_b32 s40, %13, s40
SIL 12, 2, 1
MM 8, 1, a, 192, 144
RD 164, v211, 9216
SIL 12, 2, 2
SIL 12, 2, 3
MM 12, 1, a, 192, 152
RD 168, v211, 10240
SIL 12, 2, 4
STADR 12
MM 28, 1, a, 200, 152
RD 172, v211, 11264
SETW 0, 42
SIL 12, 2, 5
SIL 12, 2, 6
MM 24, 1, a, 200, 144
LDW 0
SIL 12, 2, 7
v_accvgpr_read_b32 v49, a177
MM 20, 1, a, 200, 136
RD 176, v211, 12288
STORE 12
MM 16, 1, a, 200, 128
RD 180, v211, 13312
LDW 1024
v_accvgpr_read_b32 v53, a181
MM 32, 1, a, 208, 128
RD 184, v211, 14336
v_accvgpr_read_b32 v57, a185
v_accvgpr_read_b32 v61, a189
SIL 13, 0, 0
SIL 13, 0, 1
SIL 13, 0, 2
MM 36, 1, a, 208, 136
SIL 13, 0, 3
SIL 13, 0, 4
MM 40, 1, a, 208, 144
RD 188, v211, 15360
SIL 13, 0, 5
SIL 13, 0, 6
MM 44, 1, a, 208, 152
SIL 13, 0, 7
MM 60, 1, a, 216, 152
s_add_u32 s50, s40, s48
SIL 13, 1, 0
SIL 13, 1, 1
MM 56, 1, a, 216, 144
LDX v, 64, 0
SIL 13, 1, 2
MM 52, 1, a, 216, 136
SIL 13, 1, 3
SIL 13, 1, 4
SIL 13, 1, 5
MM 48, 1, a, 216, 128
LDX v, 68, 1024
SIL 13, 1, 6
SIL 13, 1, 7
s_waitcnt lgkmcnt(0)
MM 112, 1, a, 216, 160
s_waitcnt vmcnt(14)
s_barrier
MM 116, 1, a, 216, 168
RD 128, v211, 32768
RD 132, v211, 33792
SIL 13, 2, 0
SIL 13, 2, 1
MM 120, 1, a, 216, 176
LDX v, 72, 2048
SIL 13, 2, 2
MM 124, 1, a, 216, 184
SIL 13, 2, 3
SIL 13, 2, 4
STADR 13
SIL 13, 2, 5
SIL 13, 2, 6
MM 108, 1, a, 208, 184
RD 136, v211, 34816
RD 140, v211, 35840
LDX v, 76, 3072
MM 104, 1, a, 208, 176
s_add_u32 s50, s50, 4096
SIL 13, 2, 7
v_accvgpr_read_b32 v50, a178
v_accvgpr_read_b32 v54, a182
MM 100, 1, a, 208, 168
LDX v, 80, 0
v_accvgpr_read_b32 v58, a186
v_accvgpr_read_b32 v62, a190
MM 96, 1, a, 208, 160
RD 144, v211, 36864
RD 148, v211, 37888
STORE 13
MM 80, 1, a, 200, 160
RD 152, v211, 38912
LDX v, 84, 1024
SIL 14, 0, 0
MM 84, 1, a, 200, 168
SIL 14, 0, 1
SIL 14, 0, 2
SIL 14, 0, 3
SIL 14, 0, 4
SIL 14, 0, 5
MM 88, 1, a, 200, 176
LDX v, 88, 2048
SIL 14, 0, 6
SIL 14, 0, 7
MM 92, 1, a, 200, 184
RD 156, v211, 39936
SIL 14, 1, 0
SIL 14, 1, 1
MM 76, 1, a, 192, 184
LDX v, 92, 3072
SIL 14, 1, 2
MM 72, 1, a, 192, 176
SIL 14, 1, 3
SIL 14, 1, 4
SIL 14, 1, 5
MM 68, 1, a, 192, 168
SIL 14, 1, 6
SIL 14, 1, 7
MM 64, 1, a, 192, 160
SIL 14, 2, 0
s_waitcnt vmcnt(19) lgkmcnt(0)
MM 0, 1, a, 224, 128
RD 160, v211, 40960
CHKF 1
s_cbranch_scc0 .Lslow%=_T2_7_2n
.Lback%=_T2_7_2n:
SIL 14, 2, 1
MM 4, 1, a, 224, 136
s_add_u32 s40, s40, 2621440
SIL 14, 2, 2
SIL 14, 2, 3
SIL 14, 2, 4
MM 8, 1, a, 224, 144
RD 164, v211, 41984
STADR 14
SIL 14, 2, 5
MM 12, 1, a, 224, 152
RD 168, v211, 43008
buffer_load_dword v208, v206, %6, s49 offen sc0 sc1
SIL 14, 2, 6
s_waitcnt vmcnt(18)
MM 28, 1, a, 232, 152
RD 172, v211, 44032
SETW 32768, 42
SIL 14, 2, 7
v_accvgpr_read_b32 v51, a179
v_accvgpr_read_b32 v55, a183
MM 24, 1, a, 232, 144
LDW 0
v_accvgpr_read_b32 v59, a187
v_accvgpr_read_b32 v63, a191
MM 20, 1, a, 232, 136
RD 176, v211, 45056
STORE 14
MM 16, 1, a, 232, 128
RD 180, v211, 46080
LDW 1024
SIL 15, 0, 0
s_waitcnt vmcnt(18)
MM 32, 1, a, 240, 128
RD 184, v211, 47104
SIL 15, 0, 1
SIL 15, 0, 2
SIL 15, 0, 3
SIL 15, 0, 4
SIL 15, 0, 5
MM 36, 1, a, 240, 136
SIL 15, 0, 6
MM 40, 1, a, 240, 144
RD 188, v211, 48128
SIL 15, 0, 7
SIL 15, 1, 0
MM 44, 1, a, 240, 152
SIL 15, 1, 1
SIL 15, 1, 2
s_waitcnt vmcnt(16)
MM 60, 1, a, 248, 152
s_add_u32 s50, s40, s48
SIL 15, 1, 3
SIL 15, 1, 4
MM 56, 1, a, 248, 144
LDX v, 96, 0
SIL 15, 1, 5
SIL 15, 1, 6
MM 52, 1, a, 248, 136
v_readlane_b32 s62, v205, 4
v_readlane_b32 s63, v205, 5
SIL 15, 1, 7
SIL 15, 2, 0
MM 48, 1, a, 248, 128
LDX v, 100, 1024
SIL 15, 2, 1
s_waitcnt lgkmcnt(0)
MM 112, 1, a, 248, 160
s_waitcnt vmcnt(15)
s_barrier
MM 116, 1, a, 248, 168
RD 128, %2, 0
RD 132, %2, 1024
SIL 15, 2, 2
SIL 15, 2, 3
SIL 15, 2, 4
MM 120, 1, a, 248, 176
LDX v, 104, 2048
STADR 15
SIL 15, 2, 5
MM 124, 1, a, 248, 184
SIL 15, 2, 6
SIL 15, 2, 7
MM 108, 1, a, 240, 184
RD 136, %2, 2048
RD 140, %2, 3072
LDX v, 108, 3072
MM 104, 1, a, 240, 176
s_add_u32 s50, s50, 4096
STORE 15
MM 100, 1, a, 240, 168
LDX v, 112, 0
MM 96, 1, a, 240, 160
RD 144, %2, 4096
RD 148, %2, 5120
MM 80, 1, a, 232, 160
RD 152, %2, 6144
LDX v, 116, 1024
MM 84, 1, a, 232, 168
MM 88, 1, a, 232, 176
LDX v, 120, 2048
MM 92, 1, a, 232, 184
RD 156, %2, 7168
MM 76, 1, a, 224, 184
LDX v, 124, 3072
MM 72, 1, a, 224, 176
MM 68, 1, a, 224, 168
s_waitcnt vmcnt(12)
v_cmp_le_u32 vcc, s53, v208
MM 64, 1, a, 224, 160
s_mov_b64 s[46:47], s[62:63]
s_waitcnt lgkmcnt(0)
MM 0, 1, v, 64, 128
s_waitcnt vmcnt(9)
s_barrier
CHKF 2
s_cbranch_scc0 .Lslow%=_T1_0_2n
.Lback%=_T1_0_2n:
MM 4, 1, v, 64, 136
RD 160, %2, 32768
RD 164, %2, 33792
s_add_u32 s40, s40, 2621440
SETW 65536, 42
MM 8, 1, v, 64, 144
LDW 0
LDW 1024
s_add_u32 s50, s40, s48
MM 12, 1, v, 64, 152
LDX a, 192, 0
LDX a, 196, 1024
MM 28, 1, v, 72, 152
RD 168, %2, 34816
RD 172, %2, 35840
LDX a, 200, 2048
LDX a, 204, 3072
s_add_u32 s50, s50, 4096
MM 24, 1, v, 72, 144
LDX a, 208, 0
LDX a, 212, 1024
v_accvgpr_read_b32 v0, a64
v_accvgpr_read_b32 v4, a68
MM 20, 1, v, 72, 136
LDX a, 216, 2048
LDX a, 220, 3072
v_accvgpr_read_b32 v8, a72
v_accvgpr_read_b32 v12, a76
MM 16, 1, v, 72, 128
RD 176, %2, 36864
RD 180, %2, 37888
CHKF 3
s_cbranch_scc0 .Lslow%=_T1_0_3n
.Lback%=_T1_0_3n:
SIL 0, 0, 0
SIL 0, 0, 1
SIL 0, 0, 2
SIL 0, 0, 3
SIL 0, 0, 4
SIL 0, 0, 5
SIL 0, 0, 6
MM 32, 1, v, 80, 128
RD 184, %2, 38912
s_add_u32 s40, s40, 2621440
SETW 98304, 42
SIL 0, 0, 7
SIL 0, 1, 0
SIL 0, 1, 1
SIL 0, 1, 2
MM 36, 1, v, 80, 136
LDW 0
SIL 0, 1, 3
SIL 0, 1, 4
SIL 0, 1, 5
SIL 0, 1, 6
SIL 0, 1, 7
MM 40, 1, v, 80, 144
LDW 1024
s_add_u32 s50, s40, s48
SIL 0, 2, 0
SIL 0, 2, 1
SIL 0, 2, 2
MM 44, 1, v, 80, 152
RD 188, %2, 39936
LDX a, 224, 0
LDX a, 228, 1024
MM 60, 1, v, 88, 152
LDX a, 232, 2048
LDX a, 236, 3072
s_add_u32 s50, s50, 4096
MM 56, 1, v, 88, 144
LDX a, 240, 0
LDX a, 244, 1024
SIL 0, 2, 3
MM 52, 1, v, 88, 136
LDX a, 248, 2048
SIL 0, 2, 4
STADR 0
SIL 0, 2, 5
SIL 0, 2, 6
SIL 0, 2, 7
v_accvgpr_read_b32 v1, a65
v_accvgpr_read_b32 v5, a69
MM 48, 1, v, 88, 128
LDX a, 252, 3072
STORE 0
s_waitcnt vmcnt(28) lgkmcnt(0)
MM 0, 1, v, 96, 160
s_waitcnt vmcnt(19)
s_barrier
v_accvgpr_read_b32 v9, a73
v_accvgpr_read_b32 v13, a77
SIL 1, 0, 0
SIL 1, 0, 1
SIL 1, 0, 2
SIL 1, 0, 3
MM 4, 1, v, 96, 168
RD 128, v211, 0
RD 132, v211, 1024
CHKF 4
s_cbranch_scc0 .Lslow%=_T1_1_3n
.Lback%=_T1_1_3n:
SIL 1, 0, 4
SIL 1, 0, 5
SIL 1, 0, 6
SIL 1, 0, 7
SIL 1, 1, 0
MM 8, 1, v, 96, 176
s_add_u32 s40, s40, 2621440
SETW 0, 42
SIL 1, 1, 1
SIL 1, 1, 2
SIL 1, 1, 3
SIL 1, 1, 4
SIL 1, 1, 5
SIL 1, 1, 6
MM 12, 1, v, 96, 184
LDW 0
SIL 1, 1, 7
SIL 1, 2, 0
SIL 1, 2, 1
SIL 1, 2, 2
MM 28, 1, v, 104, 184
RD 136, v211, 2048
RD 140, v211, 3072
LDW 1024
SIL 1, 2, 3
SIL 1, 2, 4
STADR 1
SIL 1, 2, 5
MM 24, 1, v, 104, 176
s_add_u32 s50, s40, s48
SIL 1, 2, 6
SIL 1, 2, 7
v_accvgpr_read_b32 v2, a66
v_accvgpr_read_b32 v6, a70
v_accvgpr_read_b32 v10, a74
v_accvgpr_read_b32 v14, a78
SIL 2, 0, 0
SIL 2, 0, 1
SIL 2, 0, 2
SIL 2, 0, 3
SIL 2, 0, 4
MM 20, 1, v, 104, 168
LDX v, 64, 0
STORE 1
MM 16, 1, v, 104, 160
RD 144, v211, 4096
RD 148, v211, 5120
LDX v, 68, 1024
SIL 2, 0, 5
SIL 2, 0, 6
SIL 2, 0, 7
SIL 2, 1, 0
MM 32, 1, v, 112, 160
RD 152, v211, 6144
LDX v, 72, 2048
SIL 2, 1, 1
SIL 2, 1, 2
SIL 2, 1, 3
MM 36, 1, v, 112, 168
SIL 2, 1, 4
SIL 2, 1, 5
SIL 2, 1, 6
SIL 2, 1, 7
SIL 2, 2, 0
SIL 2, 2, 1
SIL 2, 2, 2
SIL 2, 2, 3
MM 40, 1, v, 112, 176
LDX v, 76, 3072
s_add_u32 s50, s50, 4096
SIL 2, 2, 4
STADR 2
SIL 2, 2, 5
SIL 2, 2, 6
SIL 2, 2, 7
v_accvgpr_read_b32 v3, a67
MM 44, 1, v, 112, 184
RD 156, v211, 7168
LDX v, 80, 0
STORE 2
MM 60, 1, v, 120, 184
LDX v, 84, 1024
v_accvgpr_read_b32 v7, a71
v_accvgpr_read_b32 v11, a75
v_accvgpr_read_b32 v15, a79
SIL 3, 0, 0
SIL 3, 0, 1
SIL 3, 0, 2
SIL 3, 0, 3
MM 56, 1, v, 120, 176
LDX v, 88, 2048
SIL 3, 0, 4
SIL 3, 0, 5
SIL 3, 0, 6
SIL 3, 0, 7
SIL 3, 1, 0
MM 52, 1, v, 120, 168
LDX v, 92, 3072
SIL 3, 1, 1
SIL 3, 1, 2
SIL 3, 1, 3
SIL 3, 1, 4
MM 48, 1, v, 120, 160
SIL 3, 1, 5
SIL 3, 1, 6
SIL 3, 1, 7
SIL 3, 2, 0
SIL 3, 2, 1
SIL 3, 2, 2
SIL 3, 2, 3
SIL 3, 2, 4
s_waitcnt vmcnt(29) lgkmcnt(0)
MM 0, 1, a, 192, 128
s_waitcnt vmcnt(21)
s_barrier
STADR 3
SIL 3, 2, 5
SIL 3, 2, 6
SIL 3, 2, 7
v_accvgpr_read_b32 v16, a80
v_accvgpr_read_b32 v20, a84
MM 4, 1, a, 192, 136
RD 160, v211, 32768
RD 164, v211, 33792
CHKF 5
s_cbranch_scc0 .Lslow%=_T1_2_3n
.Lback%=_T1_2_3n:
STORE 3
MM 8, 1, a, 192, 144
s_add_u32 s40, s40, 2621440
SETW 32768, 42
v_accvgpr_read_b32 v24, a88
v_accvgpr_read_b32 v28, a92
SIL 4, 0, 0
SIL 4, 0, 1
SIL 4, 0, 2
SIL 4, 0, 3
SIL 4, 0, 4
SIL 4, 0, 5
SIL 4, 0, 6
MM 12, 1, a, 192, 152
LDW 0
SIL 4, 0, 7
SIL 4, 1, 0
SIL 4, 1, 1
SIL 4, 1, 2
MM 28, 1, a, 200, 152
RD 168, v211, 34816
RD 172, v211, 35840
LDW 1024
SIL 4, 1, 3
SIL 4, 1, 4
SIL 4, 1, 5
SIL 4, 1, 6
MM 24, 1, a, 200, 144
s_add_u32 s50, s40, s48
SIL 4, 1, 7
SIL 4, 2, 0
SIL 4, 2, 1
SIL 4, 2, 2
SIL 4, 2, 3
SIL 4, 2, 4
STADR 4
MM 20, 1, a, 200, 136
LDX v, 96, 0
SIL 4, 2, 5
SIL 4, 2, 6
SIL 4, 2, 7
v_accvgpr_read_b32 v17, a81
v_accvgpr_read_b32 v21, a85
v_accvgpr_read_b32 v25, a89
v_accvgpr_read_b32 v29, a93
MM 16, 1, a, 200, 128
RD 176, v211, 36864
RD 180, v211, 37888
LDX v, 100, 1024
SIL 5, 0, 0
SIL 5, 0, 1
SIL 5, 0, 2
SIL 5, 0, 3
SIL 5, 0, 4
MM 32, 1, a, 208, 128
RD 184, v211, 38912
LDX v, 104, 2048
STORE 4
MM 36, 1, a, 208, 136
SIL 5, 0, 5
SIL 5, 0, 6
SIL 5, 0, 7
SIL 5, 1, 0
SIL 5, 1, 1
SIL 5, 1, 2
SIL 5, 1, 3
MM 40, 1, a, 208, 144
LDX v, 108, 3072
s_add_u32 s50, s50, 4096
SIL 5, 1, 4
SIL 5, 1, 5
SIL 5, 1, 6
SIL 5, 1, 7
MM 44, 1, a, 208, 152
RD 188, v211, 39936
LDX v, 112, 0
SIL 5, 2, 0
SIL 5, 2, 1
SIL 5, 2, 2
MM 60, 1, a, 216, 152
LDX v, 116, 1024
SIL 5, 2, 3
SIL 5, 2, 4
STADR 5
SIL 5, 2, 5
SIL 5, 2, 6
MM 56, 1, a, 216, 144
LDX v, 120, 2048
SIL 5, 2, 7
v_accvgpr_read_b32 v18, a82
v_accvgpr_read_b32 v22, a86
MM 52, 1, a, 216, 136
LDX v, 124, 3072
v_add_u32 v210, 64, v210
STORE 5
MM 48, 1, a, 216, 128
v_accvgpr_read_b32 v26, a90
v_accvgpr_read_b32 v30, a94
SIL 6, 0, 0
SIL 6, 0, 1
SIL 6, 0, 2
SIL 6, 0, 3
SIL 6, 0, 4
SIL 6, 0, 5
s_waitcnt vmcnt(32) lgkmcnt(0)
MM 0, 1, a, 224, 160
s_waitcnt vmcnt(23)
s_barrier
SIL 6, 0, 6
SIL 6, 0, 7
SIL 6, 1, 0
SIL 6, 1, 1
MM 4, 1, a, 224, 168
RD 128, %2, 0
RD 132, %2, 1024
CHKF 6
s_cbranch_scc0 .Lslow%=_T1_3_3n
.Lback%=_T1_3_3n:
SIL 6, 1, 2
SIL 6, 1, 3
SIL 6, 1, 4
SIL 6, 1, 5
MM 8, 1, a, 224, 176
s_add_u32 s40, s40, 2621440
SETW 65536, 42
SIL 6, 1, 6
SIL 6, 1, 7
SIL 6, 2, 0
MM 12, 1, a, 224, 184
LDW 0
SIL 6, 2, 1
SIL 6, 2, 2
SIL 6, 2, 3
SIL 6, 2, 4
MM 28, 1, a, 232, 184
RD 136, %2, 2048
RD 140, %2, 3072
LDW 1024
STADR 6
SIL 6, 2, 5
SIL 6, 2, 6
SIL 6, 2, 7
MM 24, 1, a, 232, 176
s_add_u32 s50, s40, s48
STORE 6
MM 20, 1, a, 232, 168
LDX a, 192, 0
v_accvgpr_read_b32 v19, a83
v_accvgpr_read_b32 v23, a87
v_accvgpr_read_b32 v27, a91
v_accvgpr_read_b32 v31, a95
SIL 7, 0, 0
SIL 7, 0, 1
SIL 7, 0, 2
MM 16, 1, a, 232, 160
RD 144, %2, 4096
RD 148, %2, 5120
LDX a, 196, 1024
SIL 7, 0, 3
SIL 7, 0, 4
SIL 7, 0, 5
SIL 7, 0, 6
MM 32, 1, a, 240, 160
RD 152, %2, 6144
LDX a, 200, 2048
SIL 7, 0, 7
SIL 7, 1, 0
SIL 7, 1, 1
MM 36, 1, a, 240, 168
SIL 7, 1, 2
SIL 7, 1, 3
SIL 7, 1, 4
SIL 7, 1, 5
SIL 7, 1, 6
MM 40, 1, a, 240, 176
LDX a, 204, 3072
s_add_u32 s50, s50, 4096
SIL 7, 1, 7
SIL 7, 2, 0
SIL 7, 2, 1
MM 44, 1, a, 240, 184
RD 156, %2, 7168
LDX a, 208, 0
SIL 7, 2, 2
SIL 7, 2, 3
SIL 7, 2, 4
STADR 7
MM 60, 1, a, 248, 184
LDX a, 212, 1024
SIL 7, 2, 5
SIL 7, 2, 6
SIL 7, 2, 7
v_accvgpr_read_b32 v32, a96
MM 56, 1, a, 248, 176
LDX a, 216, 2048
STORE 7
MM 52, 1, a, 248, 168
LDX a, 220, 3072
ds_read_b32 v204, v210
ds_read_b32 v205, v210 offset:32
v_accvgpr_read_b32 v36, a100
v_accvgpr_read_b32 v40, a104
v_accvgpr_read_b32 v44, a108
SIL 8, 0, 0
SIL 8, 0, 1
MM 48, 1, a, 248, 160
SIL 8, 0, 2
SIL 8, 0, 3
SIL 8, 0, 4
SIL 8, 0, 5
SIL 8, 0, 6
SIL 8, 0, 7
SIL 8, 1, 0
s_waitcnt vmcnt(32) lgkmcnt(2)
MM 0, 1, v, 64, 128
s_waitcnt vmcnt(22)
s_barrier
SIL 8, 1, 1
SIL 8, 1, 2
SIL 8, 1, 3
MM 4, 1, v, 64, 136
RD 160, %2, 32768
RD 164, %2, 33792
CHKF 7
s_cbranch_scc0 .Lslow%=_T1_4_3n
.Lback%=_T1_4_3n:
SIL 8, 1, 4
SIL 8, 1, 5
SIL 8, 1, 6
SIL 8, 1, 7
SIL 8, 2, 0
MM 8, 1, v, 64, 144
s_add_u32 s40, s40, 2621440
SETW 98304, 42
SIL 8, 2, 1
SIL 8, 2, 2
SIL 8, 2, 3
MM 12, 1, v, 64, 152
LDW 0
SIL 8, 2, 4
STADR 8
SIL 8, 2, 5
SIL 8, 2, 6
MM 28, 1, v, 72, 152
RD 168, %2, 34816
RD 172, %2, 35840
LDW 1024
SIL 8, 2, 7
v_accvgpr_read_b32 v33, a97
v_accvgpr_read_b32 v37, a101
v_accvgpr_read_b32 v41, a105
MM 24, 1, v, 72, 144
s_add_u32 s50, s40, s48
STORE 8
MM 20, 1, v, 72, 136
LDX a, 224, 0
v_accvgpr_read_b32 v45, a109
SIL 9, 0, 0
SIL 9, 0, 1
SIL 9, 0, 2
SIL 9, 0, 3
SIL 9, 0, 4
SIL 9, 0, 5
MM 16, 1, v, 72, 128
RD 176, %2, 36864
RD 180, %2, 37888
LDX a, 228, 1024
SIL 9, 0, 6
SIL 9, 0, 7
SIL 9, 1, 0
MM 32, 1, v, 80, 128
RD 184, %2, 38912
LDX a, 232, 2048
SIL 9, 1, 1
SIL 9, 1, 2
SIL 9, 1, 3
MM 36, 1, v, 80, 136
SIL 9, 1, 4
SIL 9, 1, 5
SIL 9, 1, 6
SIL 9, 1, 7
SIL 9, 2, 0
SIL 9, 2, 1
MM 40, 1, v, 80, 144
LDX a, 236, 3072
s_add_u32 s50, s50, 4096
SIL 9, 2, 2
SIL 9, 2, 3
SIL 9, 2, 4
STADR 9
MM 44, 1, v, 80, 152
RD 188, %2, 39936
LDX a, 240, 0
SIL 9, 2, 5
SIL 9, 2, 6
SIL 9, 2, 7
v_accvgpr_read_b32 v34, a98
MM 60, 1, v, 88, 152
LDX a, 244, 1024
STORE 9
MM 56, 1, v, 88, 144
LDX a, 248, 2048
v_accvgpr_read_b32 v38, a102
v_accvgpr_read_b32 v42, a106
v_accvgpr_read_b32 v46, a110
SIL 10, 0, 0
SIL 10, 0, 1
SIL 10, 0, 2
SIL 10, 0, 3
MM 52, 1, v, 88, 136
LDX a, 252, 3072
s_waitcnt lgkmcnt(8)
v_add_u32 v207, v204, %5
SIL 10, 0, 4
SIL 10, 0, 5
SIL 10, 0, 6
SIL 10, 0, 7
MM 48, 1, v, 88, 128
SIL 10, 1, 0
SIL 10, 1, 1
SIL 10, 1, 2
SIL 10, 1, 3
s_waitcnt vmcnt(32) lgkmcnt(0)
MM 0, 1, v, 96, 160
s_waitcnt vmcnt(22)
s_barrier
SIL 10, 1, 4
SIL 10, 1, 5
SIL 10, 1, 6
SIL 10, 1, 7
MM 4, 1, v, 96, 168
RD 128, v211, 0
RD 132, v211, 1024
SIL 10, 2, 0
SIL 10, 2, 1
SIL 10, 2, 2
MM 8, 1, v, 96, 176
SIL 10, 2, 3
SIL 10, 2, 4
STADR 10
SIL 10, 2, 5
MM 12, 1, v, 96, 184
SIL 10, 2, 6
SIL 10, 2, 7
v_accvgpr_read_b32 v35, a99
v_accvgpr_read_b32 v39, a103
MM 28, 1, v, 104, 184
RD 136, v211, 2048
RD 140, v211, 3072
STORE 10
MM 24, 1, v, 104, 176
v_accvgpr_read_b32 v43, a107
v_accvgpr_read_b32 v47, a111
SIL 11, 0, 0
SIL 11, 0, 1
SIL 11, 0, 2
SIL 11, 0, 3
SIL 11, 0, 4
SIL 11, 0, 5
MM 20, 1, v, 104, 168
SIL 11, 0, 6
SIL 11, 0, 7
SIL 11, 1, 0
MM 16, 1, v, 104, 160
RD 144, v211, 4096
RD 148, v211, 5120
SIL 11, 1, 1
SIL 11, 1, 2
SIL 11, 1, 3
SIL 11, 1, 4
MM 32, 1, v, 112, 160
RD 152, v211, 6144
SIL 11, 1, 5
SIL 11, 1, 6
SIL 11, 1, 7
SIL 11, 2, 0
MM 36, 1, v, 112, 168
SIL 11, 2, 1
SIL 11, 2, 2
SIL 11, 2, 3
SIL 11, 2, 4
MM 40, 1, v, 112, 176
STADR 11
SIL 11, 2, 5
SIL 11, 2, 6
SIL 11, 2, 7
MM 44, 1, v, 112, 184
RD 156, v211, 7168
STORE 11
MM 60, 1, v, 120, 184
v_accvgpr_read_b32 v48, a112
v_accvgpr_read_b32 v52, a116
v_accvgpr_read_b32 v56, a120
v_accvgpr_read_b32 v60, a124
SIL 12, 0, 0
SIL 12, 0, 1
SIL 12, 0, 2
MM 56, 1, v, 120, 176
SIL 12, 0, 3
SIL 12, 0, 4
SIL 12, 0, 5
SIL 12, 0, 6
MM 52, 1, v, 120, 168
SIL 12, 0, 7
SIL 12, 1, 0
SIL 12, 1, 1
SIL 12, 1, 2
MM 48, 1, v, 120, 160
SIL 12, 1, 3
SIL 12, 1, 4
SIL 12, 1, 5
SIL 12, 1, 6
s_waitcnt vmcnt(21) lgkmcnt(0)
MM 0, 1, a, 192, 128
s_waitcnt vmcnt(12)
s_barrier
SIL 12, 1, 7
SIL 12, 2, 0
SIL 12, 2, 1
MM 4, 1, a, 192, 136
RD 160, v211, 32768
RD 164, v211, 33792
SIL 12, 2, 2
SIL 12, 2, 3
SIL 12, 2, 4
STADR 12
MM 8, 1, a, 192, 144
SIL 12, 2, 5
SIL 12, 2, 6
SIL 12, 2, 7
v_accvgpr_read_b32 v49, a113
MM 12, 1, a, 192, 152
STORE 12
MM 28, 1, a, 200, 152
RD 168, v211, 34816
RD 172, v211, 35840
v_accvgpr_read_b32 v53, a117
v_accvgpr_read_b32 v57, a121
v_accvgpr_read_b32 v61, a125
SIL 13, 0, 0
SIL 13, 0, 1
SIL 13, 0, 2
SIL 13, 0, 3
SIL 13, 0, 4
MM 24, 1, a, 200, 144
SIL 13, 0, 5
SIL 13, 0, 6
SIL 13, 0, 7
MM 20, 1, a, 200, 136
SIL 13, 1, 0
SIL 13, 1, 1
SIL 13, 1, 2
SIL 13, 1, 3
MM 16, 1, a, 200, 128
RD 176, v211, 36864
RD 180, v211, 37888
SIL 13, 1, 4
SIL 13, 1, 5
SIL 13, 1, 6
SIL 13, 1, 7
MM 32, 1, a, 208, 128
RD 184, v211, 38912
SIL 13, 2, 0
SIL 13, 2, 1
SIL 13, 2, 2
SIL 13, 2, 3
MM 36, 1, a, 208, 136
SIL 13, 2, 4
STADR 13
SIL 13, 2, 5
SIL 13, 2, 6
MM 40, 1, a, 208, 144
SIL 13, 2, 7
v_accvgpr_read_b32 v50, a114
v_accvgpr_read_b32 v54, a118
MM 44, 1, a, 208, 152
RD 188, v211, 39936
STORE 13
MM 60, 1, a, 216, 152
v_accvgpr_read_b32 v58, a122
v_accvgpr_read_b32 v62, a126
SIL 14, 0, 0
SIL 14, 0, 1
SIL 14, 0, 2
SIL 14, 0, 3
SIL 14, 0, 4
SIL 14, 0, 5
MM 56, 1, a, 216, 144
SIL 14, 0, 6
SIL 14, 0, 7
SIL 14, 1, 0
SIL 14, 1, 1
MM 52, 1, a, 216, 136
SIL 14, 1, 2
SIL 14, 1, 3
SIL 14, 1, 4
SIL 14, 1, 5
MM 48, 1, a, 216, 128
SIL 14, 1, 6
SIL 14, 1, 7
SIL 14, 2, 0
s_waitcnt vmcnt(11) lgkmcnt(0)
MM 0, 1, a, 224, 160
SIL 14, 2, 1
SIL 14, 2, 2
SIL 14, 2, 3
SIL 14, 2, 4
MM 4, 1, a, 224, 168
STADR 14
SIL 14, 2, 5
SIL 14, 2, 6
SIL 14, 2, 7
MM 8, 1, a, 224, 176
STORE 14
MM 12, 1, a, 224, 184
v_accvgpr_read_b32 v51, a115
v_accvgpr_read_b32 v55, a119
v_accvgpr_read_b32 v59, a123
v_accvgpr_read_b32 v63, a127
SIL 15, 0, 0
SIL 15, 0, 1
SIL 15, 0, 2
SIL 15, 0, 3
s_waitcnt vmcnt(10)
MM 28, 1, a, 232, 184
SIL 15, 0, 4
SIL 15, 0, 5
SIL 15, 0, 6
MM 24, 1, a, 232, 176
SIL 15, 0, 7
SIL 15, 1, 0
SIL 15, 1, 1
SIL 15, 1, 2
MM 20, 1, a, 232, 168
SIL 15, 1, 3
SIL 15, 1, 4
SIL 15, 1, 5
SIL 15, 1, 6
MM 16, 1, a, 232, 160
SIL 15, 1, 7
SIL 15, 2, 0
SIL 15, 2, 1
SIL 15, 2, 2
s_waitcnt vmcnt(8)
MM 32, 1, a, 240, 160
SIL 15, 2, 3
SIL 15, 2, 4
STADR 15
SIL 15, 2, 5
MM 36, 1, a, 240, 168
SIL 15, 2, 6
SIL 15, 2, 7
MM 40, 1, a, 240, 176
STORE 15
MM 44, 1, a, 240, 184
s_waitcnt vmcnt(6)
MM 60, 1, a, 248, 184
MM 56, 1, a, 248, 176
MM 52, 1, a, 248, 168
v_readlane_b32 s62, v205, 4
v_readlane_b32 s63, v205, 5
MM 48, 1, a, 248, 160
s_mov_b64 s[46:47], s[62:63]
.Lfin%=:
  s_nop 15
  LAST4 0
  LAST4 1
  LAST4 2
  LAST4 3
  s_waitcnt vmcnt(0) lgkmcnt(0)
.Lend%=:
.purgem SIL
.purgem MM
.purgem CPY
.purgem STORE
.purgem STADR
.purgem CHKF
.purgem DUP
.purgem DUPO
.purgem SETW
.purgem LDW
.purgem LDX
.purgem RD
.purgem SLOW
.purgem SIL8
.purgem LAST
.purgem LAST4
)"
    :: "v"(voffx), "v"(voffw), "v"(boff), "v"(ooff), "v"(tab), "v"(fbase), "s"(rw), "s"(ro),
       "s"(mw), "v"(c2), "v"(is), "s"(call), "s"(np), "s"(dbase), "s"(fwd), "s"(pc0)
    : GEMM_CLOBBERS);
}
static_assert(SLAB == 2621440 && TMO_OFF == 32768 && DEADLINE == 30000000u && NPC * 4 * 8 == 10240 && N == 16384, "constants spelled out in gemm_loop");

__global__ __launch_bounds__(256, 1) void fused_kernel(const Params p) {
  extern __shared__ __attribute__((aligned(16))) char shm_[];
  LAS char* lds = (LAS char*)shm_;
  const int tid = threadIdx.x, wid = rfl(tid >> 6), lane = tid & 63, bid = blockIdx.x;
  const unsigned cnt = CNT_OFF + bid * 128;
  unsigned cv; unsigned long long pc0;
  asm volatile("s_getpc_b64 %0" : "=s"(pc0));
  const int xc = bid & 7, slot = bid >> 3, nc = (xc & 1) ? 4 - CEVEN : CEVEN;
  if (__builtin_expect(slot < nc, 1)) {
    cv = push_role(p, lds, (xc >> 1) * 4 + ((xc & 1) ? CEVEN : 0) + slot, wid, lane, cnt);
    __syncthreads();
  } else {
    char* ws = p.me;
    cv = __hip_atomic_load((unsigned*)(ws + cnt), __ATOMIC_RELAXED, __HIP_MEMORY_SCOPE_AGENT);
    const int idx = ((xc & 1) ? 32 - CEVEN : 0) + slot - nc;
    const int id = wid * (GRID - NCOMM) + idx * 4 + (xc >> 1);
    for (int d = RBASE + (idx & 15) * RSTEP; d > 0; --d) __builtin_amdgcn_s_sleep(8);
    if (id < 2 * NPC) retile(p, ws, lds + wid * 2 * PIECE, id / (NPC / 2), id % (NPC / 2) * 2, lane, cv);
    const int row = (xc >> 1) * 4 + (idx & 3), cs = idx >> 2, ng = ngroups(cs), rank = p.rank;
    for (int e = tid; e < 24 * 16; e += 256) {
      const int pi = e >> 4, f = e & 15;
      auto grp = [&](int slot) { const int i = pi - ((pi - ng + 1 + slot) & 3); return (i >= 0 && i < ng) ? group(cs, i, rank) : -1; };
      unsigned v = 0;
      if (f < 4) v = (row * 4 + f) * 32;
      else if (f < 8) { const int g = grp(f - 4); v = (g >= 0 ? 64 + g : row * 4) * 32; }
      else if (f < 12) { const int g = pi < 3 ? group(cs, (ng - 1 - (f - 8)) & 3, rank) : grp(f - 8); v = g >= 0 ? (unsigned)(64 + g) * PIECE : OOB; }
      else if (f == 12) { unsigned idle = 0; for (int s = 0; s < 4; ++s) idle |= (grp(s) < 0 ? 1u : 0u) << s;
        unsigned first = pi < ng ? 1u << ((ng - 1 - pi) & 3) : 0u, skip0 = idle;
        if (pi < 4) { first = pi == 0 ? 15u : 0u; skip0 = 0; }
        else for (int i = pi - 3; i < 4; ++i) skip0 |= 1u << ((ng - 1 - i) & 3);
        v = (skip0 | first) | idle << 4 | first << 12 | (cs < 2 && pi == IO + 4 ? 1u : 0u) << 16; }
      else if (f == 13) v = (pi >= 4 && pi - 4 < ng) ? (unsigned)(row * 256 * N + group(cs, pi - 4, rank) * 64) * 4u : OOB;
      else if (f == 14) v = row * 4 * PIECE;
      else v = (cs < 2 && pi == IO + 5) ? SIG_OFF + (row * 2 + cs) * 16 : OOB;
      *(LAS unsigned*)(lds + TAB_OFF + e * 4) = v;
    }
    if (idx < 2 && tid < (int)(sizeof(d_send.e) / 64)) { const unsigned v = d_send.e[tid * 16]; asm volatile("" :: "v"(v)); }
    const float s = p.sx[0] * p.sw[0];
    const unsigned call = rfl(cv) + 1u;
    float is = 1.0f / s;
    if (wid == 0) {
      const unsigned t0 = now();
      const unsigned* f = (const unsigned*)(ws + FLAG_OFF) + ((lane & 3) * NPC + 64 + group(cs, (lane >> 2) & 3, rank)) * 8;
      for (int d = 6; d > 0; --d) __builtin_amdgcn_s_sleep(32);
      while (!__all(__hip_atomic_load(f, __ATOMIC_RELAXED, __HIP_MEMORY_SCOPE_SYSTEM) >= call)) {
        if (rfl(__hip_atomic_load((unsigned*)(ws + TMO_OFF), __ATOMIC_RELAXED, __HIP_MEMORY_SCOPE_AGENT)) == call || now() - t0 > DEADLINE) {
          __hip_atomic_store((unsigned*)(ws + TMO_OFF), call, __ATOMIC_RELAXED, __HIP_MEMORY_SCOPE_AGENT); is = __builtin_nanf(""); break; }
        __builtin_amdgcn_s_sleep(32);
      }
    }
    __syncthreads();
    gemm_loop(lane * 16 + wid * 8192, lane * 16 + wid * 2048, lane * 16, ((wid * 64 + (lane >> 4) * 4) * N + (lane & 15) * 4) * 4,
              TAB_OFF + (lane & 7) * 4, FLAG_OFF + (lane >> 3) * 4,
              __builtin_amdgcn_make_buffer_rsrc(ws, 0, WS_USED, 0x00020000), __builtin_amdgcn_make_buffer_rsrc(p.out, 0, (unsigned)M * N * 4u, 0x00020000),
              wid * 2048, s * -1.44269504089f, is, call, rfl(ng), DATA_OFF + (call & 1u) * 32u * SLAB, FWD_OFF + ((row * 2 + (cs & 1)) * 4 + wid) * 16384, pc0);
  }
  const unsigned call = rfl(cv) + 1u;
  if (wid == 0 && __builtin_amdgcn_mbcnt_hi(~0u, __builtin_amdgcn_mbcnt_lo(~0u, 0u)) == 0) {
    __hip_atomic_store((unsigned*)(p.me + cnt), call, __ATOMIC_RELAXED, __HIP_MEMORY_SCOPE_AGENT);
  }
}
}

extern "C" void kernel_launch(int rank, int nranks, ncclComm_t comm, hipStream_t stream,
                              void* const* d_in, const int* in_sizes, int n_in,
                              void* d_out, int out_size, const int* in_dtypes, int out_dtype,
                              void* const* d_out_all, void* const* d_ws_all, size_t ws_bytes) {
  if (nranks != NR || (unsigned)rank >= (unsigned)NR || n_in != 4 || ws_bytes < WS_USED || in_sizes[0] != M * KS || in_sizes[1] != KS * N || out_size != M * N) {
    fprintf(stderr, "unexpected problem shape\n"); return; }
  HIP_CHECK(hipFuncSetAttribute((const void*)fused_kernel, hipFuncAttributeMaxDynamicSharedMemorySize, LDS_BYTES));
  Params p{};
  p.x = (const unsigned char*)d_in[0]; p.w = (const unsigned char*)d_in[1]; p.sx = (const float*)d_in[2]; p.sw = (const float*)d_in[3];
  p.out = (float*)d_out; p.me = (char*)d_ws_all[rank]; p.rank = rank;
  for (int j = 0; j < 13; ++j) { p.dst[j] = (char*)d_ws_all[(rank + j % 7 + 1) % NR] + (size_t)(7 - j % 7) * SLAB; p.odst[j] = (char*)d_out_all[(rank + j % 7 + 1) % NR]; }
  hipLaunchKernelGGL(fused_kernel, dim3(GRID), dim3(256), LDS_BYTES, stream, p);
  HIP_CHECK(hipGetLastError());
}
```
